```python
import numpy as np

kernel_name = "gpu_ag_gemm_m2048_k2048_n2048_f32_none"


ABSMAX_RTOL = 0.001

COMPUTE_DTYPE = "float32"
EPILOGUE = "none"
IN_QUANT = None
OUT_QUANT = None
IN_DTYPES = {"x": "float32", "w_mat": "float32"}
WORLD_SIZE = 8

def setup_inputs():

    x = np.random.randn(2048, 2048).astype(np.float32)
    w_mat = (np.random.randn(2048, 2048) / np.sqrt(2048)).astype(np.float32)
    return {"x": x, "w_mat": w_mat}

def _epilogue(y):
    return y

def reference(x, w_mat):
    y = (x @ w_mat).astype(np.float32)
    return _epilogue(y)
```

```cpp
#include <hip/hip_runtime.h>
#include <rccl/rccl.h>
#include <cstdio>
#include <cstdlib>
typedef unsigned v4u __attribute__((ext_vector_type(4)));
typedef unsigned v2u __attribute__((ext_vector_type(2)));
typedef float v4f __attribute__((ext_vector_type(4)));
typedef _Float16 v8h __attribute__((ext_vector_type(8)));
typedef _Float16 v2h __attribute__((ext_vector_type(2)));
#define WINI 6
#define WINC 8
#define PS 150
#define PDI 5600
#define PDW 5600
#define PDX 5600
#define PSC PS
#define TY0 0
#define DY 1390
#define YPOLL 0
#define YPT 1700
#define XA 3
#define RPUB 9
#define NPUB 32
#define EPOCH 1
#define CATCH 90
#define YORD 1
#define YOFF 0
#define ZDELAY 300
#define ZSLP 2
#define ZWIN 1
#define NOZERO 0
#define SLP 2
#define FI 12
#define BT0 4
#define BT1 4
#define SPEC 1
#define WARM 1
#define STAMPS 0
#define DETAIL 0
#define FC (16 - FI)
#define NE ((FI + 3 * BT0) / 4)
constexpr int KD = 2048, NS = 256, MS = 256;
constexpr size_t SLOT = 8u << 20;
constexpr size_t DATA_OFF = 1u << 20;
constexpr size_t B_OFF = 4u << 20, Y_OFF = 6u << 20;
constexpr size_t G_OFF = 7u << 20;
constexpr int ZERO_KB = 7168 + 5;
constexpr size_t DBG_BYTES = 256 * 1024;
constexpr unsigned long long DEADLINE = 20000000ull;
constexpr int NT = 512;
constexpr int LDS_BYTES = 131072 + 4096;
struct Args {
  char* me;
  char* peer[7];
  const float* x; const float* w; float* out;
  unsigned long long* dbg;
  unsigned tab[16];
  int rank;
};
__device__ __forceinline__ unsigned long long rt() { return __builtin_amdgcn_s_memrealtime(); }
#define STR2(x) #x
#define STR(x) STR2(x)
__device__ __forceinline__ void st16_sys(char* p, v4u v) { asm volatile("global_store_dwordx4 %0, %1, off sc0 sc1\n\ts_nop 1" :: "v"(p), "v"(v) : "memory"); }
#define LDMOD " sc0 sc1"
#define L4M(a, b, c, d, base) "global_load_dwordx4 %" #a ", %17, %" #base LDMOD "\n\tglobal_load_dwordx4 %" #b ", %17, %" #base " offset:1024" LDMOD \
  "\n\tglobal_load_dwordx4 %" #c ", %17, %" #base " offset:2048" LDMOD "\n\tglobal_load_dwordx4 %" #d ", %17, %" #base " offset:3072" LDMOD "\n\t"
__device__ __forceinline__ void ld16m(v4u* f, unsigned& h, unsigned loff, const char* b0, const char* b1, const char* b2, const char* b3, unsigned hoff, const char* hb) {
  asm volatile("s_nop 4\n\t" L4M(0, 1, 2, 3, 18) L4M(4, 5, 6, 7, 19) L4M(8, 9, 10, 11, 20) L4M(12, 13, 14, 15, 21) "global_load_dword %16, %22, %23" LDMOD "\n\ts_waitcnt vmcnt(0)"
               : "=&v"(f[0]), "=&v"(f[1]), "=&v"(f[2]), "=&v"(f[3]), "=&v"(f[4]), "=&v"(f[5]), "=&v"(f[6]), "=&v"(f[7]),
                 "=&v"(f[8]), "=&v"(f[9]), "=&v"(f[10]), "=&v"(f[11]), "=&v"(f[12]), "=&v"(f[13]), "=&v"(f[14]), "=&v"(f[15]), "=&v"(h)
               : "v"(loff), "s"(b0), "s"(b1), "s"(b2), "s"(b3), "v"(hoff), "s"(hb) : "memory");
}
#define L4N(a, b, c, d, base) "global_load_dwordx4 %" #a ", %9, %" #base LDMOD "\n\tglobal_load_dwordx4 %" #b ", %9, %" #base " offset:1024" LDMOD \
  "\n\tglobal_load_dwordx4 %" #c ", %9, %" #base " offset:2048" LDMOD "\n\tglobal_load_dwordx4 %" #d ", %9, %" #base " offset:3072" LDMOD "\n\t"
__device__ __forceinline__ void ld8m(v4u* f, unsigned& h, unsigned loff, const char* b0, const char* b1, unsigned hoff, const char* hb) {
  asm volatile("s_nop 4\n\t" L4N(0, 1, 2, 3, 10) L4N(4, 5, 6, 7, 11) "global_load_dword %8, %12, %13" LDMOD "\n\ts_waitcnt vmcnt(0)"
               : "=&v"(f[0]), "=&v"(f[1]), "=&v"(f[2]), "=&v"(f[3]), "=&v"(f[4]), "=&v"(f[5]), "=&v"(f[6]), "=&v"(f[7]), "=&v"(h)
               : "v"(loff), "s"(b0), "s"(b1), "v"(hoff), "s"(hb) : "memory");
}
__device__ __forceinline__ unsigned min4(unsigned m, v4u f) { m = min(min(m, f.x), f.y); return min(min(m, f.z), f.w); }
template <int N> __device__ __forceinline__ bool all_ok(const v4u* f) {
  unsigned m = 0xffffffffu;
#pragma unroll
  for (int i = 0; i < N; ++i) m = min4(m, f[i]);
  return __all(m != 0);
}
__device__ __forceinline__ unsigned pkz(float a, float b) {
  v2h h; h.x = (_Float16)a; h.y = (_Float16)b; unsigned u = __builtin_bit_cast(unsigned, h); return u == 0 ? 0x8000u : u; }
__device__ __forceinline__ unsigned ld_sys(const void* p) { return __hip_atomic_load((const unsigned*)p, __ATOMIC_RELAXED, __HIP_MEMORY_SCOPE_SYSTEM); }
__device__ __forceinline__ void touch(const void* p) { unsigned v; asm volatile("global_load_dword %0, %1, off sc0 sc1\n\ts_waitcnt vmcnt(0)" : "=&v"(v) : "v"(p) : "memory"); }
__device__ __forceinline__ bool wait16(v4u* f, unsigned& h, unsigned loff, const char* b0, const char* b1, const char* b2, const char* b3,
                                       const char* hp, unsigned hoff, const char* hb, unsigned long long t0) {
  for (;;) {
    int it = 0;
    while (!__all(ld_sys(hp) != 0) && ++it < 64) __builtin_amdgcn_s_sleep(SLP);
    ld16m(f, h, loff, b0, b1, b2, b3, hoff, hb);
    if (all_ok<16>(f)) return true;
    if (rt() - t0 > DEADLINE) return false;
  }
}
__host__ __device__ __forceinline__ int idx_row(int G, int idx) {
  if (idx < 4 * FI) return (G * 4 + idx / FI) * MS + (idx % FI) * 16;
  return ((G ^ 1) * 4 + (idx - 4 * FI) / FC) * MS + (FI + (idx - 4 * FI) % FC) * 16;
}
__device__ __forceinline__ int rot_ks(int kpos, int rb) { return (kpos + 16 * (rb & 3)) & 63; }
__device__ __forceinline__ void intra_order(int f, int& rb, int& ks) {
  constexpr int B2 = FI - BT0 - BT1;
  if (f < BT0 * 64) { ks = f / BT0; rb = f % BT0; }
  else if (f < (BT0 + BT1) * 64) { f -= BT0 * 64; ks = f / BT1; rb = BT0 + f % BT1; }
  else { f -= (BT0 + BT1) * 64; ks = f / (B2 > 0 ? B2 : 1); rb = BT0 + BT1 + f % (B2 > 0 ? B2 : 1); }
  ks = rot_ks(ks, rb);
}
struct XL { v4f p0, p1; };
__device__ __forceinline__ XL x_load(const float* x, int rb, int ks, int lane) {
  const float* src = x + (size_t)(rb * 16 + (lane >> 3)) * KD + ks * 32 + (lane & 7) * 4;
  XL r; r.p0 = *(const v4f*)src; r.p1 = *(const v4f*)(src + 8 * KD); return r;
}
__device__ __forceinline__ void x_stage(char* frag, const XL& r, int lane) {
  char* ldst = frag + ((lane >> 3) + 16 * ((lane & 7) >> 1)) * 16 + (lane & 1) * 8;
  v2u v0, v1; v0.x = pkz(r.p0.x, r.p0.y); v0.y = pkz(r.p0.z, r.p0.w); v1.x = pkz(r.p1.x, r.p1.y); v1.y = pkz(r.p1.z, r.p1.w);
  *(v2u*)ldst = v0; *(v2u*)(ldst + 128) = v1;
}
__device__ __forceinline__ void pace(unsigned long long t0, unsigned when100) {
  const unsigned long long until = t0 + when100 / 100;
  while (rt() < until) __builtin_amdgcn_s_sleep(1);
}
__device__ __forceinline__ int gate_read(const char* slot, unsigned long long t0, int lane) {
  const unsigned e = __hip_atomic_load((const unsigned*)(slot + G_OFF + (lane & (NPUB - 1)) * 128), __ATOMIC_RELAXED, __HIP_MEMORY_SCOPE_AGENT);
  if (!__all(e != 0)) return -1;
  int rel = (int)(e - (unsigned)t0);
#pragma unroll
  for (int m = 1; m < NPUB; m <<= 1) rel = max(rel, __shfl_xor(rel, m));
  return min(max(__builtin_amdgcn_readfirstlane(rel) + YOFF, 0), 100000);
}
#define MFMA(b, a, c) __builtin_amdgcn_mfma_f32_16x16x32_f16(__builtin_bit_cast(v8h, b), __builtin_bit_cast(v8h, a), c, 0, 0, 0)

#define ST(i)
#define PST(i)
#define FLUSH()
#define PFLUSH()

template <int NB> __device__ __forceinline__ void mma_task(const v4u* B, const v4u* f, char* sl, bool dead) {
  v4f acc[2 * NB];
#pragma unroll
  for (int c = 0; c < 2 * NB; ++c) acc[c] = (v4f){0, 0, 0, 0};
#pragma unroll
  for (int k = 0; k < 8; ++k)
#pragma unroll
    for (int c = 0; c < 2 * NB; ++c) acc[c] = MFMA(B[(c >> 1) * 16 + 2 * k + (c & 1)], f[k], acc[c]);
#pragma unroll
  for (int c = 0; c < 2 * NB; ++c) {
    if (dead) acc[c] = (v4f){__builtin_nanf(""), __builtin_nanf(""), __builtin_nanf(""), __builtin_nanf("")};
    *(v4f*)(sl + c * 1024) = acc[c];
  }
}

__device__ __forceinline__ void run_pair(const v4u* B, char* sl, const char* abase, unsigned tb, int lane, unsigned long long t0, bool& dead
) {
  const unsigned loff = lane * 16;
  const unsigned hoff = ((tb >> (8 * (lane & 1))) & 63u) * 65536u + 8188u;
  const char* a0 = abase + (size_t)(tb & 63) * 65536;
  const char* a1 = abase + (size_t)((tb >> 8) & 63) * 65536;
  unsigned pending = 3, h = 0;
  while (pending) {
    unsigned c = pending;
    if (pending == 3 || !SPEC) {
      c = (dead ? 3u : (unsigned)__ballot(ld_sys(abase + hoff) != 0)) & pending;
      if (!c) { __builtin_amdgcn_s_sleep(SLP); if (rt() - t0 > DEADLINE) dead = true; continue; }
    }
    v4u f[16];
    if (c == 3) {
      ld16m(f, h, loff, a0, a0 + 4096, a1, a1 + 4096, hoff, abase);
      if (dead || all_ok<8>(f + 8)) { mma_task<2>(B, f + 8, sl + 32768, dead); pending &= ~2u; ST(5); }
    } else { const char* ap = c == 1 ? a0 : a1; ld8m(f, h, loff, ap, ap + 4096, hoff, abase); }
    if (dead || all_ok<8>(f)) { mma_task<2>(B, f, sl + (c == 2 ? 32768 : 0), dead); pending &= ~(c == 2 ? 2u : 1u); ST(c == 2 ? 5 : 4); }
    else if (rt() - t0 > DEADLINE) dead = true;
  }
}
#define LSTARG

extern "C" __global__ void __launch_bounds__(NT) agk(Args a) {
  extern __shared__ __attribute__((aligned(16))) char lds[];
  const int tid = threadIdx.x, lane = tid & 63, wave = __builtin_amdgcn_readfirstlane(tid >> 6), b = blockIdx.x;
  unsigned long long t0 = rt();
  const int G = a.rank >> 2, gi4 = a.rank & 3;
  char* me = a.me;
  unsigned* ctr = (unsigned*)(me + 65536) + b * 16 + wave;
  unsigned* tmo = (unsigned*)(me + 4096);
  const unsigned cprev = *ctr;
  const unsigned loff = lane * 16;
  if (b < 64) {
    const int u = b & 31;
    const bool intra = b < 32;
    const int NX = intra ? 2 * FI : 2 * FC;
    unsigned* lflag = (unsigned*)(lds + 131072);
    auto item = [&](int j, int& rb, int& ks) __attribute__((always_inline)) {
      if (intra) intra_order(u + 32 * j, rb, ks);
      else { const int f = u + 32 * j; rb = FI + f % FC; ks = rot_ks(f / FC, f % FC); }
    };
    if (wave == 0) {
      int rb, ks; item(0, rb, ks);
      const float* src = a.x + (size_t)(rb * 16 + (lane & 15)) * KD + ks * 32 + (lane >> 4) * 8;
      v4f p0 = *(const v4f*)src, p1 = *(const v4f*)(src + 4);
      if (lane < 32) lflag[lane] = 0;
      asm volatile("s_waitcnt lgkmcnt(0)\n\ts_barrier" ::: "memory");
      const unsigned call = __builtin_amdgcn_readfirstlane(cprev) + 1;
      const size_t sl = DATA_OFF + (size_t)(call % 3) * SLOT;
      v4u v; v.x = pkz(p0.x, p0.y); v.y = pkz(p0.z, p0.w); v.z = pkz(p1.x, p1.y); v.w = pkz(p1.z, p1.w);
      *(v4u*)(lds + lane * 16) = v;
      asm volatile("s_waitcnt lgkmcnt(0)" ::: "memory");
      if (lane == 0) __hip_atomic_store(lflag, 1u, __ATOMIC_RELAXED, __HIP_MEMORY_SCOPE_WORKGROUP);
      ST(5);
      unsigned late = 0;
      if (intra) {
        char* dst[3];
#pragma unroll
        for (int d = 0; d < 3; ++d) dst[d] = a.peer[(d + b) % 3] + sl + (size_t)(gi4 * FI * 64) * 1024 + lane * 16;
        unsigned when = PS * 100 + u * (PDI / 32);
#pragma unroll 1
        for (int j = 0; j < 2 * FI; ++j) {
          if (j > 0) {
            while (__hip_atomic_load(lflag + j, __ATOMIC_RELAXED, __HIP_MEMORY_SCOPE_WORKGROUP) == 0) __builtin_amdgcn_s_sleep(1);
            asm volatile("" ::: "memory");
            v = *(const v4u*)(lds + j * 1024 + lane * 16);
            item(j, rb, ks);
          }
          const size_t o = (size_t)(rb * 64 + ks) * 1024;
          if (EPOCH && !late) late = __builtin_amdgcn_readfirstlane(__hip_atomic_load(lflag + 25, __ATOMIC_RELAXED, __HIP_MEMORY_SCOPE_WORKGROUP));
          pace(t0 - late, when); when += PDI;
          asm volatile("s_waitcnt vmcnt(" STR(WINI) ")" ::: "memory");
          PST(j);
#pragma unroll
          for (int d = 0; d < 3; ++d) st16_sys(dst[d] + o, v);
        }
      } else {
        char* dst[4]; int lo[4];
#pragma unroll
        for (int d = 0; d < 4; ++d) { const int j = (d + b) & 3; dst[d] = a.peer[3 + j] + sl + lane * 16; lo[d] = 32768 + j * 4096 + lane * 16; }
        unsigned when = PSC * 100 + u * (PDW / 32);
#pragma unroll 1
        for (int r = 0; r < 2 * FC + 8; ++r) {
          v4u vv[4]; size_t o;
          if (r >= XA && r < XA + 8) {
            const int n = r - XA, t = n >> 2, kk = (n >> 1) & 1, cb = n & 1;
            if (n == 0) { while (__hip_atomic_load(lflag + 24, __ATOMIC_RELAXED, __HIP_MEMORY_SCOPE_WORKGROUP) < 8) __builtin_amdgcn_s_sleep(1); asm volatile("" ::: "memory"); }
#pragma unroll
            for (int d = 0; d < 4; ++d) vv[d] = *(const v4u*)(lds + (kk * 16 + t * 2 + cb) * 1024 + lo[d]);
            o = B_OFF + (size_t)(8 + gi4 * 2 + t) * 131072 + (size_t)((2 * u + kk) * 2 + cb) * 1024;
          } else {
            const int j = r < XA ? r : r - 8;
            if (j > 0) {
              while (__hip_atomic_load(lflag + j, __ATOMIC_RELAXED, __HIP_MEMORY_SCOPE_WORKGROUP) == 0) __builtin_amdgcn_s_sleep(1);
              asm volatile("" ::: "memory");
              v = *(const v4u*)(lds + j * 1024 + lane * 16);
              item(j, rb, ks);
            }
            vv[0] = vv[1] = vv[2] = vv[3] = v;
            o = (size_t)((4 * FI + gi4 * FC + rb - FI) * 64 + ks) * 1024;
          }
          if (EPOCH && !late) late = __builtin_amdgcn_readfirstlane(__hip_atomic_load(lflag + 25, __ATOMIC_RELAXED, __HIP_MEMORY_SCOPE_WORKGROUP));
          pace(t0 - late, when); when += PDW;
          asm volatile("s_waitcnt vmcnt(" STR(WINC) ")" ::: "memory");
          PST(r);
          if (TY0 < 0 && r == RPUB && u < NPUB) {
            const unsigned sched = (unsigned)(t0 - late) + PSC + (unsigned)((2 * FC + 8) * PDW) / 100;
            const unsigned pred = (unsigned)rt() + (unsigned)((2 * FC + 8 - RPUB) * (PDW / 100 * CATCH) - u * (PDW / 32)) / 100;
            const unsigned te = (int)(pred - sched) > 0 ? pred : sched;
            if (lane == 0) __hip_atomic_store((unsigned*)(me + sl + G_OFF + u * 128), te | 1u, __ATOMIC_RELAXED, __HIP_MEMORY_SCOPE_AGENT);
          }
#pragma unroll
          for (int d = 0; d < 4; ++d) st16_sys(dst[d] + o, vv[d]);
        }
      }
      if (lane == 0) *ctr = call;
      PFLUSH();
      FLUSH();
      return;
    }
    {
      constexpr int NI = (2 * FI + 5) / 7;
      XL r[NI];
      v4f q[8], q2[8];
      { int rb, ks; item(wave, rb, ks); r[0] = x_load(a.x, rb, ks, lane); }
      asm volatile("s_barrier" ::: "memory");
      if (intra) {
#pragma unroll
        for (int i = 1; i < NI; ++i) { const int j = min(wave + 7 * i, NX - 1); int rb, ks; item(j, rb, ks); r[i] = x_load(a.x, rb, ks, lane); }
      } else {
        const float* wsrc = a.w + (size_t)(u * 64 + wave * 8) * NS + lane * 4;
#pragma unroll
        for (int j = 0; j < 8; ++j) q[j] = *(const v4f*)(wsrc + j * NS);
        if (wave == 7) {
#pragma unroll
          for (int j = 0; j < 8; ++j) q2[j] = *(const v4f*)(wsrc + (j - 56) * NS);
        } else { const int j = min(wave + 7, NX - 1); int rb, ks; item(j, rb, ks); r[1] = x_load(a.x, rb, ks, lane); }
      }
      auto put = [&](int j, const XL& x) __attribute__((always_inline)) {
        x_stage(lds + j * 1024, x, lane);
        asm volatile("s_waitcnt lgkmcnt(0)" ::: "memory");
        if (lane == 0) __hip_atomic_store(lflag + j, 1u, __ATOMIC_RELAXED, __HIP_MEMORY_SCOPE_WORKGROUP);
      };
      auto putw = [&](int oct, const v4f* qq) __attribute__((always_inline)) {
        char* dstl = lds + 32768 + ((oct >> 2) * 16 + (lane >> 2)) * 1024 + ((oct & 3) * 16 + (lane & 3) * 4) * 16;
#pragma unroll
        for (int c = 0; c < 4; ++c) {
          v4u o; o.x = pkz(qq[0][c], qq[1][c]); o.y = pkz(qq[2][c], qq[3][c]); o.z = pkz(qq[4][c], qq[5][c]); o.w = pkz(qq[6][c], qq[7][c]);
          *(v4u*)(dstl + c * 16) = o;
        }
        asm volatile("s_waitcnt lgkmcnt(0)" ::: "memory");
        if (lane == 0) __hip_atomic_fetch_add(lflag + 24, 1u, __ATOMIC_RELAXED, __HIP_MEMORY_SCOPE_WORKGROUP);
      };
      if (intra) {
#pragma unroll
        for (int i = 0; i < NI; ++i) put(min(wave + 7 * i, NX - 1), r[i]);
      } else {
        put(wave, r[0]);
        putw(wave, q);
        if (wave == 7) putw(0, q2); else put(min(wave + 7, NX - 1), r[1]);
      }
    }
    ST(4);
    const unsigned call = __builtin_amdgcn_readfirstlane(cprev) + 1;
    if (lane == 0) *ctr = call;
    const size_t sl = DATA_OFF + (size_t)(call % 3) * SLOT;
    char* slot = me + sl;
    if (EPOCH && wave == 1) {
      unsigned e = 0;
      for (int it = 0; it < 8 && e == 0; ++it) e = __hip_atomic_load((const unsigned*)(slot + G_OFF + 32 * 128), __ATOMIC_RELAXED, __HIP_MEMORY_SCOPE_AGENT);
      const int d = (int)((unsigned)t0 - e);
      if (e != 0 && d > 0 && d < 200 && lane == 0) __hip_atomic_store(lflag + 25, (unsigned)d, __ATOMIC_RELAXED, __HIP_MEMORY_SCOPE_WORKGROUP);
    }
    if (intra) {
      for (int j = wave - 1; j < 2 * FI; j += 7) {
        while (__hip_atomic_load(lflag + j, __ATOMIC_RELAXED, __HIP_MEMORY_SCOPE_WORKGROUP) == 0) __builtin_amdgcn_s_sleep(1);
        asm volatile("" ::: "memory");
        int rb, ks; item(j, rb, ks);
        st16_sys(slot + (size_t)((gi4 * FI + rb) * 64 + ks) * 1024 + lane * 16, *(const v4u*)(lds + j * 1024 + lane * 16));
      }
    } else {
      while (__hip_atomic_load(lflag + 24, __ATOMIC_RELAXED, __HIP_MEMORY_SCOPE_WORKGROUP) < 8) __builtin_amdgcn_s_sleep(1);
      asm volatile("" ::: "memory");
      for (int n = wave - 1; n < 32; n += 7) {
        const int kk = n >> 4, ct = (n >> 1) & 7, cb = n & 1;
        st16_sys(slot + B_OFF + (size_t)ct * 131072 + (size_t)((2 * u + kk) * 2 + cb) * 1024 + lane * 16, *(const v4u*)(lds + 32768 + n * 1024 + lane * 16));
      }
    }
    const int ax = b + 64 * (wave - 1);
    float* o = a.out + (size_t)(idx_row(G ^ 1, (ax >> 2) & 63) + (lane & 15)) * NS + (ax & 3) * 64 + (lane >> 4) * 4;
    if (WARM && ax < 256) touch(o);
    char* z = me + DATA_OFF + (size_t)((call + 2) % 3) * SLOT + lane * 16;
    v4u zero = {0, 0, 0, 0};
    while (rt() - t0 < ZDELAY) __builtin_amdgcn_s_sleep(8);
    for (int k = ax; k < (NOZERO ? 0 : ZERO_KB); k += 448) { st16_sys(z + (size_t)k * 1024, zero); asm volatile("s_waitcnt vmcnt(" STR(ZWIN) ")" ::: "memory"); __builtin_amdgcn_s_sleep(ZSLP); }
    ST(1);
    if (ax < 256) {
      const char* ysrc = slot + Y_OFF + (size_t)ax * 4096;
      v4u y[4]; bool dead = false;
      if (YPOLL == 2) pace(t0, YPT * 100);
      for (;;) {
        if (YPOLL == 1) { int it = 0; while (!__all(ld_sys(ysrc + (lane & 1) * 2048 + 2044) != 0) && ++it < 64) __builtin_amdgcn_s_sleep(SLP); }
        asm volatile("s_nop 4\n\tglobal_load_dwordx4 %0, %4, %5" LDMOD "\n\tglobal_load_dwordx4 %1, %4, %5 offset:1024" LDMOD
                     "\n\tglobal_load_dwordx4 %2, %4, %5 offset:2048" LDMOD "\n\tglobal_load_dwordx4 %3, %4, %5 offset:3072" LDMOD "\n\ts_waitcnt vmcnt(0)"
                     : "=&v"(y[0]), "=&v"(y[1]), "=&v"(y[2]), "=&v"(y[3]) : "v"(loff), "s"(ysrc) : "memory");
        if (all_ok<4>(y)) break;
        __builtin_amdgcn_s_sleep(SLP);
        if (rt() - t0 > DEADLINE) { __hip_atomic_store(tmo, call, __ATOMIC_RELAXED, __HIP_MEMORY_SCOPE_AGENT); dead = true; break; }
      }
#pragma unroll
      for (int c = 0; c < 4; ++c) { if (dead) y[c] = (v4u){0x7fc00000u, 0x7fc00000u, 0x7fc00000u, 0x7fc00000u}; *(v4u*)(o + c * 16) = y[c]; }
      ST(2);
    }
    FLUSH();
    return;
  }
  const int X = b & 7;
  const int ci = (b - 64) >> 3;
  const bool local = ci < 8;
  const int half = ci & 1;
  const int g = local ? X : X + 8 * ((ci >> 1) & 1);
  const int c64 = local ? ci >> 1 : 4 + ((ci - 8) >> 2);
  const unsigned call = __builtin_amdgcn_readfirstlane(cprev) + 1;
  if (lane == 0) *ctr = call;
  const size_t sl = DATA_OFF + (size_t)(call % 3) * SLOT;
  const char* slot = me + sl;
  if (EPOCH && b == 65 && tid == 0) __hip_atomic_store((unsigned*)(me + sl + G_OFF + 32 * 128), (unsigned)t0 | 1u, __ATOMIC_RELAXED, __HIP_MEMORY_SCOPE_AGENT);
  bool dead = false;
  int gate = -1;
  const char* abase = slot + wave * 8192;
  const char* bsrc = slot + B_OFF + (size_t)c64 * 262144 + wave * 16384;
  char* part = lds + wave * 4096 + lane * 16;
  const char* red = lds + (wave >> 2) * 32768 + (wave & 3) * 1024 + lane * 16;
  const unsigned tb0 = (a.tab[g] >> (16 * half)) & 0xffffu;
  char* yd = a.peer[3 + (c64 & 3)] + sl + Y_OFF + (size_t)(((tb0 >> (8 * (wave >> 2))) & 63) * 4 + gi4) * 4096 + (wave & 3) * 1024 + lane * 16;
  if (WARM && wave == 7) touch(local ? (const char*)(a.out + (size_t)idx_row(G, a.tab[g + 8] & 63) * NS) : yd);
  unsigned h = 0;
  if (!local && g < NE) {
    v4u A[16], B[16];
    const char* a0 = abase + (size_t)(tb0 & 63) * 65536;
    const char* a1 = abase + (size_t)(tb0 >> 8) * 65536;
    if (!wait16(A, h, loff, a0, a0 + 4096, a1, a1 + 4096, ((lane & 1) ? a1 : a0) + 8188, 0, abase, t0)) dead = true;
    ST(2);
#pragma unroll 1
    for (int t = 0; t < 2; ++t) {
      const char* bs = bsrc + t * 131072;
      if (!wait16(B, h, loff, bs, bs + 4096, bs + 8192, bs + 12288, bs + (lane & 3) * 4096 + 4092, 0, abase, t0)) dead = true;
      ST(1);
      mma_task<1>(B, A, part + t * 2048, dead);
      mma_task<1>(B, A + 8, part + 32768 + t * 2048, dead);
      if (t == 0 && TY0 < 0) gate = gate_read(slot, t0, lane);
    }
  } else {
    v4u B[32];
#pragma unroll
    for (int t = 0; t < 2; ++t) {
      const char* bs = bsrc + t * 131072;
      if (!wait16(B + 16 * t, h, loff, bs, bs + 4096, bs + 8192, bs + 12288, bs + (lane & 3) * 4096 + 4092, 0, abase, t0)) dead = true;
      if (t == 0 && !local && TY0 < 0) gate = gate_read(slot, t0, lane);
    }
    ST(1);
#pragma unroll 1
    for (int pass = 0; pass < (local ? 2 : 1); ++pass) {
      const unsigned tb = (a.tab[g + 8 * pass] >> (16 * half)) & 0xffffu;
      run_pair(B, part + pass * 65536, abase, tb, lane, t0, dead LSTARG);
      if (local) {
        __syncthreads();
        if (pass) { ST(8); }
        const char* p = red + pass * 65536;
        v4f s0 = *(const v4f*)p;
#pragma unroll
        for (int w = 1; w < 8; ++w) s0 += *(const v4f*)(p + w * 4096);
        *(v4f*)(a.out + (size_t)(idx_row(G, (tb >> (8 * (wave >> 2))) & 63) + (lane & 15)) * NS + c64 * 64 + (wave & 3) * 16 + (lane >> 4) * 4) = s0;
        if (!pass) { ST(2); }
      }
    }
  }
  if (!local) {
    if (TY0 < 0 && gate < 0) gate = gate_read(slot, t0, lane);
    __syncthreads();
    ST(8);
    v4f s0 = *(const v4f*)red;
#pragma unroll
    for (int w = 1; w < 8; ++w) s0 += *(const v4f*)(red + w * 4096);
    v4u y0 = __builtin_bit_cast(v4u, s0);
#pragma unroll
    for (int i = 0; i < 4; ++i) if (y0[i] == 0) y0[i] = 0x80000000u;
    const int yslot = YORD == 0 ? 2 * g + half : YORD == 1 ? (g >> 2) * 8 + ((g + c64) & 3) + 4 * half
                    : YORD == 2 ? (g >> 3) * 16 + ((g + 2 * c64) & 7) + 8 * half
                    : (g < 8 ? ((g + 2 * c64) & 7) + 8 * half : (g >> 2) * 8 + ((g + c64) & 3) + 4 * half);
    if (TY0 > 0) pace(t0, TY0 * 100 + yslot * DY);
    if (TY0 < 0) {
      for (int it = 0; gate < 0 && it < 64; ++it) { gate = gate_read(slot, t0, lane); if (gate < 0) __builtin_amdgcn_s_sleep(16); }
      if (gate >= 0) pace(t0, gate * 100 + yslot * DY);
    }
    st16_sys(yd, y0);
  }
  ST(3);
  if (dead && lane == 0) __hip_atomic_store(tmo, call, __ATOMIC_RELAXED, __HIP_MEMORY_SCOPE_AGENT);
  FLUSH();
}
extern "C" void kernel_launch(int rank, int nranks, ncclComm_t comm, hipStream_t stream, void* const* d_in, const int* in_sizes, int n_in,
                              void* d_out, int out_size, const int* in_dtypes, int out_dtype, void* const* d_out_all, void* const* d_ws_all, size_t ws_bytes) {
  Args a;
  a.me = (char*)d_ws_all[rank];
  for (int d = 0; d < 3; ++d) a.peer[d] = (char*)d_ws_all[(rank & 4) | ((rank + 1 + d) & 3)];
  for (int j = 0; j < 4; ++j) a.peer[3 + j] = (char*)d_ws_all[((rank & 4) ^ 4) | j];
  a.x = (const float*)d_in[0]; a.w = (const float*)d_in[1]; a.out = (float*)d_out;
  a.dbg = (unsigned long long*)((char*)d_ws_all[rank] + ws_bytes - DBG_BYTES);
  a.rank = rank;
  {
    const int gi = rank & 3; int order[64], n = 0;
    for (int rb = 0; rb < FI; ++rb) order[n++] = gi * FI + rb;
    for (int d = 1; d < 4; ++d) for (int rb = 0; rb < BT0; ++rb) order[n++] = ((gi + d) & 3) * FI + rb;
    for (int d = 1; d < 4; ++d) for (int rb = BT0; rb < BT0 + BT1; ++rb) order[n++] = ((gi + d) & 3) * FI + rb;
    for (int i = 0; i < 4 * FC; ++i) order[n++] = 4 * FI + i;
    for (int d = 1; d < 4; ++d) for (int rb = BT0 + BT1; rb < FI; ++rb) order[n++] = ((gi + d) & 3) * FI + rb;
    for (int g = 0; g < 16; ++g) a.tab[g] = order[4 * g] | (order[4 * g + 1] << 8) | (order[4 * g + 2] << 16) | (order[4 * g + 3] << 24);
  }
  (void)hipFuncSetAttribute((const void*)agk, hipFuncAttributeMaxDynamicSharedMemorySize, LDS_BYTES);
  agk<<<256, NT, LDS_BYTES, stream>>>(a);
}
```

```cpp
#include <hip/hip_runtime.h>
#include <rccl/rccl.h>
typedef unsigned long long u64;
typedef unsigned v4u __attribute__((ext_vector_type(4)));
typedef unsigned v2u __attribute__((ext_vector_type(2)));
typedef unsigned s8u __attribute__((ext_vector_type(8)));
typedef float v4f __attribute__((ext_vector_type(4)));
typedef float v2f __attribute__((ext_vector_type(2)));
typedef _Float16 v8h __attribute__((ext_vector_type(8)));
typedef _Float16 v2h __attribute__((ext_vector_type(2)));

constexpr int PS = 60;
constexpr int PD = 5550;
constexpr int DY = 1375;
constexpr int RPUB = 9;
constexpr int TG1 = 50, TG2 = 150, TG3 = 250, TX4 = 350, TI2 = 150, TI3 = 300;
constexpr int TCL = 150, DCL = 3000;
constexpr int ZDELAY = 300, ZSPLIT = 3584, YV = 50;
constexpr int FI = 11, FC = 5, XA = 2, NEP = 10;
constexpr int KD = 2048, NS = 256, MS = 256;
constexpr size_t SLOT = 8u << 20;
constexpr size_t DATA_OFF = 1u << 20;
constexpr size_t B_OFF = 4u << 20;
constexpr size_t G_OFF = 6u << 20;
constexpr int ZERO_KB = 6144 + 5;
constexpr u64 DEADLINE = 20000000ull;
constexpr int NT = 512;
constexpr int LDS_BYTES = 131072 + 256;
struct Args {
  char* me; const float* x; const float* w; float* out;
  int rank; int pad[7];
  char* ptr[12][4];
  unsigned tab[16];
};
__device__ __forceinline__ char* gp(u64 v) { return (char*)(__attribute__((address_space(1))) char*)v; }
__device__ __forceinline__ u64 rt() { return __builtin_amdgcn_s_memrealtime(); }
__device__ __forceinline__ void st16_sys(void* p, v4u v) { asm volatile("global_store_dwordx4 %0, %1, off sc0 sc1\n\ts_nop 1" :: "v"(p), "v"(v) : "memory"); }
__device__ __forceinline__ void st16_so(u64 base, unsigned off, v4u v) { asm volatile("global_store_dwordx4 %0, %1, %2 sc0 sc1\n\ts_nop 1" :: "v"(off), "v"(v), "s"(base) : "memory"); }
__device__ __forceinline__ void st4_sys(void* p, unsigned v) { __hip_atomic_store((unsigned*)p, v, __ATOMIC_RELAXED, __HIP_MEMORY_SCOPE_SYSTEM); }
__device__ __forceinline__ unsigned ld_sys(const void* p) { return __hip_atomic_load((const unsigned*)p, __ATOMIC_RELAXED, __HIP_MEMORY_SCOPE_SYSTEM); }
__device__ __forceinline__ unsigned ld_dev(const void* p) { return __hip_atomic_load((const unsigned*)p, __ATOMIC_RELAXED, __HIP_MEMORY_SCOPE_AGENT); }
#define LDMOD " sc0 sc1"
#define L4(a, b, c, d, off, base) "global_load_dwordx4 %" #a ", %" #off ", %" #base LDMOD "\n\tglobal_load_dwordx4 %" #b ", %" #off ", %" #base " offset:1024" LDMOD \
  "\n\tglobal_load_dwordx4 %" #c ", %" #off ", %" #base " offset:2048" LDMOD "\n\tglobal_load_dwordx4 %" #d ", %" #off ", %" #base " offset:3072" LDMOD "\n\t"
__device__ __forceinline__ void ld16m(v4u* f, unsigned loff, const char* b0, const char* b1) {
  asm volatile("s_nop 4\n\t" L4(0, 1, 2, 3, 16, 17) L4(4, 5, 6, 7, 16, 18) L4(8, 9, 10, 11, 16, 19) L4(12, 13, 14, 15, 16, 20) "s_waitcnt vmcnt(0)"
               : "=&v"(f[0]), "=&v"(f[1]), "=&v"(f[2]), "=&v"(f[3]), "=&v"(f[4]), "=&v"(f[5]), "=&v"(f[6]), "=&v"(f[7]),
                 "=&v"(f[8]), "=&v"(f[9]), "=&v"(f[10]), "=&v"(f[11]), "=&v"(f[12]), "=&v"(f[13]), "=&v"(f[14]), "=&v"(f[15])
               : "v"(loff), "s"(b0), "s"(b0 + 4096), "s"(b1), "s"(b1 + 4096) : "memory");
}
__device__ __forceinline__ void ld8m(v4u* f, unsigned loff, const char* b0) {
  asm volatile("s_nop 4\n\t" L4(0, 1, 2, 3, 8, 9) L4(4, 5, 6, 7, 8, 10) "s_waitcnt vmcnt(0)"
               : "=&v"(f[0]), "=&v"(f[1]), "=&v"(f[2]), "=&v"(f[3]), "=&v"(f[4]), "=&v"(f[5]), "=&v"(f[6]), "=&v"(f[7])
               : "v"(loff), "s"(b0), "s"(b0 + 4096) : "memory");
}
__device__ __forceinline__ unsigned min4(unsigned m, v4u f) { m = min(min(m, f.x), f.y); return min(min(m, f.z), f.w); }
template <int N> __device__ __forceinline__ bool all_ok(const v4u* f) {
  unsigned m = 0xffffffffu;
#pragma unroll
  for (int i = 0; i < N; ++i) m = min4(m, f[i]);
  return __all(m != 0);
}
__device__ __forceinline__ unsigned pkz(float a, float b) {
  v2h h; h.x = (_Float16)a; h.y = (_Float16)b; unsigned u = __builtin_bit_cast(unsigned, h); return u == 0 ? 0x8000u : u; }
template <int N> __device__ __forceinline__ bool wait_frags(v4u* f, unsigned loff, const char* b0, const char* b1, const char* hp, u64 t0) {
  for (;;) {
    int it = 0;
    while (!__all(ld_sys(hp) != 0) && ++it < 64) __builtin_amdgcn_s_sleep(1);
    if (N == 16) ld16m(f, loff, b0, b1); else ld8m(f, loff, b0);
    if (all_ok<N>(f)) return true;
    if (rt() - t0 > DEADLINE) return false;
  }
}
__device__ __forceinline__ int idx_row(int G, int idx) {
  if (idx < 4 * FI) return (G * 4 + idx / FI) * MS + (idx % FI) * 16;
  return ((G ^ 1) * 4 + (idx - 4 * FI) / FC) * MS + (FI + (idx - 4 * FI) % FC) * 16;
}
__device__ __forceinline__ int ks_of(int kp, int rb) {
  const int w6 = (kp * 43) >> 8, m = (kp - 48) >> 31;
  const int w = (w6 & m) | (kp & ~m), k = ((kp - 6 * w6) & m) | ((kp >> 3) & ~m);
  return ((w + 2 * rb) & 7) * 8 + (k & 7);
}
__device__ __forceinline__ int send_order(int u, int lane, bool intra) {
  const int f = u + 32 * lane;
  const int g3 = max(f - 512, 0) / 3, g5 = f / 5;
  const int rbi = f < 512 ? (f & 3) + ((f >> 8) << 2) : 8 + (f - 512) - 3 * g3, kpi = f < 512 ? f >> 2 : g3;
  const int rb = intra ? rbi : f - 5 * g5, kp = intra ? kpi : g5;
  return ((intra ? rb : rb + FI) * 64 + ks_of(kp & 63, rb)) & 1023;
}
struct XL { v4f p0, p1; };
__device__ __forceinline__ XL x_load(const float* x, int e, int lane) {
  const float* src = x + (size_t)((e >> 6) * 16 + (lane >> 3)) * KD + (e & 63) * 32 + (lane & 7) * 4;
  XL r; r.p0 = *(const v4f*)src; r.p1 = *(const v4f*)(src + 8 * KD); return r;
}
__device__ __forceinline__ void x_stage(char* frag, const XL& r, int lane) {
  char* ldst = frag + ((lane >> 3) + 16 * ((lane & 7) >> 1)) * 16 + (lane & 1) * 8;
  v2u v0, v1; v0.x = pkz(r.p0.x, r.p0.y); v0.y = pkz(r.p0.z, r.p0.w); v1.x = pkz(r.p1.x, r.p1.y); v1.y = pkz(r.p1.z, r.p1.w);
  *(v2u*)ldst = v0; *(v2u*)(ldst + 128) = v1;
}
__device__ __forceinline__ void pace(u64 t0, unsigned when100) {
  const u64 until = t0 + when100 / 100;
  while (rt() < until) __builtin_amdgcn_s_sleep(1);
}
__device__ __forceinline__ int gate_read(const char* slot, u64 t0, int lane) {
  const unsigned e = ld_dev(slot + G_OFF + (lane & 31) * 128);
  if (!__all(e != 0)) return -1;
  int rel = min(max((int)(e - (unsigned)t0), 0), 100000);
#pragma unroll
  for (int m = 1; m < 32; m <<= 1) rel += __shfl_xor(rel, m);
  return __builtin_amdgcn_readfirstlane(rel) >> 5;
}
__device__ __forceinline__ unsigned lds_ld(const unsigned* p) { return __hip_atomic_load(p, __ATOMIC_RELAXED, __HIP_MEMORY_SCOPE_WORKGROUP); }
__device__ __forceinline__ void lds_wait(const unsigned* p, unsigned n, u64 t0) {
  while (lds_ld(p) < n && rt() - t0 < DEADLINE) __builtin_amdgcn_s_sleep(1);
  asm volatile("" ::: "memory");
}
#define MFMA(b, a, c) __builtin_amdgcn_mfma_f32_16x16x32_f16(__builtin_bit_cast(v8h, b), __builtin_bit_cast(v8h, a), c, 0, 0, 0)
#define BIDX(c, k) (((c) >> 1) * 16 + ((k) & 1) * 8 + ((k) >> 1) * 2 + ((c) & 1))
template <int K0, int K1> __device__ __forceinline__ void mma(const v4u* B, const v4u* f, v4f* acc) {
#pragma unroll
  for (int k = K0; k < K1; ++k)
#pragma unroll
    for (int c = 0; c < 4; ++c) acc[c] = MFMA(B[BIDX(c, k)], f[k], (k == 0 ? (v4f){0, 0, 0, 0} : acc[c]));
}
__device__ __forceinline__ void put_part(char* p, v4f v, bool dead) {
  if (dead) v = (v4f){__builtin_nanf(""), __builtin_nanf(""), __builtin_nanf(""), __builtin_nanf("")};
  *(v4f*)p = v;
}
__device__ __forceinline__ bool task_head(const v4u* B, const v4u* f, char* sl, bool dead, bool& full) {
  if (!dead && !all_ok<6>(f)) return false;
  v4f acc[4];
  mma<0, 6>(B, f, acc);
  full = dead || all_ok<2>(f + 6);
  if (full) mma<6, 8>(B, f, acc);
#pragma unroll
  for (int c = 0; c < 4; ++c) put_part(sl + c * 1024, acc[c], dead);
  return true;
}
__device__ __forceinline__ bool task_tail(const v4u* B, const v4u* f, v4f* acc, char* p, bool dead) {
  if (!dead && !all_ok<2>(f + 6)) return false;
  mma<6, 8>(B, f, acc);
#pragma unroll
  for (int i = 0; i < 4; ++i) put_part(p + i * 1024, acc[i], dead);
  return true;
}
__device__ __forceinline__ void run_pair(const v4u* B, char* sl, const char* abase, unsigned tb, int lane, u64 t0, bool& dead) {
  const unsigned loff = lane * 16;
  const unsigned hb = ((tb >> (8 * (lane & 1))) & 63u) * 65536u;
  const char* a0 = abase + (size_t)(tb & 63) * 65536;
  const char* a1 = abase + (size_t)((tb >> 8) & 63) * 65536;
  unsigned head = 3, tail = 0, it = 0;
  while (head | tail) {
    if ((++it & 31) == 0 && rt() - t0 > DEADLINE) dead = true;
    unsigned c = tail;
    if (head) {
      const unsigned h = ld_sys(abase + hb + ((tail >> (lane & 1)) & 1 ? 8188u : 6140u));
      c = (dead ? 3u : (unsigned)__ballot(h != 0)) & (head | tail);
      if (!c) { __builtin_amdgcn_s_sleep(1); continue; }
    }
    if ((c & tail) == 3) {
      v4u f[8], e[8]; v4f acc0[4], acc1[4];
#pragma unroll
      for (int i = 0; i < 4; ++i) { acc0[i] = *(const v4f*)(sl + i * 1024); acc1[i] = *(const v4f*)(sl + 32768 + i * 1024); }
      asm volatile("s_nop 4\n\tglobal_load_dwordx4 %0, %4, %5 offset:3072" LDMOD "\n\tglobal_load_dwordx4 %1, %4, %5 offset:2048" LDMOD
                   "\n\tglobal_load_dwordx4 %2, %4, %6 offset:3072" LDMOD "\n\tglobal_load_dwordx4 %3, %4, %6 offset:2048" LDMOD "\n\ts_waitcnt vmcnt(0)"
                   : "=&v"(f[7]), "=&v"(f[6]), "=&v"(e[7]), "=&v"(e[6]) : "v"(loff), "s"(a0 + 4096), "s"(a1 + 4096) : "memory");
      if (task_tail(B, f, acc0, sl, dead)) tail &= ~1u;
      if (task_tail(B, e, acc1, sl + 32768, dead)) tail &= ~2u;
      continue;
    }
    if (c & tail) {
      const unsigned bit = c & tail;
      char* p = sl + (bit >> 1) * 32768;
      v4u f[8]; v4f acc[4];
#pragma unroll
      for (int i = 0; i < 4; ++i) acc[i] = *(const v4f*)(p + i * 1024);
      asm volatile("s_nop 4\n\tglobal_load_dwordx4 %0, %2, %3 offset:3072" LDMOD "\n\tglobal_load_dwordx4 %1, %2, %3 offset:2048" LDMOD "\n\ts_waitcnt vmcnt(0)"
                   : "=&v"(f[7]), "=&v"(f[6]) : "v"(loff), "s"((bit == 1 ? a0 : a1) + 4096) : "memory");
      if (task_tail(B, f, acc, p, dead)) tail &= ~bit;
    }
    c &= head;
    if (!c) continue;
    v4u f[16]; bool full;
    if (c == 3) {
      ld16m(f, loff, a0, a1);
      if (task_head(B, f + 8, sl + 32768, dead, full)) { head &= ~2u; if (!full) tail |= 2u; }
    } else ld8m(f, loff, c == 1 ? a0 : a1);
    const unsigned bit = c == 2 ? 2u : 1u;
    if (task_head(B, f, sl + (bit >> 1) * 32768, dead, full)) { head &= ~bit; if (!full) tail |= bit; }
  }
}
__device__ __forceinline__ void finish(const char* red, float* dst, const char* slot, u64 t0, int lane, int gate, int yslot) {
  if (gate == -1) gate = gate_read(slot, t0, lane);
  __syncthreads();
  v4f s0 = *(const v4f*)red;
#pragma unroll
  for (int w = 1; w < 8; ++w) s0 += *(const v4f*)(red + w * 4096);
  v4u y0 = __builtin_bit_cast(v4u, s0);
#pragma unroll
  for (int i = 0; i < 4; ++i) if (y0[i] == 0) y0[i] = 0x80000000u;
  for (int it = 0; gate == -1 && it < 64; ++it) { gate = gate_read(slot, t0, lane); if (gate < 0) __builtin_amdgcn_s_sleep(16); }
  if (gate >= 0) pace(t0, gate * 100 + yslot * DY);
  st16_sys(dst, y0);
}
__device__ __forceinline__ void verify(float* outp, const char* slot, char* me, int G, int tile, int lane, u64 t0, unsigned call) {
  const char* ysrc = (const char*)(outp + (size_t)idx_row(G ^ 1, tile >> 2) * NS + (tile & 3) * 64);
  const unsigned yo = (lane >> 3) * 1024 + (lane & 7) * 16;
  bool dead = false, ok = false;
  int gate;
  while ((gate = gate_read(slot, t0, lane)) < 0) { __builtin_amdgcn_s_sleep(16); if (rt() - t0 > DEADLINE) { dead = true; break; } }
  if (!dead) pace(t0, (gate + YV) * 100);
  for (unsigned it = 1; !dead; ++it) {
    if (!ok) {
      v4u y[4];
      asm volatile("s_nop 4\n\tglobal_load_dwordx4 %0, %4, %5" LDMOD "\n\tglobal_load_dwordx4 %1, %4, %5 offset:128" LDMOD
                   "\n\tglobal_load_dwordx4 %2, %4, %6" LDMOD "\n\tglobal_load_dwordx4 %3, %4, %6 offset:128" LDMOD "\n\ts_waitcnt vmcnt(0)"
                   : "=&v"(y[0]), "=&v"(y[1]), "=&v"(y[2]), "=&v"(y[3]) : "v"(yo), "s"(ysrc), "s"(ysrc + 8192) : "memory");
      ok = min4(min4(min4(min4(0xffffffffu, y[0]), y[1]), y[2]), y[3]) != 0;
    }
    if (__all(ok)) break;
    __builtin_amdgcn_s_sleep(1);
    if ((it & 31) == 0 && rt() - t0 > DEADLINE) dead = true;
  }
  if (dead) {
    const v4u nan4 = {0x7fc00000u, 0x7fc00000u, 0x7fc00000u, 0x7fc00000u};
#pragma unroll
    for (int c = 0; c < 4; ++c) st16_sys((char*)ysrc + yo + (c & 1) * 128 + (c >> 1) * 8192, nan4);
    if (lane == 0) st4_sys(me + 4096, call);
  }
}

extern "C" __global__ void __launch_bounds__(NT) agk(Args a) {
  extern __shared__ __attribute__((aligned(16))) char lds[];
  const s8u ka = *(const volatile __attribute__((address_space(4))) s8u*)__builtin_amdgcn_kernarg_segment_ptr();
  const u64 t0 = rt();
  __builtin_amdgcn_sched_barrier(0);
  const int tid = threadIdx.x, lane = tid & 63, wave = __builtin_amdgcn_readfirstlane(tid >> 6), b = blockIdx.x;
  const bool intra = b < 32;
  const int u = b & 31;
  unsigned* lflag = (unsigned*)(lds + 131072);
  lflag[tid & 31] = 0;
  const int f0 = u + 32 * wave, kp0 = (f0 * (intra ? 16384 : 13108)) >> 16, rb0 = f0 - (intra ? 4 : 5) * kp0;
  const int e0 = (rb0 + (intra ? 0 : FI)) * 64 + ks_of(kp0, rb0);
  const int xl0 = (lane & 15) * KD + (lane >> 4) * 8, xl1 = (lane >> 3) * KD + (lane & 7) * 4;
  unsigned xa = (unsigned)((e0 >> 6) * 16 * KD + (e0 & 63) * 32 + (tid < 64 ? xl0 : xl1)) * 4u, xb = xa + (tid < 64 ? 16u : 32u * KD);
  asm volatile("" : "+v"(xa), "+v"(xb));
  __builtin_amdgcn_sched_barrier(0);
  char* const me = gp(ka[0] | ((u64)ka[1] << 32));
  const float* const xp = (const float*)gp(ka[2] | ((u64)ka[3] << 32));
  const float* const wp = (const float*)gp(ka[4] | ((u64)ka[5] << 32));
  float* const outp = (float*)gp(ka[6] | ((u64)ka[7] << 32));
  XL r0;
  if (b < 64 && (intra || wave < XA)) { r0.p0 = *(const v4f*)((const char*)xp + xa); r0.p1 = *(const v4f*)((const char*)xp + xb); }
  unsigned* ctr = (unsigned*)(me + 65536) + b * 16 + wave;
  const int mb = (b - 64) >> 31;
  const unsigned rowo = 64 + 32 * (((((b >> 5) << 2) + (b & 3)) & mb) | ((8 + (b & 1) + ((b >> 2) & 2)) & ~mb));
  unsigned cprev; s8u pr8; int rank;
  asm volatile("s_load_dword %0, %3, 0x0 glc\n\ts_load_dwordx8 %1, %4, %5\n\ts_load_dword %2, %4, 0x20\n\ts_waitcnt lgkmcnt(0)\n\ts_barrier"
               : "=&s"(cprev), "=&s"(pr8), "=&s"(rank) : "s"(ctr), "s"(__builtin_amdgcn_kernarg_segment_ptr()), "s"(rowo) : "memory");
  const u64 q0 = pr8[0] | ((u64)pr8[1] << 32), q1 = pr8[2] | ((u64)pr8[3] << 32), q2 = pr8[4] | ((u64)pr8[5] << 32), q3 = pr8[6] | ((u64)pr8[7] << 32);
  const unsigned call = cprev + 1;
  const unsigned so = (call % 3) * (unsigned)SLOT;
  char* const slot = me + DATA_OFF + so;
  const unsigned loff = lane * 16;
  const int G = rank >> 2, gi4 = rank & 3;
  if (__builtin_expect(b < 64 && wave == 0, 1)) {
    {
      unsigned o = so + e0 * 1024 + loff;
      asm volatile("" : "+v"(o));
      asm volatile("" : "+v"(r0.p0), "+v"(r0.p1));
      v4u v; v.x = pkz(r0.p0.x, r0.p0.y); v.y = pkz(r0.p0.z, r0.p0.w); v.z = pkz(r0.p1.x, r0.p1.y); v.w = pkz(r0.p1.z, r0.p1.w);
      st16_so(q0, o, v); st16_so(q1, o, v); st16_so(q2, o, v); st16_so(q3, o, v);
    }
    const int etab = send_order(u, lane, intra);
    unsigned late = 0, when = PS * 100 + PD + u * (PD / 32);
    if (!intra) {
#pragma unroll 1
      for (int r = 1; r < 2 * FC + 8; ++r) {
        v4u vv[4]; unsigned o;
        if (r >= XA && r < XA + 8) {
          const int n = r - XA, t = n >> 2, kk = (n >> 1) & 1, cb = n & 1;
          if (cb == 0) lds_wait(lflag + 24 + (n >> 1), 2, t0);
#pragma unroll
          for (int d = 0; d < 4; ++d) vv[d] = *(const v4u*)(lds + 32768 + (kk * 16 + t * 2 + cb) * 1024 + ((d + b) & 3) * 4096 + loff);
          o = (unsigned)B_OFF + (8 + gi4 * 2 + t) * 131072 + (kk * 64 + u * 2 + cb) * 1024 - (3 * FI + gi4 * FC) * 65536;
        } else {
          const int j = r < XA ? r : r - 8;
          lds_wait(lflag + j, 1, t0);
          vv[0] = vv[1] = vv[2] = vv[3] = *(const v4u*)(lds + j * 1024 + loff);
          o = __builtin_amdgcn_readlane(etab, j) * 1024;
        }
        o += so + loff;
        if (r == RPUB) lds_wait(lflag + 29, 3, t0);
        if (!late) late = __builtin_amdgcn_readfirstlane(lds_ld(lflag + 28));
        pace(t0 - late, when); when += PD;
        asm volatile("s_waitcnt vmcnt(8)" ::: "memory");
        if (r == RPUB) {
          const unsigned sched = (unsigned)(t0 - late) + PS + (unsigned)((2 * FC + 8) * PD) / 100;
          const unsigned pred = (unsigned)rt() + (unsigned)((2 * FC + 8 - RPUB) * PD - u * (PD / 32)) / 100;
          if (lane == 0) st4_sys(slot + G_OFF + u * 128, ((int)(pred - sched) > 0 ? pred : sched) | 1u);
        }
        st16_so(q0, o, vv[0]); st16_so(q1, o, vv[1]); st16_so(q2, o, vv[2]); st16_so(q3, o, vv[3]);
      }
    } else {
#pragma unroll 1
      for (int j = 1; j < 2 * FI; ++j) {
        lds_wait(lflag + j, 1, t0);
        const v4u v = *(const v4u*)(lds + j * 1024 + loff);
        const unsigned o = __builtin_amdgcn_readlane(etab, j) * 1024 + so + loff;
        if (!late) late = __builtin_amdgcn_readfirstlane(lds_ld(lflag + 28));
        pace(t0 - late, when); when += PD;
        asm volatile("s_waitcnt vmcnt(6)" ::: "memory");
        st16_so(q0, o, v); st16_so(q1, o, v); st16_so(q2, o, v);
      }
    }
    if (lane == 0) st4_sys(ctr, call);
    if (!intra) verify(outp, slot, me, G, u, lane, t0, call);
    return;
  }
  if (b < 64) {
    auto wload = [&](v4f* q, int g, int hf) __attribute__((always_inline)) {
      const float* wsrc = wp + (size_t)(u * 64 + (g & 1) * 32 + (hf * 2 + (lane >> 5)) * 8) * NS + ((lane & 31) >> 3) * 64 + (g >> 1) * 32 + (lane & 7) * 4;
#pragma unroll
      for (int i = 0; i < 8; ++i) q[i] = *(const v4f*)(wsrc + i * NS);
    };
    auto wstore = [&](const v4f* q, int g, int hf) __attribute__((always_inline)) {
      const int t = g >> 1, kk = g & 1, kg = hf * 2 + (lane >> 5), j = (lane & 31) >> 3, c4 = lane & 7;
      char* dstl = lds + 32768 + (kk * 16 + j * 4 + t * 2 + (c4 >> 2)) * 1024 + (kg * 16 + (c4 & 3) * 4) * 16;
#pragma unroll
      for (int c = 0; c < 4; ++c) {
        v4u o; o.x = pkz(q[0][c], q[1][c]); o.y = pkz(q[2][c], q[3][c]); o.z = pkz(q[4][c], q[5][c]); o.w = pkz(q[6][c], q[7][c]);
        *(v4u*)(dstl + c * 16) = o;
      }
      asm volatile("s_waitcnt lgkmcnt(0)" ::: "memory");
      if (lane == 0) __hip_atomic_fetch_add(lflag + 24 + g, 1u, __ATOMIC_RELAXED, __HIP_MEMORY_SCOPE_WORKGROUP);
    };
    auto put = [&](int j, const XL& x) __attribute__((always_inline)) {
      x_stage(lds + j * 1024, x, lane);
      asm volatile("s_waitcnt lgkmcnt(0)" ::: "memory");
      if (lane == 0) __hip_atomic_store(lflag + j, 1u, __ATOMIC_RELAXED, __HIP_MEMORY_SCOPE_WORKGROUP);
    };
    v4f wq0[8];
    if (!intra && (wave >> 1) == 1) wload(wq0, 0, wave & 1);
    const int etab = send_order(u, lane, intra);
    if (intra || wave < XA) { asm volatile("" : "+v"(r0.p0), "+v"(r0.p1)); put(wave, r0); }
#pragma unroll 1
    for (int tk = wave - 2; tk < 8 && !intra && wave >= 2; tk += 6) {
      if (tk >= 2) { pace(t0, (tk < 4 ? TG1 : tk < 6 ? TG2 : TG3) * 100); wload(wq0, tk >> 1, tk & 1); }
      wstore(wq0, tk >> 1, tk & 1);
    }
    if (lane == 0) st4_sys(ctr, call);
    const v4u zero = {0, 0, 0, 0};
    if (wave == 1) {
      unsigned e = 0;
      for (int it = 0; it < 8 && e == 0; ++it) e = ld_dev(slot + G_OFF + 32 * 128);
      const int d = (int)((unsigned)t0 - e);
      if (e != 0 && d > 0 && d < 200 && lane == 0) __hip_atomic_store(lflag + 28, (unsigned)d, __ATOMIC_RELAXED, __HIP_MEMORY_SCOPE_WORKGROUP);
    }
    if (intra) {
      char* ldst = slot + (size_t)(gi4 * FI * 64) * 1024 + loff;
#pragma unroll 1
      for (int i = 1; i < 4; ++i) {
        const int j = wave + 7 * i;
        XL r;
        if (i < 3) { pace(t0, (i == 1 ? TI2 : TI3) * 100); r = x_load(xp, __builtin_amdgcn_readlane(etab, j), lane); }
        st16_sys(ldst + (size_t)__builtin_amdgcn_readlane(etab, j - 7) * 1024, *(const v4u*)(lds + (j - 7) * 1024 + loff));
        if (i < 3) put(j, r);
      }
    } else if (wave == 1 || wave == 4 || wave == 5) {
      const int n0 = max(wave - 3, 0);
      unsigned when = TCL * 100 + n0 * (DCL / 3);
#pragma unroll 1
      for (int n = n0; n < 32; n += 3) {
        pace(t0, when); when += DCL;
        st16_sys(outp + (size_t)(idx_row(G ^ 1, 2 * u + (n >> 4)) + (n & 15)) * NS + lane * 4, zero);
      }
      asm volatile("s_waitcnt vmcnt(0)" ::: "memory");
      if (lane == 0) __hip_atomic_fetch_add(lflag + 29, 1u, __ATOMIC_RELAXED, __HIP_MEMORY_SCOPE_WORKGROUP);
    } else {
      const int j = wave >= 6 ? 2 * wave - 10 : 2 * wave + 2;
      pace(t0, TX4 * 100);
      XL r[2];
#pragma unroll
      for (int i = 0; i < 2; ++i) r[i] = x_load(xp, __builtin_amdgcn_readlane(etab, j + i), lane);
#pragma unroll
      for (int i = 0; i < 2; ++i) put(j + i, r[i]);
    }
    char* z = me + DATA_OFF + (size_t)((call + 2) % 3) * SLOT + loff;
    pace(t0, ZDELAY * 100);
    for (int k = u + 32 * (wave - 1) + (intra ? 0 : ZSPLIT); k < (intra ? ZSPLIT : ZERO_KB); k += 224) {
      st16_sys(z + (size_t)k * 1024, zero); asm volatile("s_waitcnt vmcnt(1)" ::: "memory"); __builtin_amdgcn_s_sleep(2); }
    if (!intra) verify(outp, slot, me, G, wave * 32 + u, lane, t0, call);
    return;
  }
  const int X = b & 7, ci = (b - 64) >> 3;
  const bool local = ci < 8;
  v2f wq[8];
  if (local) {
    const float* wsrc = wp + (size_t)((b - 64) * 32 + (wave & 3) * 8) * NS + (wave >> 2) * 128 + lane * 2;
#pragma unroll
    for (int j = 0; j < 8; ++j) wq[j] = *(const v2f*)(wsrc + j * NS);
  }
  const int pr = ((local ? ci : ci - 8) >> 1) * 4 + (X >> 1);
  const int c64 = (local ? 0 : 4) + (X & 1) + 2 * (ci & 1);
  const int half = pr & 1, g = pr >> 1, g2 = 14 - (g & 6) + (g & 1);
  const int yslot = (pr & ~3) + ((pr + (c64 & 2)) & 3);
  if (b == 129 && tid == 0) st4_sys(slot + G_OFF + 32 * 128, (unsigned)t0 | 1u);
  if (lane == 0) st4_sys(ctr, call);
  bool dead = false;
  const char* abase = slot + wave * 8192;
  const char* bsrc = slot + B_OFF + (size_t)c64 * 262144 + wave * 8192;
  char* part = lds + wave * 4096 + loff;
  const char* red = lds + (wave >> 2) * 32768 + (wave & 3) * 1024 + loff;
  const unsigned tb0 = (a.tab[g] >> (16 * half)) & 0xffffu;
  const size_t ooff = (size_t)(lane & 15) * NS + (wave & 3) * 16 + (lane >> 4) * 4;
  float* yd = (float*)gp(q0) + (size_t)idx_row(G, (tb0 >> (8 * (wave >> 2))) & 63) * NS + gi4 * 64 + ooff;
  if (local) {
    const int col = (wave >> 2) * 128 + lane * 2;
    char* dstl = lds + 65536 + (col >> 4) * 1024 + ((wave & 3) * 16 + (col & 15)) * 16;
#pragma unroll
    for (int c = 0; c < 2; ++c) {
      v4u o; o.x = pkz(wq[0][c], wq[1][c]); o.y = pkz(wq[2][c], wq[3][c]); o.z = pkz(wq[4][c], wq[5][c]); o.w = pkz(wq[6][c], wq[7][c]);
      *(v4u*)(dstl + c * 16) = o;
    }
    __syncthreads();
    const v4u f0v = *(const v4u*)(lds + 65536 + wave * 2048 + loff), f1v = *(const v4u*)(lds + 65536 + wave * 2048 + 1024 + loff);
    char* bl = slot + B_OFF + (size_t)wave * 131072 + (b & 1) * 65536 + ((b - 64) >> 1) * 2048 + loff;
    st16_sys(bl, f0v); st16_sys(bl + 1024, f1v);
  }
  if (tid == 448) (void)ld_sys(local ? outp : yd);
  if (!local && pr < NEP) {
    v4u A[16], B[8];
    const char* a0 = abase + (size_t)(tb0 & 63) * 65536;
    const char* a1 = abase + (size_t)((tb0 >> 8) & 63) * 65536;
    if (!wait_frags<16>(A, loff, a0, a1, ((lane & 1) ? a1 : a0) + 8188, t0)) dead = true;
#pragma unroll 1
    for (int t = 0; t < 2; ++t) {
      v4f acc[4];
#pragma unroll
      for (int c = 0; c < 4; ++c) acc[c] = (v4f){0, 0, 0, 0};
#pragma unroll
      for (int kk = 0; kk < 2; ++kk) {
        const char* bs = bsrc + (t * 2 + kk) * 65536;
        if (!wait_frags<8>(B, loff, bs, bs, bs + (lane & 3) * 2048 + 2044, t0)) dead = true;
#pragma unroll
        for (int i = 0; i < 4; ++i)
#pragma unroll
          for (int c = 0; c < 4; ++c) acc[c] = MFMA(B[i * 2 + (c & 1)], A[(c >> 1) * 8 + i * 2 + kk], acc[c]);
      }
#pragma unroll
      for (int c = 0; c < 4; ++c) put_part(part + (c >> 1) * 32768 + t * 2048 + (c & 1) * 1024, acc[c], dead);
    }
    finish(red, yd, slot, t0, lane, -1, yslot);
  } else {
    v4u B[32];
#pragma unroll
    for (int q = 0; q < 4; ++q) {
      const char* bs = bsrc + q * 65536;
      if (!wait_frags<8>(B + 8 * q, loff, bs, bs, bs + (lane & 3) * 2048 + 2044, t0)) dead = true;
    }
#pragma unroll 1
    for (int pass = local ? 0 : 1; pass < 2; ++pass) {
      const unsigned tb = local && pass ? (a.tab[g2] >> (16 * half)) & 0xffffu : tb0;
      const int gate = local ? -2 : gate_read(slot, t0, lane);
      run_pair(B, part + pass * 65536, abase, tb, lane, t0, dead);
      float* dst = local ? outp + (size_t)idx_row(G, (tb >> (8 * (wave >> 2))) & 63) * NS + c64 * 64 + ooff : yd;
      finish(red + pass * 65536, dst, slot, t0, lane, gate, yslot);
    }
  }
  if (dead && lane == 0) st4_sys(me + 4096, call);
}

extern "C" void kernel_launch(int rank, int nranks, ncclComm_t comm, hipStream_t stream, void* const* d_in, const int* in_sizes, int n_in,
                              void* d_out, int out_size, const int* in_dtypes, int out_dtype, void* const* d_out_all, void* const* d_ws_all, size_t ws_bytes) {
  if (nranks != 8 || n_in < 2 || ws_bytes < DATA_OFF + 3 * SLOT) return;
  Args a;
  a.me = (char*)d_ws_all[rank];
  a.x = (const float*)d_in[0]; a.w = (const float*)d_in[1]; a.out = (float*)d_out;
  a.rank = rank;
  for (int i = 0; i < 7; ++i) a.pad[i] = 0;
  const int gi = rank & 3, own = rank & 4, oth = own ^ 4;
  for (int r = 0; r < 4; ++r)
    for (int d = 0; d < 4; ++d) {
      a.ptr[r][d] = (char*)d_ws_all[d == 3 ? rank : own | ((rank + 1 + (d + r) % 3) & 3)] + DATA_OFF + (size_t)gi * FI * 65536;
      a.ptr[4 + r][d] = (char*)d_ws_all[oth | ((d + r) & 3)] + DATA_OFF + (size_t)(3 * FI + gi * FC) * 65536;
      a.ptr[8 + r][d] = (char*)d_out_all[oth | r];
    }
  int order[64], n = 0;
  for (int rb = 0; rb < FI; ++rb) order[n++] = gi * FI + rb;
  for (int d = 1; d < 4; ++d) for (int rb = 0; rb < 4; ++rb) order[n++] = ((gi + d) & 3) * FI + rb;
  for (int d = 1; d < 4; ++d) for (int rb = 4; rb < 8; ++rb) order[n++] = ((gi + d) & 3) * FI + rb;
  for (int i = 0; i < 4 * FC; ++i) order[n++] = 4 * FI + i;
  for (int d = 1; d < 4; ++d) for (int rb = 8; rb < FI; ++rb) order[n++] = ((gi + d) & 3) * FI + rb;
  for (int g = 0; g < 16; ++g) a.tab[g] = order[4 * g] | (order[4 * g + 1] << 8) | (order[4 * g + 2] << 16) | (order[4 * g + 3] << 24);
  (void)hipFuncSetAttribute((const void*)agk, hipFuncAttributeMaxDynamicSharedMemorySize, LDS_BYTES);
  agk<<<256, NT, LDS_BYTES, stream>>>(a);
}
```

```cpp
#include <hip/hip_runtime.h>
#include <rccl/rccl.h>
typedef unsigned long long u64;
typedef unsigned v4u __attribute__((ext_vector_type(4)));
typedef unsigned v2u __attribute__((ext_vector_type(2)));
typedef unsigned s8u __attribute__((ext_vector_type(8)));
typedef float v4f __attribute__((ext_vector_type(4)));
typedef float v2f __attribute__((ext_vector_type(2)));
typedef _Float16 v8h __attribute__((ext_vector_type(8)));
typedef _Float16 v2h __attribute__((ext_vector_type(2)));

constexpr int PS = 60;
constexpr int PD = 5550;
constexpr int DY = 1375;
constexpr int RPUB = 9;
constexpr int TG1 = 50, TG2 = 150, TG3 = 250, TX4 = 350, TI2 = 150, TI3 = 300;
constexpr int TCL = 100, TCW = 150, TCF = 360;
constexpr int ZDELAY = 300, ZSPLIT = 3584, YV = 50;
constexpr int FI = 11, FC = 5, XA = 2, NEP = 10;
constexpr int KD = 2048, NS = 256, MS = 256;
constexpr size_t SLOT = 8u << 20;
constexpr size_t DATA_OFF = 1u << 20;
constexpr size_t B_OFF = 4u << 20;
constexpr size_t G_OFF = 6u << 20;
constexpr int ZERO_KB = 6144 + 9;
constexpr u64 DEADLINE = 20000000ull;
constexpr int NT = 512;
constexpr int LDS_BYTES = 131072 + 256;
struct Args {
  char* me; const float* x; const float* w; float* out;
  int rank; int pad[7];
  char* ptr[12][4];
  unsigned tab[16];
};
__device__ __forceinline__ char* gp(u64 v) { return (char*)(__attribute__((address_space(1))) char*)v; }
__device__ __forceinline__ u64 rt() { return __builtin_amdgcn_s_memrealtime(); }
__device__ __forceinline__ void st16_sys(void* p, v4u v) { asm volatile("global_store_dwordx4 %0, %1, off sc0 sc1\n\ts_nop 1" :: "v"(p), "v"(v) : "memory"); }
__device__ __forceinline__ void st16_so(u64 base, unsigned off, v4u v) { asm volatile("global_store_dwordx4 %0, %1, %2 sc0 sc1\n\ts_nop 1" :: "v"(off), "v"(v), "s"(base) : "memory"); }
__device__ __forceinline__ void st4_sys(void* p, unsigned v) { __hip_atomic_store((unsigned*)p, v, __ATOMIC_RELAXED, __HIP_MEMORY_SCOPE_SYSTEM); }
__device__ __forceinline__ unsigned ld_sys(const void* p) { return __hip_atomic_load((const unsigned*)p, __ATOMIC_RELAXED, __HIP_MEMORY_SCOPE_SYSTEM); }
__device__ __forceinline__ unsigned ld_dev(const void* p) { return __hip_atomic_load((const unsigned*)p, __ATOMIC_RELAXED, __HIP_MEMORY_SCOPE_AGENT); }
#define LDMOD " sc0 sc1"
#define L4(a, b, c, d, off, base) "global_load_dwordx4 %" #a ", %" #off ", %" #base LDMOD "\n\tglobal_load_dwordx4 %" #b ", %" #off ", %" #base " offset:1024" LDMOD \
  "\n\tglobal_load_dwordx4 %" #c ", %" #off ", %" #base " offset:2048" LDMOD "\n\tglobal_load_dwordx4 %" #d ", %" #off ", %" #base " offset:3072" LDMOD "\n\t"
__device__ __forceinline__ void ld16m(v4u* f, unsigned loff, const char* b0, const char* b1) {
  asm volatile("s_nop 4\n\t" L4(0, 1, 2, 3, 16, 17) L4(4, 5, 6, 7, 16, 18) L4(8, 9, 10, 11, 16, 19) L4(12, 13, 14, 15, 16, 20) "s_waitcnt vmcnt(0)"
               : "=&v"(f[0]), "=&v"(f[1]), "=&v"(f[2]), "=&v"(f[3]), "=&v"(f[4]), "=&v"(f[5]), "=&v"(f[6]), "=&v"(f[7]),
                 "=&v"(f[8]), "=&v"(f[9]), "=&v"(f[10]), "=&v"(f[11]), "=&v"(f[12]), "=&v"(f[13]), "=&v"(f[14]), "=&v"(f[15])
               : "v"(loff), "s"(b0), "s"(b0 + 4096), "s"(b1), "s"(b1 + 4096) : "memory");
}
__device__ __forceinline__ void ld8m(v4u* f, unsigned loff, const char* b0, const char* b1) {
  asm volatile("s_nop 4\n\t" L4(0, 1, 2, 3, 8, 9) L4(4, 5, 6, 7, 8, 10) "s_waitcnt vmcnt(0)"
               : "=&v"(f[0]), "=&v"(f[1]), "=&v"(f[2]), "=&v"(f[3]), "=&v"(f[4]), "=&v"(f[5]), "=&v"(f[6]), "=&v"(f[7])
               : "v"(loff), "s"(b0), "s"(b1) : "memory");
}
__device__ __forceinline__ unsigned min4(unsigned m, v4u f) { m = min(min(m, f.x), f.y); return min(min(m, f.z), f.w); }
template <int N> __device__ __forceinline__ bool all_ok(const v4u* f) {
  unsigned m = 0xffffffffu;
#pragma unroll
  for (int i = 0; i < N; ++i) m = min4(m, f[i]);
  return __all(m != 0);
}
__device__ __forceinline__ unsigned pkz(float a, float b) {
  v2h h; h.x = (_Float16)a; h.y = (_Float16)b; unsigned u = __builtin_bit_cast(unsigned, h); return u == 0 ? 0x8000u : u; }
template <int N> __device__ __forceinline__ bool wait_frags(v4u* f, unsigned loff, const char* b0, const char* b1, const char* hp, u64 t0) {
  for (;;) {
    int it = 0;
    while (!__all(ld_sys(hp) != 0) && ++it < 64) __builtin_amdgcn_s_sleep(1);
    if (N == 16) ld16m(f, loff, b0, b1); else ld8m(f, loff, b0, b0 + 4096);
    if (all_ok<N>(f)) return true;
    if (rt() - t0 > DEADLINE) return false;
  }
}
__device__ __forceinline__ int idx_row(int G, int idx) {
  if (idx < 4 * FI) return (G * 4 + idx / FI) * MS + (idx % FI) * 16;
  return ((G ^ 1) * 4 + (idx - 4 * FI) / FC) * MS + (FI + (idx - 4 * FI) % FC) * 16;
}
__device__ __forceinline__ int ks_of(int kp, int rb) {
  const int w6 = (kp * 43) >> 8, m = (kp - 48) >> 31;
  const int w = (w6 & m) | (kp & ~m), k = ((kp - 6 * w6) & m) | ((kp >> 3) & ~m);
  return ((w + 2 * rb) & 7) * 8 + (k & 7);
}
__device__ __forceinline__ int send_order(int u, int lane, bool intra) {
  const int f = u + 32 * lane, fl = intra ? 512 : 128;
  const int g3 = max(f - fl, 0) / 3;
  const int rb = f >= fl ? (intra ? 8 : 2) + (f - fl) - 3 * g3 : intra ? (f & 3) + ((f >> 8) << 2) : f & 1;
  const int kp = f >= fl ? g3 : intra ? f >> 2 : f >> 1;
  const int ks = intra && f >= fl ? ((kp & 7) << 3) | ((kp >> 3) & 7) : ks_of(kp & 63, rb);
  return ((intra ? rb : rb + FI) * 64 + ks) & 1023;
}
struct XL { v4f p0, p1; };
__device__ __forceinline__ XL x_load(const float* x, int e, int lane) {
  const float* src = x + (size_t)((e >> 6) * 16 + (lane >> 3)) * KD + (e & 63) * 32 + (lane & 7) * 4;
  XL r; r.p0 = *(const v4f*)src; r.p1 = *(const v4f*)(src + 8 * KD); return r;
}
__device__ __forceinline__ void x_stage(char* frag, const XL& r, int lane) {
  char* ldst = frag + ((lane >> 3) + 16 * ((lane & 7) >> 1)) * 16 + (lane & 1) * 8;
  v2u v0, v1; v0.x = pkz(r.p0.x, r.p0.y); v0.y = pkz(r.p0.z, r.p0.w); v1.x = pkz(r.p1.x, r.p1.y); v1.y = pkz(r.p1.z, r.p1.w);
  *(v2u*)ldst = v0; *(v2u*)(ldst + 128) = v1;
}
__device__ __forceinline__ void pace(u64 t0, unsigned when100) {
  const u64 until = t0 + when100 / 100;
  while (rt() < until) __builtin_amdgcn_s_sleep(1);
}
__device__ __forceinline__ int gate_read(const char* slot, u64 t0, int lane) {
  const unsigned e = ld_dev(slot + G_OFF + (lane & 31) * 128);
  if (!__all(e != 0)) return -1;
  int rel = min(max((int)(e - (unsigned)t0), 0), 100000);
#pragma unroll
  for (int m = 1; m < 32; m <<= 1) rel += __shfl_xor(rel, m);
  return __builtin_amdgcn_readfirstlane(rel) >> 5;
}
__device__ __forceinline__ unsigned lds_ld(const unsigned* p) { return __hip_atomic_load(p, __ATOMIC_RELAXED, __HIP_MEMORY_SCOPE_WORKGROUP); }
__device__ __forceinline__ void lds_wait(const unsigned* p, unsigned n, u64 t0) {
  for (unsigned it = 1; lds_ld(p) < n; ++it) { __builtin_amdgcn_s_sleep(1); if ((it & 255) == 0 && rt() - t0 > DEADLINE) break; }
  asm volatile("" ::: "memory");
}
#define MFMA(b, a, c) __builtin_amdgcn_mfma_f32_16x16x32_f16(__builtin_bit_cast(v8h, b), __builtin_bit_cast(v8h, a), c, 0, 0, 0)
#define BIDX(c, k) (((c) >> 1) * 16 + ((k) & 1) * 8 + ((k) >> 1) * 2 + ((c) & 1))
template <int K0, int K1> __device__ __forceinline__ void mma(const v4u* B, const v4u* f, v4f* acc) {
#pragma unroll
  for (int k = K0; k < K1; ++k)
#pragma unroll
    for (int c = 0; c < 4; ++c) acc[c] = MFMA(B[BIDX(c, k)], f[k], (k == 0 ? (v4f){0, 0, 0, 0} : acc[c]));
}
__device__ __forceinline__ void put_part(char* p, v4f v, bool dead) {
  if (dead) v = (v4f){__builtin_nanf(""), __builtin_nanf(""), __builtin_nanf(""), __builtin_nanf("")};
  *(v4f*)p = v;
}
__device__ __forceinline__ bool task_head(const v4u* B, const v4u* f, char* sl, bool dead, bool& full) {
  if (!dead && !all_ok<6>(f)) return false;
  v4f acc[4];
  mma<0, 6>(B, f, acc);
  full = dead || all_ok<2>(f + 6);
  if (full) mma<6, 8>(B, f, acc);
#pragma unroll
  for (int c = 0; c < 4; ++c) put_part(sl + c * 1024, acc[c], dead);
  return true;
}
__device__ __forceinline__ bool task_tail(const v4u* B, const v4u* f, v4f* acc, char* p, bool dead) {
  if (!dead && !all_ok<2>(f + 6)) return false;
  mma<6, 8>(B, f, acc);
#pragma unroll
  for (int i = 0; i < 4; ++i) put_part(p + i * 1024, acc[i], dead);
  return true;
}
__device__ __forceinline__ void run_pair(const v4u* B, char* sl, const char* abase, unsigned tb, int lane, u64 t0, bool& dead) {
  const unsigned loff = lane * 16;
  const unsigned hb = ((tb >> (8 * (lane & 1))) & 63u) * 65536u;
  const char* a0 = abase + (size_t)(tb & 63) * 65536;
  const char* a1 = abase + (size_t)((tb >> 8) & 63) * 65536;
  unsigned head = 3, tail = 0, it = 0;
  while (head | tail) {
    if ((++it & 31) == 0 && rt() - t0 > DEADLINE) dead = true;
    unsigned c = tail;
    if (head) {
      const unsigned h = ld_sys(abase + hb + ((tail >> (lane & 1)) & 1 ? 8188u : 6140u));
      c = (dead ? 3u : (unsigned)__ballot(h != 0)) & (head | tail);
      if (!c) { __builtin_amdgcn_s_sleep(1); continue; }
    }
    if ((c & tail) == 3) {
      v4u f[8], e[8]; v4f acc0[4], acc1[4];
#pragma unroll
      for (int i = 0; i < 4; ++i) { acc0[i] = *(const v4f*)(sl + i * 1024); acc1[i] = *(const v4f*)(sl + 32768 + i * 1024); }
      asm volatile("s_nop 4\n\tglobal_load_dwordx4 %0, %4, %5 offset:3072" LDMOD "\n\tglobal_load_dwordx4 %1, %4, %5 offset:2048" LDMOD
                   "\n\tglobal_load_dwordx4 %2, %4, %6 offset:3072" LDMOD "\n\tglobal_load_dwordx4 %3, %4, %6 offset:2048" LDMOD "\n\ts_waitcnt vmcnt(0)"
                   : "=&v"(f[7]), "=&v"(f[6]), "=&v"(e[7]), "=&v"(e[6]) : "v"(loff), "s"(a0 + 4096), "s"(a1 + 4096) : "memory");
      if (task_tail(B, f, acc0, sl, dead)) tail &= ~1u;
      if (task_tail(B, e, acc1, sl + 32768, dead)) tail &= ~2u;
      continue;
    }
    if (c & tail) {
      const unsigned bit = c & tail;
      char* p = sl + (bit >> 1) * 32768;
      v4u f[8]; v4f acc[4];
#pragma unroll
      for (int i = 0; i < 4; ++i) acc[i] = *(const v4f*)(p + i * 1024);
      asm volatile("s_nop 4\n\tglobal_load_dwordx4 %0, %2, %3 offset:3072" LDMOD "\n\tglobal_load_dwordx4 %1, %2, %3 offset:2048" LDMOD "\n\ts_waitcnt vmcnt(0)"
                   : "=&v"(f[7]), "=&v"(f[6]) : "v"(loff), "s"((bit == 1 ? a0 : a1) + 4096) : "memory");
      if (task_tail(B, f, acc, p, dead)) tail &= ~bit;
    }
    c &= head;
    if (!c) continue;
    v4u f[16]; bool full;
    if (c == 3) {
      ld16m(f, loff, a0, a1);
      if (task_head(B, f + 8, sl + 32768, dead, full)) { head &= ~2u; if (!full) tail |= 2u; }
    } else { const char* ap = c == 1 ? a0 : a1; ld8m(f, loff, ap, ap + 4096); }
    const unsigned bit = c == 2 ? 2u : 1u;
    if (task_head(B, f, sl + (bit >> 1) * 32768, dead, full)) { head &= ~bit; if (!full) tail |= bit; }
  }
}
template <int K> __device__ __forceinline__ void ld_step(v4u* f, unsigned loff, const char* a0, const char* a1) {
  if (K == 7) asm volatile("s_nop 4\n\tglobal_load_dwordx4 %0, %2, %3" LDMOD "\n\tglobal_load_dwordx4 %1, %2, %4" LDMOD "\n\ts_waitcnt vmcnt(0)"
                           : "=&v"(f[7]), "=&v"(f[15]) : "v"(loff), "s"(a0 + 7168), "s"(a1 + 7168) : "memory");
  else asm volatile("s_nop 4\n\tglobal_load_dwordx4 %0, %4, %5" LDMOD "\n\tglobal_load_dwordx4 %1, %4, %6" LDMOD
                    "\n\tglobal_load_dwordx4 %2, %4, %5 offset:1024" LDMOD "\n\tglobal_load_dwordx4 %3, %4, %6 offset:1024" LDMOD "\n\ts_waitcnt vmcnt(0)"
                    : "=&v"(f[K]), "=&v"(f[8 + K]), "=&v"(f[(K + 1) & 7]), "=&v"(f[8 + ((K + 1) & 7)]) : "v"(loff), "s"(a0 + K * 1024), "s"(a1 + K * 1024) : "memory");
}
template <int K> __device__ __forceinline__ void inc_step(const v4u* B, v4u* f, v4f* acc0, v4f* acc1, unsigned loff, const char* a0, const char* a1, u64 t0, bool& dead) {
  for (unsigned it = 0; !dead && !__all(min4(min4(0xffffffffu, f[K]), f[8 + K]) != 0); ++it) {
    if (K < 7 && it) __builtin_amdgcn_s_sleep(2);
    ld_step<K>(f, loff, a0, a1);
    if ((it & 31) == 31 && rt() - t0 > DEADLINE) dead = true;
  }
  mma<K, K + 1>(B, f, acc0); mma<K, K + 1>(B, f + 8, acc1);
}
__device__ __forceinline__ void run_inc(const v4u* B, char* sl, const char* abase, unsigned tb, int lane, u64 t0, bool& dead) {
  const unsigned loff = lane * 16;
  const char* a0 = abase + (size_t)(tb & 63) * 65536;
  const char* a1 = abase + (size_t)((tb >> 8) & 63) * 65536;
  for (unsigned it = 1; !dead && !__all(ld_sys((lane & 1 ? a1 : a0) + 1020) != 0); ++it) {
    __builtin_amdgcn_s_sleep(1);
    if ((it & 31) == 0 && rt() - t0 > DEADLINE) dead = true;
  }
  v4u f[16], g[8]; v4f acc0[4], acc1[4];
  ld8m(g, loff, a0, a1);
#pragma unroll
  for (int i = 0; i < 4; ++i) { f[i] = g[i]; f[8 + i] = g[4 + i]; f[4 + i] = f[12 + i] = (v4u){0, 0, 0, 0}; }
  inc_step<0>(B, f, acc0, acc1, loff, a0, a1, t0, dead); inc_step<1>(B, f, acc0, acc1, loff, a0, a1, t0, dead);
  inc_step<2>(B, f, acc0, acc1, loff, a0, a1, t0, dead); inc_step<3>(B, f, acc0, acc1, loff, a0, a1, t0, dead);
  inc_step<4>(B, f, acc0, acc1, loff, a0, a1, t0, dead); inc_step<5>(B, f, acc0, acc1, loff, a0, a1, t0, dead);
  inc_step<6>(B, f, acc0, acc1, loff, a0, a1, t0, dead); inc_step<7>(B, f, acc0, acc1, loff, a0, a1, t0, dead);
#pragma unroll
  for (int i = 0; i < 4; ++i) { put_part(sl + i * 1024, acc0[i], dead); put_part(sl + 32768 + i * 1024, acc1[i], dead); }
}
__device__ __forceinline__ void finish(const char* red, float* dst, const char* slot, u64 t0, int lane, int gate, int yslot) {
  if (gate == -1) gate = gate_read(slot, t0, lane);
  __syncthreads();
  v4f s0 = *(const v4f*)red;
#pragma unroll
  for (int w = 1; w < 8; ++w) s0 += *(const v4f*)(red + w * 4096);
  v4u y0 = __builtin_bit_cast(v4u, s0);
#pragma unroll
  for (int i = 0; i < 4; ++i) if (y0[i] == 0) y0[i] = 0x80000000u;
  for (int it = 0; gate == -1 && it < 64; ++it) { gate = gate_read(slot, t0, lane); if (gate < 0) __builtin_amdgcn_s_sleep(16); }
  if (gate >= 0) pace(t0, gate * 100 + yslot * DY);
  st16_sys(dst, y0);
}
__device__ __forceinline__ void verify(float* outp, const char* slot, char* me, int G, int tile, int lane, u64 t0, unsigned call) {
  const char* ysrc = (const char*)(outp + (size_t)idx_row(G ^ 1, tile >> 2) * NS + (tile & 3) * 64);
  const unsigned yo = (lane >> 3) * 1024 + (lane & 7) * 16;
  bool dead = false, ok = false;
  int gate;
  while ((gate = gate_read(slot, t0, lane)) < 0) { __builtin_amdgcn_s_sleep(16); if (rt() - t0 > DEADLINE) { dead = true; break; } }
  if (!dead) pace(t0, (gate + YV) * 100);
  for (unsigned it = 1; !dead; ++it) {
    if (!ok) {
      v4u y[4];
      asm volatile("s_nop 4\n\tglobal_load_dwordx4 %0, %4, %5" LDMOD "\n\tglobal_load_dwordx4 %1, %4, %5 offset:128" LDMOD
                   "\n\tglobal_load_dwordx4 %2, %4, %6" LDMOD "\n\tglobal_load_dwordx4 %3, %4, %6 offset:128" LDMOD "\n\ts_waitcnt vmcnt(0)"
                   : "=&v"(y[0]), "=&v"(y[1]), "=&v"(y[2]), "=&v"(y[3]) : "v"(yo), "s"(ysrc), "s"(ysrc + 8192) : "memory");
      ok = min4(min4(min4(min4(0xffffffffu, y[0]), y[1]), y[2]), y[3]) != 0;
    }
    if (__all(ok)) break;
    __builtin_amdgcn_s_sleep(1);
    if ((it & 31) == 0 && rt() - t0 > DEADLINE) dead = true;
  }
  if (dead) {
    const v4u nan4 = {0x7fc00000u, 0x7fc00000u, 0x7fc00000u, 0x7fc00000u};
#pragma unroll
    for (int c = 0; c < 4; ++c) st16_sys((char*)ysrc + yo + (c & 1) * 128 + (c >> 1) * 8192, nan4);
    if (lane == 0) st4_sys(me + 4096, call);
  }
}

extern "C" __global__ void __launch_bounds__(NT) agk(Args a) {
  extern __shared__ __attribute__((aligned(16))) char lds[];
  const s8u ka = *(const volatile __attribute__((address_space(4))) s8u*)__builtin_amdgcn_kernarg_segment_ptr();
  const u64 t0 = rt();
  __builtin_amdgcn_sched_barrier(0);
  const int tid = threadIdx.x, lane = tid & 63, wave = __builtin_amdgcn_readfirstlane(tid >> 6), b = blockIdx.x;
  const bool intra = b < 32;
  const int u = b & 31;
  unsigned* lflag = (unsigned*)(lds + 131072);
  lflag[tid & 31] = 0;
  const int f0 = u + 32 * wave, kp0 = (f0 >> (intra ? 2 : 1)) & 63, rb0 = f0 & (intra ? 3 : 1);
  const int e0 = (rb0 + (intra ? 0 : FI)) * 64 + ks_of(kp0, rb0);
  const int xl0 = (lane & 15) * KD + (lane >> 4) * 8, xl1 = (lane >> 3) * KD + (lane & 7) * 4;
  unsigned xa = (unsigned)((e0 >> 6) * 16 * KD + (e0 & 63) * 32 + (tid < 64 ? xl0 : xl1)) * 4u, xb = xa + (tid < 64 ? 16u : 32u * KD);
  asm volatile("" : "+v"(xa), "+v"(xb));
  __builtin_amdgcn_sched_barrier(0);
  char* const me = gp(ka[0] | ((u64)ka[1] << 32));
  const float* const xp = (const float*)gp(ka[2] | ((u64)ka[3] << 32));
  const float* const wp = (const float*)gp(ka[4] | ((u64)ka[5] << 32));
  float* const outp = (float*)gp(ka[6] | ((u64)ka[7] << 32));
  XL r0;
  if (b < 64 && (intra || wave < XA)) { r0.p0 = *(const v4f*)((const char*)xp + xa); r0.p1 = *(const v4f*)((const char*)xp + xb); }
  unsigned* ctr = (unsigned*)(me + 65536) + b * 16 + wave;
  const int mb = (b - 64) >> 31;
  const unsigned rowo = 64 + 32 * (((((b >> 5) << 2) + (b & 3)) & mb) | ((8 + (b & 1) + ((b >> 2) & 2)) & ~mb));
#define BOOT() \
  unsigned cprev; s8u pr8; int rank; \
  asm volatile("s_load_dword %0, %3, 0x0 glc\n\ts_load_dwordx8 %1, %4, %5\n\ts_load_dword %2, %4, 0x20\n\ts_waitcnt lgkmcnt(0)" \
               : "=&s"(cprev), "=&s"(pr8), "=&s"(rank) : "s"(ctr), "s"(__builtin_amdgcn_kernarg_segment_ptr()), "s"(rowo) : "memory"); \
  const unsigned call = cprev + 1, so = (call % 3) * (unsigned)SLOT; \
  char* const slot = me + DATA_OFF + so; \
  const int G = rank >> 2, gi4 = rank & 3
  const unsigned loff = lane * 16;
  if (__builtin_expect(b < 64 && wave != 0, 0)) {
    auto wload = [&](v4f* q, int g, int hf) __attribute__((always_inline)) {
      const float* wsrc = wp + (size_t)(u * 64 + (g & 1) * 32 + (hf * 2 + (lane >> 5)) * 8) * NS + ((lane & 31) >> 3) * 64 + (g >> 1) * 32 + (lane & 7) * 4;
#pragma unroll
      for (int i = 0; i < 8; ++i) q[i] = *(const v4f*)(wsrc + i * NS);
    };
    auto wstore = [&](const v4f* q, int g, int hf) __attribute__((always_inline)) {
      const int t = g >> 1, kk = g & 1, kg = hf * 2 + (lane >> 5), j = (lane & 31) >> 3, c4 = lane & 7;
      char* dstl = lds + 32768 + (kk * 16 + j * 4 + t * 2 + (c4 >> 2)) * 1024 + (kg * 16 + (c4 & 3) * 4) * 16;
#pragma unroll
      for (int c = 0; c < 4; ++c) {
        v4u o; o.x = pkz(q[0][c], q[1][c]); o.y = pkz(q[2][c], q[3][c]); o.z = pkz(q[4][c], q[5][c]); o.w = pkz(q[6][c], q[7][c]);
        *(v4u*)(dstl + c * 16) = o;
      }
      asm volatile("s_waitcnt lgkmcnt(0)" ::: "memory");
      if (lane == 0) __hip_atomic_fetch_add(lflag + 24 + g, 1u, __ATOMIC_RELAXED, __HIP_MEMORY_SCOPE_WORKGROUP);
    };
    auto put = [&](int j, const XL& x) __attribute__((always_inline)) {
      x_stage(lds + j * 1024, x, lane);
      asm volatile("s_waitcnt lgkmcnt(0)" ::: "memory");
      if (lane == 0) __hip_atomic_store(lflag + j, 1u, __ATOMIC_RELAXED, __HIP_MEMORY_SCOPE_WORKGROUP);
    };
    v4f wq0[8];
    if (!intra && (wave >> 1) == 1) wload(wq0, 0, wave & 1);
    BOOT();
    asm volatile("s_barrier" ::: "memory");
    if (intra || wave < XA) { asm volatile("" : "+v"(r0.p0), "+v"(r0.p1)); put(wave, r0); }
#pragma unroll 1
    for (int tk = wave - 2; tk < 8 && !intra && wave >= 2; tk += 6) {
      if (tk >= 2) { pace(t0, (tk < 4 ? TG1 : tk < 6 ? TG2 : TG3) * 100); wload(wq0, tk >> 1, tk & 1); }
      wstore(wq0, tk >> 1, tk & 1);
    }
    int lane2 = lane; asm volatile("" : "+v"(lane2));
    const int etab = send_order(u, lane2, intra);
    if (lane == 0) st4_sys(ctr, call);
    const v4u zero = {0, 0, 0, 0};
    if (wave == 1) {
      unsigned e = 0;
      for (int it = 0; it < 8 && e == 0; ++it) e = ld_dev(slot + G_OFF + 32 * 128);
      const int d = (int)((unsigned)t0 - e);
      if (e != 0 && d > 0 && d < 200 && lane == 0) __hip_atomic_store(lflag + 28, (unsigned)d, __ATOMIC_RELAXED, __HIP_MEMORY_SCOPE_WORKGROUP);
    }
    if (intra) {
      char* ldst = slot + (size_t)(gi4 * FI * 64) * 1024 + loff;
#pragma unroll 1
      for (int i = 1; i < 4; ++i) {
        const int j = wave + 7 * i;
        XL r;
        if (i < 3) { pace(t0, (i == 1 ? TI2 : TI3) * 100); r = x_load(xp, __builtin_amdgcn_readlane(etab, j), lane); }
        st16_sys(ldst + (size_t)__builtin_amdgcn_readlane(etab, j - 7) * 1024, *(const v4u*)(lds + (j - 7) * 1024 + loff));
        if (i < 3) put(j, r);
      }
    } else if (wave == 1) {
      pace(t0, TCF * 100);
      while (!__all(ld_sys(slot + G_OFF + (40 + u) * 128 + (lane & 31) * 4) != 0) && rt() - t0 < DEADLINE) __builtin_amdgcn_s_sleep(4);
      if (lane == 0) __hip_atomic_store(lflag + 29, 1u, __ATOMIC_RELAXED, __HIP_MEMORY_SCOPE_WORKGROUP);
    } else if (wave >= 4) {
      const int j = 2 * wave - 6;
      pace(t0, (TX4 + (wave >= 6 ? 60 : 0)) * 100);
      XL r[2];
#pragma unroll
      for (int i = 0; i < 2; ++i) r[i] = x_load(xp, __builtin_amdgcn_readlane(etab, j + i), lane);
#pragma unroll
      for (int i = 0; i < 2; ++i) put(j + i, r[i]);
    }
    char* z = me + DATA_OFF + (size_t)((call + 2) % 3) * SLOT + loff;
    pace(t0, ZDELAY * 100);
    for (int k = u + 32 * (wave - 1) + (intra ? 0 : ZSPLIT); k < (intra ? ZSPLIT : ZERO_KB); k += 224) {
      st16_sys(z + (size_t)k * 1024, zero); asm volatile("s_waitcnt vmcnt(1)" ::: "memory"); __builtin_amdgcn_s_sleep(2); }
    if (!intra) verify(outp, slot, me, G, wave * 32 + u, lane, t0, call);
    return;
  }
  BOOT();
  const u64 q0 = pr8[0] | ((u64)pr8[1] << 32), q1 = pr8[2] | ((u64)pr8[3] << 32), q2 = pr8[4] | ((u64)pr8[5] << 32), q3 = pr8[6] | ((u64)pr8[7] << 32);
  if (__builtin_expect(b < 64, 1)) {
    {
      unsigned o = so + e0 * 1024 + loff;
      asm volatile("" : "+v"(o));
      asm volatile("" : "+v"(r0.p0), "+v"(r0.p1));
      v4u v; v.x = pkz(r0.p0.x, r0.p0.y); v.y = pkz(r0.p0.z, r0.p0.w); v.z = pkz(r0.p1.x, r0.p1.y); v.w = pkz(r0.p1.z, r0.p1.w);
      st16_so(q0, o, v); st16_so(q1, o, v); st16_so(q2, o, v); st16_so(q3, o, v);
    }
    asm volatile("s_barrier" ::: "memory");
    const int etab = send_order(u, lane, intra);
    unsigned late = 0, when = PS * 100 + PD + u * (PD / 32);
    if (!intra) {
#pragma unroll 1
      for (int r = 1; r < 2 * FC + 8; ++r) {
        v4u vv[4]; unsigned o;
        if (r >= XA && r < XA + 8) {
          const int n = r - XA, t = n >> 2, kk = (n >> 1) & 1, cb = n & 1;
          if (cb == 0) lds_wait(lflag + 24 + (n >> 1), 2, t0);
#pragma unroll
          for (int d = 0; d < 4; ++d) vv[d] = *(const v4u*)(lds + 32768 + (kk * 16 + t * 2 + cb) * 1024 + ((d + b) & 3) * 4096 + loff);
          o = (unsigned)B_OFF + (8 + gi4 * 2 + t) * 131072 + (kk * 64 + u * 2 + cb) * 1024 - (3 * FI + gi4 * FC) * 65536;
        } else {
          const int j = r < XA ? r : r - 8;
          lds_wait(lflag + j, 1, t0);
          vv[0] = vv[1] = vv[2] = vv[3] = *(const v4u*)(lds + j * 1024 + loff);
          o = __builtin_amdgcn_readlane(etab, j) * 1024;
        }
        o += so + loff;
        if (r == RPUB) lds_wait(lflag + 29, 1, t0);
        if (!late) late = __builtin_amdgcn_readfirstlane(lds_ld(lflag + 28));
        pace(t0 - late, when); when += PD;
        asm volatile("s_waitcnt vmcnt(8)" ::: "memory");
        if (r == RPUB) {
          const unsigned sched = (unsigned)(t0 - late) + PS + (unsigned)((2 * FC + 8) * PD) / 100;
          const unsigned pred = (unsigned)rt() + (unsigned)((2 * FC + 8 - RPUB) * PD - u * (PD / 32)) / 100;
          if (lane == 0) st4_sys(slot + G_OFF + u * 128, ((int)(pred - sched) > 0 ? pred : sched) | 1u);
        }
        st16_so(q0, o, vv[0]); st16_so(q1, o, vv[1]); st16_so(q2, o, vv[2]); st16_so(q3, o, vv[3]);
      }
    } else {
#pragma unroll 1
      for (int j = 1; j < 2 * FI; ++j) {
        lds_wait(lflag + j, 1, t0);
        const v4u v = *(const v4u*)(lds + j * 1024 + loff);
        const unsigned o = __builtin_amdgcn_readlane(etab, j) * 1024 + so + loff;
        if (!late) late = __builtin_amdgcn_readfirstlane(lds_ld(lflag + 28));
        pace(t0 - late, when); when += PD;
        asm volatile("s_waitcnt vmcnt(6)" ::: "memory");
        st16_so(q0, o, v); st16_so(q1, o, v); st16_so(q2, o, v);
      }
    }
    if (lane == 0) st4_sys(ctr, call);
    if (!intra) verify(outp, slot, me, G, u, lane, t0, call);
    return;
  }
  const int X = b & 7, ci = (b - 64) >> 3;
  const bool local = ci < 8;
  v2f wq[8];
  if (local) {
    const float* wsrc = wp + (size_t)((b - 64) * 32 + (wave & 3) * 8) * NS + (wave >> 2) * 128 + lane * 2;
#pragma unroll
    for (int j = 0; j < 8; ++j) wq[j] = *(const v2f*)(wsrc + j * NS);
  }
  const int pr = ((local ? ci : ci - 8) >> 1) * 4 + (X >> 1);
  const int c64 = (local ? 0 : 4) + (X & 1) + 2 * (ci & 1);
  const int half = pr & 1, g = pr >> 1;
  const int yslot = (pr & ~3) + ((pr + (c64 & 2)) & 3);
  if (b == 129 && tid == 0) st4_sys(slot + G_OFF + 32 * 128, (unsigned)t0 | 1u);
  if (lane == 0) st4_sys(ctr, call);
  bool dead = false;
  const char* abase = slot + wave * 8192;
  const char* bsrc = slot + B_OFF + (size_t)c64 * 262144 + wave * 8192;
  char* part = lds + wave * 4096 + loff;
  const char* red = lds + (wave >> 2) * 32768 + (wave & 3) * 1024 + loff;
  const unsigned tb0 = (a.tab[g] >> (16 * half)) & 0xffffu;
  const size_t ooff = (size_t)(lane & 15) * NS + (wave & 3) * 16 + (lane >> 4) * 4;
  float* yd = (float*)gp(q0) + (size_t)idx_row(G, (tb0 >> (8 * (wave >> 2))) & 63) * NS + gi4 * 64 + ooff;
  if (local) {
    const int col = (wave >> 2) * 128 + lane * 2;
    char* dstl = lds + 65536 + (col >> 4) * 1024 + ((wave & 3) * 16 + (col & 15)) * 16;
#pragma unroll
    for (int c = 0; c < 2; ++c) {
      v4u o; o.x = pkz(wq[0][c], wq[1][c]); o.y = pkz(wq[2][c], wq[3][c]); o.z = pkz(wq[4][c], wq[5][c]); o.w = pkz(wq[6][c], wq[7][c]);
      *(v4u*)(dstl + c * 16) = o;
    }
    __syncthreads();
    const v4u f0v = *(const v4u*)(lds + 65536 + wave * 2048 + loff), f1v = *(const v4u*)(lds + 65536 + wave * 2048 + 1024 + loff);
    char* bl = slot + B_OFF + (size_t)wave * 131072 + (b & 1) * 65536 + ((b - 64) >> 1) * 2048 + loff;
    st16_sys(bl, f0v); st16_sys(bl + 1024, f1v);
  }
  if (!local) {
    const int n = (b - 128) * 8 + wave;
    pace(t0, TCL * 100 + (wave * 128 + b - 128) * (TCW * 100 / 1024));
    st16_sys(outp + (size_t)(idx_row(G ^ 1, n >> 4) + (n & 15)) * NS + lane * 4, (v4u){0, 0, 0, 0});
    asm volatile("s_waitcnt vmcnt(0)" ::: "memory");
    if (lane == 0) st4_sys(slot + G_OFF + (40 + (n >> 5)) * 128 + (n & 31) * 4, 1u);
  }
  if (tid == 448) (void)ld_sys(local ? outp + b * 2048 : yd);
  if (!local && pr < NEP) {
    v4u A[16], B[8];
    const char* a0 = abase + (size_t)(tb0 & 63) * 65536;
    const char* a1 = abase + (size_t)((tb0 >> 8) & 63) * 65536;
    if (!wait_frags<16>(A, loff, a0, a1, ((lane & 1) ? a1 : a0) + 8188, t0)) dead = true;
#pragma unroll 1
    for (int t = 0; t < 2; ++t) {
      v4f acc[4];
#pragma unroll
      for (int c = 0; c < 4; ++c) acc[c] = (v4f){0, 0, 0, 0};
#pragma unroll
      for (int kk = 0; kk < 2; ++kk) {
        const char* bs = bsrc + (t * 2 + kk) * 65536;
        if (!wait_frags<8>(B, loff, bs, bs, bs + (lane & 3) * 2048 + 2044, t0)) dead = true;
#pragma unroll
        for (int i = 0; i < 4; ++i)
#pragma unroll
          for (int c = 0; c < 4; ++c) acc[c] = MFMA(B[i * 2 + (c & 1)], A[(c >> 1) * 8 + i * 2 + kk], acc[c]);
      }
#pragma unroll
      for (int c = 0; c < 4; ++c) put_part(part + (c >> 1) * 32768 + t * 2048 + (c & 1) * 1024, acc[c], dead);
    }
    finish(red, yd, slot, t0, lane, -1, yslot);
  } else {
    v4u B[32];
    if (local) {
#pragma unroll
      for (int q = 0; q < 4; q += 2) {
        const char* bs = bsrc + q * 65536;
        if (!wait_frags<16>(B + 8 * q, loff, bs, bs + 65536, bs + (lane & 3) * 2048 + (lane & 4) * 16384 + 2044, t0)) dead = true;
      }
    } else {
#pragma unroll
      for (int q = 0; q < 4; ++q) {
        const char* bs = bsrc + q * 65536;
        if (!wait_frags<8>(B + 8 * q, loff, bs, bs, bs + (lane & 3) * 2048 + 2044, t0)) dead = true;
      }
    }
#pragma unroll 1
    for (int pass = local ? 0 : 1; pass < 2; ++pass) {
      const unsigned tb = local && pass ? (a.tab[g + 8] >> (16 * half)) & 0xffffu : tb0;
      const int gate = local ? -2 : gate_read(slot, t0, lane);
      if (pass && (local ? pr + 16 : pr) >= 27) run_inc(B, part + pass * 65536, abase, tb, lane, t0, dead);
      else run_pair(B, part + pass * 65536, abase, tb, lane, t0, dead);
      float* dst = local ? outp + (size_t)idx_row(G, (tb >> (8 * (wave >> 2))) & 63) * NS + c64 * 64 + ooff : yd;
      finish(red + pass * 65536, dst, slot, t0, lane, gate, yslot);
    }
  }
  if (dead && lane == 0) st4_sys(me + 4096, call);
}

extern "C" void kernel_launch(int rank, int nranks, ncclComm_t comm, hipStream_t stream, void* const* d_in, const int* in_sizes, int n_in,
                              void* d_out, int out_size, const int* in_dtypes, int out_dtype, void* const* d_out_all, void* const* d_ws_all, size_t ws_bytes) {
  if (nranks != 8 || n_in < 2 || ws_bytes < DATA_OFF + 3 * SLOT) return;
  Args a;
  a.me = (char*)d_ws_all[rank];
  a.x = (const float*)d_in[0]; a.w = (const float*)d_in[1]; a.out = (float*)d_out;
  a.rank = rank;
  for (int i = 0; i < 7; ++i) a.pad[i] = 0;
  const int gi = rank & 3, own = rank & 4, oth = own ^ 4;
  for (int r = 0; r < 4; ++r)
    for (int d = 0; d < 4; ++d) {
      a.ptr[r][d] = (char*)d_ws_all[d == 3 ? rank : own | ((rank + 1 + (d + r) % 3) & 3)] + DATA_OFF + (size_t)gi * FI * 65536;
      a.ptr[4 + r][d] = (char*)d_ws_all[oth | ((d + r) & 3)] + DATA_OFF + (size_t)(3 * FI + gi * FC) * 65536;
      a.ptr[8 + r][d] = (char*)d_out_all[oth | r];
    }
  int order[64], n = 0;
  for (int rb = 0; rb < FI; ++rb) order[n++] = gi * FI + rb;
  for (int d = 1; d < 4; ++d) for (int rb = 0; rb < 4; ++rb) order[n++] = ((gi + d) & 3) * FI + rb;
  for (int j = 0; j < 4; ++j) for (int i = 0; i < 2; ++i) order[n++] = 4 * FI + j * FC + i;
  for (int d = 1; d < 4; ++d) for (int rb = 4; rb < 8; ++rb) order[n++] = ((gi + d) & 3) * FI + rb;
  for (int j = 0; j < 4; ++j) for (int i = 2; i < FC; ++i) order[n++] = 4 * FI + j * FC + i;
  for (int d = 1; d < 4; ++d) for (int rb = 8; rb < FI; ++rb) order[n++] = ((gi + d) & 3) * FI + rb;
  for (int g = 0; g < 16; ++g) a.tab[g] = order[4 * g] | (order[4 * g + 1] << 8) | (order[4 * g + 2] << 16) | (order[4 * g + 3] << 24);
  (void)hipFuncSetAttribute((const void*)agk, hipFuncAttributeMaxDynamicSharedMemorySize, LDS_BYTES);
  agk<<<256, NT, LDS_BYTES, stream>>>(a);
}
```

```cpp
#include <hip/hip_runtime.h>
#include <rccl/rccl.h>
typedef unsigned long long u64;
typedef unsigned v4u __attribute__((ext_vector_type(4)));
typedef unsigned v2u __attribute__((ext_vector_type(2)));
typedef unsigned s8u __attribute__((ext_vector_type(8)));
typedef float v4f __attribute__((ext_vector_type(4)));
typedef float v2f __attribute__((ext_vector_type(2)));
typedef _Float16 v8h __attribute__((ext_vector_type(8)));
typedef _Float16 v2h __attribute__((ext_vector_type(2)));

constexpr int PS = 60;
constexpr int PD = 5550;
constexpr int DY = 1375;
constexpr int RPUB = 9;
constexpr int TG1 = 50, TG2 = 150, TG3 = 250, TX4 = 350, TI2 = 150, TI3 = 300;
constexpr int TCL = 100, TCW = 150, TCF = 360;
constexpr int ZDELAY = 300, ZSPLIT = 3584, YV = 50, TGA = 640;
constexpr int CATCH = 40;
constexpr int FI = 11, FC = 5, XA = 2, NEP = 10;
constexpr int KD = 2048, NS = 256, MS = 256;
constexpr size_t SLOT = 8u << 20;
constexpr size_t DATA_OFF = 1u << 20;
constexpr size_t B_OFF = 4u << 20;
constexpr size_t G_OFF = 6u << 20;
constexpr int ZERO_KB = 6144 + 9;
constexpr u64 DEADLINE = 20000000ull;
constexpr int NT = 512;
constexpr int LDS_BYTES = 131072 + 256;
struct Args {
  char* me; const float* x; const float* w; float* out;
  int rank; int pad[7];
  char* ptr[12][4];
  unsigned tab[16];
};
__device__ __forceinline__ char* gp(u64 v) { return (char*)(__attribute__((address_space(1))) char*)v; }
__device__ __forceinline__ u64 rt() { return __builtin_amdgcn_s_memrealtime(); }
__device__ __forceinline__ void st16_sys(void* p, v4u v) { asm volatile("global_store_dwordx4 %0, %1, off sc0 sc1\n\ts_nop 1" :: "v"(p), "v"(v) : "memory"); }
__device__ __forceinline__ void st16_so(u64 base, unsigned off, v4u v) { asm volatile("global_store_dwordx4 %0, %1, %2 sc0 sc1\n\ts_nop 1" :: "v"(off), "v"(v), "s"(base) : "memory"); }
__device__ __forceinline__ void st4_sys(void* p, unsigned v) { __hip_atomic_store((unsigned*)p, v, __ATOMIC_RELAXED, __HIP_MEMORY_SCOPE_SYSTEM); }
__device__ __forceinline__ unsigned ld_sys(const void* p) { return __hip_atomic_load((const unsigned*)p, __ATOMIC_RELAXED, __HIP_MEMORY_SCOPE_SYSTEM); }
__device__ __forceinline__ unsigned ld_dev(const void* p) { return __hip_atomic_load((const unsigned*)p, __ATOMIC_RELAXED, __HIP_MEMORY_SCOPE_AGENT); }
#define LDMOD " sc0 sc1"
#define L4(a, b, c, d, off, base) "global_load_dwordx4 %" #a ", %" #off ", %" #base LDMOD "\n\tglobal_load_dwordx4 %" #b ", %" #off ", %" #base " offset:1024" LDMOD \
  "\n\tglobal_load_dwordx4 %" #c ", %" #off ", %" #base " offset:2048" LDMOD "\n\tglobal_load_dwordx4 %" #d ", %" #off ", %" #base " offset:3072" LDMOD "\n\t"
__device__ __forceinline__ void ld16m(v4u* f, unsigned loff, const char* b0, const char* b1) {
  asm volatile("s_nop 4\n\t" L4(0, 1, 2, 3, 16, 17) L4(4, 5, 6, 7, 16, 18) L4(8, 9, 10, 11, 16, 19) L4(12, 13, 14, 15, 16, 20) "s_waitcnt vmcnt(0)"
               : "=&v"(f[0]), "=&v"(f[1]), "=&v"(f[2]), "=&v"(f[3]), "=&v"(f[4]), "=&v"(f[5]), "=&v"(f[6]), "=&v"(f[7]),
                 "=&v"(f[8]), "=&v"(f[9]), "=&v"(f[10]), "=&v"(f[11]), "=&v"(f[12]), "=&v"(f[13]), "=&v"(f[14]), "=&v"(f[15])
               : "v"(loff), "s"(b0), "s"(b0 + 4096), "s"(b1), "s"(b1 + 4096) : "memory");
}
__device__ __forceinline__ void ld8m(v4u* f, unsigned loff, const char* b0, const char* b1) {
  asm volatile("s_nop 4\n\t" L4(0, 1, 2, 3, 8, 9) L4(4, 5, 6, 7, 8, 10) "s_waitcnt vmcnt(0)"
               : "=&v"(f[0]), "=&v"(f[1]), "=&v"(f[2]), "=&v"(f[3]), "=&v"(f[4]), "=&v"(f[5]), "=&v"(f[6]), "=&v"(f[7])
               : "v"(loff), "s"(b0), "s"(b1) : "memory");
}
__device__ __forceinline__ unsigned min4(unsigned m, v4u f) { m = min(min(m, f.x), f.y); return min(min(m, f.z), f.w); }
template <int N> __device__ __forceinline__ bool all_ok(const v4u* f) {
  unsigned m = 0xffffffffu;
#pragma unroll
  for (int i = 0; i < N; ++i) m = min4(m, f[i]);
  return __all(m != 0);
}
__device__ __forceinline__ unsigned pkz(float a, float b) {
  v2h h; h.x = (_Float16)a; h.y = (_Float16)b; unsigned u = __builtin_bit_cast(unsigned, h); return u == 0 ? 0x8000u : u; }
template <int N> __device__ __forceinline__ bool wait_frags(v4u* f, unsigned loff, const char* b0, const char* b1, const char* hp, u64 t0) {
  for (;;) {
    int it = 0;
    while (!__all(ld_sys(hp) != 0) && ++it < 64) __builtin_amdgcn_s_sleep(1);
    if (N == 16) ld16m(f, loff, b0, b1); else ld8m(f, loff, b0, b0 + 4096);
    if (all_ok<N>(f)) return true;
    if (rt() - t0 > DEADLINE) return false;
  }
}
__device__ __forceinline__ int idx_row(int G, int idx) {
  if (idx < 4 * FI) return (G * 4 + idx / FI) * MS + (idx % FI) * 16;
  return ((G ^ 1) * 4 + (idx - 4 * FI) / FC) * MS + (FI + (idx - 4 * FI) % FC) * 16;
}
__device__ __forceinline__ int ks_of(int kp, int rb) {
  const int w6 = (kp * 43) >> 8, m = (kp - 48) >> 31;
  const int w = (w6 & m) | (kp & ~m), k = ((kp - 6 * w6) & m) | ((kp >> 3) & ~m);
  return ((w + 2 * rb) & 7) * 8 + (k & 7);
}
__device__ __forceinline__ int send_order(int u, int lane, bool intra) {
  const int f = u + 32 * lane, fl = intra ? 512 : 128;
  const int g3 = max(f - fl, 0) / 3;
  const int rb = f >= fl ? (intra ? 8 : 2) + (f - fl) - 3 * g3 : intra ? (f & 3) + ((f >> 8) << 2) : f & 1;
  const int kp = f >= fl ? g3 : intra ? f >> 2 : f >> 1;
  const int ks = intra && f >= fl ? ((kp & 7) << 3) | ((kp >> 3) & 7) : ks_of(kp & 63, rb);
  return ((intra ? rb : rb + FI) * 64 + ks) & 1023;
}
struct XL { v4f p0, p1; };
__device__ __forceinline__ XL x_load(const float* x, int e, int lane) {
  const float* src = x + (size_t)((e >> 6) * 16 + (lane >> 3)) * KD + (e & 63) * 32 + (lane & 7) * 4;
  XL r; r.p0 = *(const v4f*)src; r.p1 = *(const v4f*)(src + 8 * KD); return r;
}
__device__ __forceinline__ void x_stage(char* frag, const XL& r, int lane) {
  char* ldst = frag + ((lane >> 3) + 16 * ((lane & 7) >> 1)) * 16 + (lane & 1) * 8;
  v2u v0, v1; v0.x = pkz(r.p0.x, r.p0.y); v0.y = pkz(r.p0.z, r.p0.w); v1.x = pkz(r.p1.x, r.p1.y); v1.y = pkz(r.p1.z, r.p1.w);
  *(v2u*)ldst = v0; *(v2u*)(ldst + 128) = v1;
}
__device__ __forceinline__ void pace(u64 t0, unsigned when100) {
  const u64 until = t0 + when100 / 100;
  while (rt() < until) __builtin_amdgcn_s_sleep(1);
}
__device__ __forceinline__ int gate_all(const char* slot, u64 t0, int lane) {
  const unsigned e = ld_dev(slot + G_OFF + (lane & 31) * 128);
  if (!__all(e != 0)) return -1;
  int rel = min(max((int)(e - (unsigned)t0), 0), 100000);
#pragma unroll
  for (int m = 1; m < 32; m <<= 1) rel += __shfl_xor(rel, m);
  return __builtin_amdgcn_readfirstlane(rel) >> 5;
}
__device__ __forceinline__ int gate_read(const char* slot, u64 t0) {
  const unsigned e = __builtin_amdgcn_readfirstlane(ld_dev(slot + G_OFF + 33 * 128));
  return e ? min(max((int)(e - (unsigned)t0), 0), 100000) : -1;
}
__device__ __forceinline__ unsigned lds_ld(const unsigned* p) { return __hip_atomic_load(p, __ATOMIC_RELAXED, __HIP_MEMORY_SCOPE_WORKGROUP); }
__device__ __forceinline__ void lds_wait(const unsigned* p, unsigned n, u64 t0) {
  for (unsigned it = 1; lds_ld(p) < n; ++it) { __builtin_amdgcn_s_sleep(1); if ((it & 255) == 0 && rt() - t0 > DEADLINE) break; }
  asm volatile("" ::: "memory");
}
#define MFMA(b, a, c) __builtin_amdgcn_mfma_f32_16x16x32_f16(__builtin_bit_cast(v8h, b), __builtin_bit_cast(v8h, a), c, 0, 0, 0)
#define BIDX(c, k) (((c) >> 1) * 16 + ((k) & 1) * 8 + ((k) >> 1) * 2 + ((c) & 1))
template <int K0, int K1> __device__ __forceinline__ void mma(const v4u* B, const v4u* f, v4f* acc) {
#pragma unroll
  for (int k = K0; k < K1; ++k)
#pragma unroll
    for (int c = 0; c < 4; ++c) acc[c] = MFMA(B[BIDX(c, k)], f[k], (k == 0 ? (v4f){0, 0, 0, 0} : acc[c]));
}
#define PO(c) ((c) * 1024 + (((c) & 1) ? ((threadIdx.x & 1) ? -16 : 16) : 0))
__device__ __forceinline__ void put_part(char* p, v4f v, bool dead) {
  if (dead) v = (v4f){__builtin_nanf(""), __builtin_nanf(""), __builtin_nanf(""), __builtin_nanf("")};
  *(v4f*)p = v;
}
__device__ __forceinline__ bool task_head(const v4u* B, const v4u* f, char* sl, bool dead, bool& full) {
  if (!dead && !all_ok<6>(f)) return false;
  v4f acc[4];
  mma<0, 6>(B, f, acc);
  full = dead || all_ok<2>(f + 6);
  if (full) mma<6, 8>(B, f, acc);
#pragma unroll
  for (int c = 0; c < 4; ++c) put_part(sl + PO(c), acc[c], dead);
  return true;
}
__device__ __forceinline__ bool task_tail(const v4u* B, const v4u* f, v4f* acc, char* p, bool dead) {
  if (!dead && !all_ok<2>(f + 6)) return false;
  mma<6, 8>(B, f, acc);
#pragma unroll
  for (int i = 0; i < 4; ++i) put_part(p + PO(i), acc[i], dead);
  return true;
}
__device__ __forceinline__ void run_pair(const v4u* B, char* sl, const char* abase, unsigned tb, int lane, u64 t0, bool& dead) {
  const unsigned loff = lane * 16;
  const unsigned hb = ((tb >> (8 * (lane & 1))) & 63u) * 65536u;
  const char* a0 = abase + (size_t)(tb & 63) * 65536;
  const char* a1 = abase + (size_t)((tb >> 8) & 63) * 65536;
  unsigned head = 3, tail = 0, it = 0;
  while (head | tail) {
    if ((++it & 31) == 0 && rt() - t0 > DEADLINE) dead = true;
    unsigned c = tail;
    if (head) {
      const unsigned h = ld_sys(abase + hb + ((tail >> (lane & 1)) & 1 ? 8188u : 6140u));
      c = (dead ? 3u : (unsigned)__ballot(h != 0)) & (head | tail);
      if (!c) { __builtin_amdgcn_s_sleep(1); continue; }
    }
    if ((c & tail) == 3) {
      v4u f[8], e[8]; v4f acc0[4], acc1[4];
#pragma unroll
      for (int i = 0; i < 4; ++i) { acc0[i] = *(const v4f*)(sl + PO(i)); acc1[i] = *(const v4f*)(sl + 32768 + PO(i)); }
      asm volatile("s_nop 4\n\tglobal_load_dwordx4 %0, %4, %5 offset:3072" LDMOD "\n\tglobal_load_dwordx4 %1, %4, %5 offset:2048" LDMOD
                   "\n\tglobal_load_dwordx4 %2, %4, %6 offset:3072" LDMOD "\n\tglobal_load_dwordx4 %3, %4, %6 offset:2048" LDMOD "\n\ts_waitcnt vmcnt(0)"
                   : "=&v"(f[7]), "=&v"(f[6]), "=&v"(e[7]), "=&v"(e[6]) : "v"(loff), "s"(a0 + 4096), "s"(a1 + 4096) : "memory");
      if (task_tail(B, f, acc0, sl, dead)) tail &= ~1u;
      if (task_tail(B, e, acc1, sl + 32768, dead)) tail &= ~2u;
      continue;
    }
    if (c & tail) {
      const unsigned bit = c & tail;
      char* p = sl + (bit >> 1) * 32768;
      v4u f[8]; v4f acc[4];
#pragma unroll
      for (int i = 0; i < 4; ++i) acc[i] = *(const v4f*)(p + PO(i));
      asm volatile("s_nop 4\n\tglobal_load_dwordx4 %0, %2, %3 offset:3072" LDMOD "\n\tglobal_load_dwordx4 %1, %2, %3 offset:2048" LDMOD "\n\ts_waitcnt vmcnt(0)"
                   : "=&v"(f[7]), "=&v"(f[6]) : "v"(loff), "s"((bit == 1 ? a0 : a1) + 4096) : "memory");
      if (task_tail(B, f, acc, p, dead)) tail &= ~bit;
    }
    c &= head;
    if (!c) continue;
    v4u f[16]; bool full;
    if (c == 3) {
      ld16m(f, loff, a0, a1);
      if (task_head(B, f + 8, sl + 32768, dead, full)) { head &= ~2u; if (!full) tail |= 2u; }
    } else { const char* ap = c == 1 ? a0 : a1; ld8m(f, loff, ap, ap + 4096); }
    const unsigned bit = c == 2 ? 2u : 1u;
    if (task_head(B, f, sl + (bit >> 1) * 32768, dead, full)) { head &= ~bit; if (!full) tail |= bit; }
  }
}
template <int K> __device__ __forceinline__ void ld_step(v4u* f, unsigned loff, const char* a0, const char* a1) {
  if (K == 7) asm volatile("s_nop 4\n\tglobal_load_dwordx4 %0, %2, %3" LDMOD "\n\tglobal_load_dwordx4 %1, %2, %4" LDMOD "\n\ts_waitcnt vmcnt(0)"
                           : "=&v"(f[7]), "=&v"(f[15]) : "v"(loff), "s"(a0 + 7168), "s"(a1 + 7168) : "memory");
  else asm volatile("s_nop 4\n\tglobal_load_dwordx4 %0, %4, %5" LDMOD "\n\tglobal_load_dwordx4 %1, %4, %6" LDMOD
                    "\n\tglobal_load_dwordx4 %2, %4, %5 offset:1024" LDMOD "\n\tglobal_load_dwordx4 %3, %4, %6 offset:1024" LDMOD "\n\ts_waitcnt vmcnt(0)"
                    : "=&v"(f[K]), "=&v"(f[8 + K]), "=&v"(f[(K + 1) & 7]), "=&v"(f[8 + ((K + 1) & 7)]) : "v"(loff), "s"(a0 + K * 1024), "s"(a1 + K * 1024) : "memory");
}
template <int K> __device__ __forceinline__ void inc_step(const v4u* B, v4u* f, v4f* acc0, v4f* acc1, unsigned loff, const char* a0, const char* a1, u64 t0, bool& dead) {
  for (unsigned it = 0; !dead && !__all(min4(min4(0xffffffffu, f[K]), f[8 + K]) != 0); ++it) {
    if (K < 7 && it) __builtin_amdgcn_s_sleep(2);
    ld_step<K>(f, loff, a0, a1);
    if ((it & 31) == 31 && rt() - t0 > DEADLINE) dead = true;
  }
  mma<K, K + 1>(B, f, acc0); mma<K, K + 1>(B, f + 8, acc1);
}
__device__ __forceinline__ void run_inc(const v4u* B, char* sl, const char* abase, unsigned tb, int lane, u64 t0, bool& dead) {
  const unsigned loff = lane * 16;
  const char* a0 = abase + (size_t)(tb & 63) * 65536;
  const char* a1 = abase + (size_t)((tb >> 8) & 63) * 65536;
  for (unsigned it = 1; !dead && !__all(ld_sys((lane & 1 ? a1 : a0) + 1020) != 0); ++it) {
    __builtin_amdgcn_s_sleep(1);
    if ((it & 31) == 0 && rt() - t0 > DEADLINE) dead = true;
  }
  v4u f[16], g[8]; v4f acc0[4], acc1[4];
  ld8m(g, loff, a0, a1);
#pragma unroll
  for (int i = 0; i < 4; ++i) { f[i] = g[i]; f[8 + i] = g[4 + i]; f[4 + i] = f[12 + i] = (v4u){0, 0, 0, 0}; }
  inc_step<0>(B, f, acc0, acc1, loff, a0, a1, t0, dead); inc_step<1>(B, f, acc0, acc1, loff, a0, a1, t0, dead);
  inc_step<2>(B, f, acc0, acc1, loff, a0, a1, t0, dead); inc_step<3>(B, f, acc0, acc1, loff, a0, a1, t0, dead);
  inc_step<4>(B, f, acc0, acc1, loff, a0, a1, t0, dead); inc_step<5>(B, f, acc0, acc1, loff, a0, a1, t0, dead);
  inc_step<6>(B, f, acc0, acc1, loff, a0, a1, t0, dead); inc_step<7>(B, f, acc0, acc1, loff, a0, a1, t0, dead);
#pragma unroll
  for (int i = 0; i < 4; ++i) { put_part(sl + PO(i), acc0[i], dead); put_part(sl + 32768 + PO(i), acc1[i], dead); }
}
__device__ __forceinline__ void finish(const char* red, float* dst, const char* slot, u64 t0, int gate, int yslot) {
  if (gate == -1) gate = gate_read(slot, t0);
  __syncthreads();
  v4f s0 = *(const v4f*)red;
#pragma unroll
  for (int w = 1; w < 8; ++w) s0 += *(const v4f*)(red + w * 4096);
  v4u y0 = __builtin_bit_cast(v4u, s0);
#pragma unroll
  for (int i = 0; i < 4; ++i) if (y0[i] == 0) y0[i] = 0x80000000u;
  for (int it = 0; gate == -1 && it < 64; ++it) { gate = gate_read(slot, t0); if (gate < 0) __builtin_amdgcn_s_sleep(16); }
  if (gate >= 0) pace(t0, gate * 100 + yslot * DY);
  st16_sys(dst, y0);
}
__device__ __forceinline__ void verify(float* outp, const char* slot, char* me, int G, int tile, int lane, u64 t0, unsigned call) {
  const char* ysrc = (const char*)(outp + (size_t)idx_row(G ^ 1, tile >> 2) * NS + (tile & 3) * 64);
  const unsigned yo = (lane >> 3) * 1024 + (lane & 7) * 16;
  bool dead = false, ok = false;
  int gate;
  while ((gate = gate_read(slot, t0)) < 0) { __builtin_amdgcn_s_sleep(16); if (rt() - t0 > DEADLINE) { dead = true; break; } }
  if (!dead) pace(t0, (gate + YV) * 100);
  for (unsigned it = 1; !dead; ++it) {
    if (!ok) {
      v4u y[4];
      asm volatile("s_nop 4\n\tglobal_load_dwordx4 %0, %4, %5" LDMOD "\n\tglobal_load_dwordx4 %1, %4, %5 offset:128" LDMOD
                   "\n\tglobal_load_dwordx4 %2, %4, %6" LDMOD "\n\tglobal_load_dwordx4 %3, %4, %6 offset:128" LDMOD "\n\ts_waitcnt vmcnt(0)"
                   : "=&v"(y[0]), "=&v"(y[1]), "=&v"(y[2]), "=&v"(y[3]) : "v"(yo), "s"(ysrc), "s"(ysrc + 8192) : "memory");
      ok = min4(min4(min4(min4(0xffffffffu, y[0]), y[1]), y[2]), y[3]) != 0;
    }
    if (__all(ok)) break;
    __builtin_amdgcn_s_sleep(1);
    if ((it & 31) == 0 && rt() - t0 > DEADLINE) dead = true;
  }
  if (dead) {
    const v4u nan4 = {0x7fc00000u, 0x7fc00000u, 0x7fc00000u, 0x7fc00000u};
#pragma unroll
    for (int c = 0; c < 4; ++c) st16_sys((char*)ysrc + yo + (c & 1) * 128 + (c >> 1) * 8192, nan4);
    if (lane == 0) st4_sys(me + 4096, call);
  }
}

extern "C" __global__ void __launch_bounds__(NT) agk(Args a) {
  extern __shared__ __attribute__((aligned(16))) char lds[];
  const s8u ka = *(const volatile __attribute__((address_space(4))) s8u*)__builtin_amdgcn_kernarg_segment_ptr();
  const u64 t0 = rt();
  __builtin_amdgcn_sched_barrier(0);
  const int tid = threadIdx.x, lane = tid & 63, wave = __builtin_amdgcn_readfirstlane(tid >> 6), b = blockIdx.x;
  const bool intra = b < 32;
  const int u = b & 31;
  unsigned* lflag = (unsigned*)(lds + 131072);
  lflag[tid & 31] = 0;
  const int f0 = u + 32 * wave, kp0 = (f0 >> (intra ? 2 : 1)) & 63, rb0 = f0 & (intra ? 3 : 1);
  const int e0 = (rb0 + (intra ? 0 : FI)) * 64 + ks_of(kp0, rb0);
  const int xl0 = (lane & 15) * KD + (lane >> 4) * 8, xl1 = (lane >> 3) * KD + (lane & 7) * 4;
  unsigned xa = (unsigned)((e0 >> 6) * 16 * KD + (e0 & 63) * 32 + (tid < 64 ? xl0 : xl1)) * 4u, xb = xa + (tid < 64 ? 16u : 32u * KD);
  asm volatile("" : "+v"(xa), "+v"(xb));
  __builtin_amdgcn_sched_barrier(0);
  char* const me = gp(ka[0] | ((u64)ka[1] << 32));
  const float* const xp = (const float*)gp(ka[2] | ((u64)ka[3] << 32));
  const float* const wp = (const float*)gp(ka[4] | ((u64)ka[5] << 32));
  float* const outp = (float*)gp(ka[6] | ((u64)ka[7] << 32));
  XL r0;
  if (b < 64 && (intra || wave < XA)) { r0.p0 = *(const v4f*)((const char*)xp + xa); r0.p1 = *(const v4f*)((const char*)xp + xb); }
  unsigned* ctr = (unsigned*)(me + 65536) + b * 16 + wave;
  const int mb = (b - 64) >> 31;
  const unsigned rowo = 64 + 32 * (((((b >> 5) << 2) + (b & 3)) & mb) | ((8 + (b & 1) + ((b >> 2) & 2)) & ~mb));
#define BOOT() \
  unsigned cprev; s8u pr8; int rank; \
  asm volatile("s_load_dword %0, %3, 0x0 glc\n\ts_load_dwordx8 %1, %4, %5\n\ts_load_dword %2, %4, 0x20\n\ts_waitcnt lgkmcnt(0)" \
               : "=&s"(cprev), "=&s"(pr8), "=&s"(rank) : "s"(ctr), "s"(__builtin_amdgcn_kernarg_segment_ptr()), "s"(rowo) : "memory"); \
  const unsigned call = cprev + 1, so = (call % 3) * (unsigned)SLOT; \
  char* const slot = me + DATA_OFF + so; \
  const int G = rank >> 2, gi4 = rank & 3
  const unsigned loff = lane * 16;
  if (__builtin_expect(b < 64 && wave != 0, 0)) {
    auto wload = [&](v4f* q, int g, int hf) __attribute__((always_inline)) {
      const float* wsrc = wp + (size_t)(u * 64 + (g & 1) * 32 + (hf * 2 + (lane >> 5)) * 8) * NS + ((lane & 31) >> 3) * 64 + (g >> 1) * 32 + (lane & 7) * 4;
#pragma unroll
      for (int i = 0; i < 8; ++i) q[i] = *(const v4f*)(wsrc + i * NS);
    };
    auto wstore = [&](const v4f* q, int g, int hf) __attribute__((always_inline)) {
      const int t = g >> 1, kk = g & 1, kg = hf * 2 + (lane >> 5), j = (lane & 31) >> 3, c4 = lane & 7;
      char* dstl = lds + 32768 + (kk * 16 + j * 4 + t * 2 + (c4 >> 2)) * 1024 + (kg * 16 + (c4 & 3) * 4) * 16;
#pragma unroll
      for (int c = 0; c < 4; ++c) {
        v4u o; o.x = pkz(q[0][c], q[1][c]); o.y = pkz(q[2][c], q[3][c]); o.z = pkz(q[4][c], q[5][c]); o.w = pkz(q[6][c], q[7][c]);
        *(v4u*)(dstl + c * 16) = o;
      }
      asm volatile("s_waitcnt lgkmcnt(0)" ::: "memory");
      if (lane == 0) __hip_atomic_fetch_add(lflag + 24 + g, 1u, __ATOMIC_RELAXED, __HIP_MEMORY_SCOPE_WORKGROUP);
    };
    auto put = [&](int j, const XL& x) __attribute__((always_inline)) {
      x_stage(lds + j * 1024, x, lane);
      asm volatile("s_waitcnt lgkmcnt(0)" ::: "memory");
      if (lane == 0) __hip_atomic_store(lflag + j, 1u, __ATOMIC_RELAXED, __HIP_MEMORY_SCOPE_WORKGROUP);
    };
    v4f wq0[8];
    if (!intra && (wave >> 1) == 1) wload(wq0, 0, wave & 1);
    BOOT();
    asm volatile("s_barrier" ::: "memory");
    if (intra || wave < XA) { asm volatile("" : "+v"(r0.p0), "+v"(r0.p1)); put(wave, r0); }
#pragma unroll 1
    for (int tk = wave - 2; tk < 8 && !intra && wave >= 2; tk += 6) {
      if (tk >= 2) { pace(t0, (tk < 4 ? TG1 : tk < 6 ? TG2 : TG3) * 100); wload(wq0, tk >> 1, tk & 1); }
      wstore(wq0, tk >> 1, tk & 1);
    }
    int lane2 = lane; asm volatile("" : "+v"(lane2));
    const int etab = send_order(u, lane2, intra);
    if (lane == 0) st4_sys(ctr, call);
    const v4u zero = {0, 0, 0, 0};
    if (wave == 1) {
      unsigned e = 0;
      for (int it = 0; it < 8 && e == 0; ++it) e = ld_dev(slot + G_OFF + 32 * 128);
      const int d = (int)((unsigned)t0 - e);
      if (e != 0 && d > 0 && d < 200 && lane == 0) __hip_atomic_store(lflag + 28, (unsigned)d, __ATOMIC_RELAXED, __HIP_MEMORY_SCOPE_WORKGROUP);
    }
    if (intra) {
      char* ldst = slot + (size_t)(gi4 * FI * 64) * 1024 + loff;
#pragma unroll 1
      for (int i = 1; i < 4; ++i) {
        const int j = wave + 7 * i;
        XL r;
        if (i < 3) { pace(t0, (i == 1 ? TI2 : TI3) * 100); r = x_load(xp, __builtin_amdgcn_readlane(etab, j), lane); }
        st16_sys(ldst + (size_t)__builtin_amdgcn_readlane(etab, j - 7) * 1024, *(const v4u*)(lds + (j - 7) * 1024 + loff));
        if (i < 3) put(j, r);
      }
    } else if (wave == 1) {
      pace(t0, TCF * 100);
      while (!__all(ld_sys(slot + G_OFF + (40 + u) * 128 + (lane & 31) * 4) != 0) && rt() - t0 < DEADLINE) __builtin_amdgcn_s_sleep(4);
      if (lane == 0) __hip_atomic_store(lflag + 29, 1u, __ATOMIC_RELAXED, __HIP_MEMORY_SCOPE_WORKGROUP);
      if (u == 7) {
        pace(t0, TGA * 100);
        int gt;
        while ((gt = gate_all(slot, t0, lane)) < 0 && rt() - t0 < DEADLINE) __builtin_amdgcn_s_sleep(8);
        if (gt >= 0 && lane == 0) st4_sys(slot + G_OFF + 33 * 128, ((unsigned)t0 + gt) | 1u);
      }
    } else if (wave >= 4) {
      const int j = 2 * wave - 6;
      pace(t0, (TX4 + (wave >= 6 ? 60 : 0)) * 100);
      XL r[2];
#pragma unroll
      for (int i = 0; i < 2; ++i) r[i] = x_load(xp, __builtin_amdgcn_readlane(etab, j + i), lane);
#pragma unroll
      for (int i = 0; i < 2; ++i) put(j + i, r[i]);
    }
    char* z = me + DATA_OFF + (size_t)((call + 2) % 3) * SLOT + loff;
    pace(t0, ZDELAY * 100);
    for (int k = u + 32 * (wave - 1) + (intra ? 0 : ZSPLIT); k < (intra ? ZSPLIT : ZERO_KB); k += 224) {
      st16_sys(z + (size_t)k * 1024, zero); asm volatile("s_waitcnt vmcnt(1)" ::: "memory"); __builtin_amdgcn_s_sleep(2); }
    if (!intra) verify(outp, slot, me, G, wave * 32 + u, lane, t0, call);
    return;
  }
  BOOT();
  const u64 q0 = pr8[0] | ((u64)pr8[1] << 32), q1 = pr8[2] | ((u64)pr8[3] << 32), q2 = pr8[4] | ((u64)pr8[5] << 32), q3 = pr8[6] | ((u64)pr8[7] << 32);
  if (__builtin_expect(b < 64, 1)) {
    {
      unsigned o = so + e0 * 1024 + loff;
      asm volatile("" : "+v"(o));
      asm volatile("" : "+v"(r0.p0), "+v"(r0.p1));
      v4u v; v.x = pkz(r0.p0.x, r0.p0.y); v.y = pkz(r0.p0.z, r0.p0.w); v.z = pkz(r0.p1.x, r0.p1.y); v.w = pkz(r0.p1.z, r0.p1.w);
      st16_so(q0, o, v); st16_so(q1, o, v); st16_so(q2, o, v); st16_so(q3, o, v);
    }
    asm volatile("s_barrier" ::: "memory");
    const int etab = send_order(u, lane, intra);
    unsigned late = 0, when = PS * 100 + PD + u * (PD / 32);
    int extra = 0;
    if (!intra) {
#pragma unroll 1
      for (int r = 1; r < 2 * FC + 8; ++r) {
        v4u vv[4]; unsigned o;
        if (r >= XA && r < XA + 8) {
          const int n = r - XA, t = n >> 2, kk = (n >> 1) & 1, cb = n & 1;
          if (cb == 0) lds_wait(lflag + 24 + (n >> 1), 2, t0);
#pragma unroll
          for (int d = 0; d < 4; ++d) vv[d] = *(const v4u*)(lds + 32768 + (kk * 16 + t * 2 + cb) * 1024 + ((d + b) & 3) * 4096 + loff);
          o = (unsigned)B_OFF + (8 + gi4 * 2 + t) * 131072 + (kk * 64 + u * 2 + cb) * 1024 - (3 * FI + gi4 * FC) * 65536;
        } else {
          const int j = r < XA ? r : r - 8;
          lds_wait(lflag + j, 1, t0);
          vv[0] = vv[1] = vv[2] = vv[3] = *(const v4u*)(lds + j * 1024 + loff);
          o = __builtin_amdgcn_readlane(etab, j) * 1024;
        }
        o += so + loff;
        if (r == RPUB) lds_wait(lflag + 29, 1, t0);
        if (!late) late = __builtin_amdgcn_readfirstlane(lds_ld(lflag + 28));
        pace(t0 - late + extra, when);
        asm volatile("s_waitcnt vmcnt(8)" ::: "memory");
        if (r == 1) extra = min(max((int)(rt() - t0) - (int)(when / 100) - CATCH, 0), 1000);
        when += PD;
        if (r == RPUB) {
          const unsigned sched = (unsigned)(t0 - late) + PS + (unsigned)((2 * FC + 8) * PD) / 100;
          const unsigned pred = (unsigned)rt() + (unsigned)((2 * FC + 8 - RPUB) * PD - u * (PD / 32)) / 100;
          if (lane == 0) st4_sys(slot + G_OFF + u * 128, ((int)(pred - sched) > 0 ? pred : sched) | 1u);
        }
        st16_so(q0, o, vv[0]); st16_so(q1, o, vv[1]); st16_so(q2, o, vv[2]); st16_so(q3, o, vv[3]);
      }
    } else {
#pragma unroll 1
      for (int j = 1; j < 2 * FI; ++j) {
        lds_wait(lflag + j, 1, t0);
        const v4u v = *(const v4u*)(lds + j * 1024 + loff);
        const unsigned o = __builtin_amdgcn_readlane(etab, j) * 1024 + so + loff;
        if (!late) late = __builtin_amdgcn_readfirstlane(lds_ld(lflag + 28));
        pace(t0 - late + extra, when);
        asm volatile("s_waitcnt vmcnt(6)" ::: "memory");
        if (j == 1) extra = min(max((int)(rt() - t0) - (int)(when / 100) - CATCH, 0), 1000);
        when += PD;
        st16_so(q0, o, v); st16_so(q1, o, v); st16_so(q2, o, v);
      }
    }
    if (lane == 0) st4_sys(ctr, call);
    if (!intra) verify(outp, slot, me, G, u, lane, t0, call);
    return;
  }
  const int X = b & 7, ci = (b - 64) >> 3;
  const bool local = ci < 8;
  v2f wq[8];
  if (local) {
    const float* wsrc = wp + (size_t)((b - 64) * 32 + (wave & 3) * 8) * NS + (wave >> 2) * 128 + lane * 2;
#pragma unroll
    for (int j = 0; j < 8; ++j) wq[j] = *(const v2f*)(wsrc + j * NS);
  }
  const int pr = ((local ? ci : ci - 8) >> 1) * 4 + (X >> 1);
  const int c64 = (local ? 0 : 4) + (X & 1) + 2 * (ci & 1);
  const int half = pr & 1, g = pr >> 1;
  const int yslot = (pr & ~3) + ((pr + (c64 & 2)) & 3);
  if (b == 129 && tid == 0) st4_sys(slot + G_OFF + 32 * 128, (unsigned)t0 | 1u);
  if (lane == 0) st4_sys(ctr, call);
  bool dead = false;
  const char* abase = slot + wave * 8192;
  const char* bsrc = slot + B_OFF + (size_t)c64 * 262144 + wave * 8192;
  char* part = lds + wave * 4096 + (lane >> 4) * 256 + ((lane & 15) ^ ((lane >> 4) * 2)) * 16;
  const int orow = (wave & 3) * 4 + (lane >> 4);
  const char* red = lds + (wave >> 2) * 32768 + ((lane >> 2) & 3) * 1024 + (lane & 3) * 256 + (orow ^ ((lane & 3) * 2 + ((lane >> 2) & 1))) * 16;
  const unsigned tb0 = (a.tab[g] >> (16 * half)) & 0xffffu;
  const size_t ooff = (size_t)orow * NS + (lane & 15) * 4;
  float* yd = (float*)gp(q0) + (size_t)idx_row(G, (tb0 >> (8 * (wave >> 2))) & 63) * NS + gi4 * 64 + ooff;
  if (local) {
    const int col = (wave >> 2) * 128 + lane * 2;
    char* dstl = lds + 65536 + (col >> 4) * 1024 + ((wave & 3) * 16 + (col & 15)) * 16;
#pragma unroll
    for (int c = 0; c < 2; ++c) {
      v4u o; o.x = pkz(wq[0][c], wq[1][c]); o.y = pkz(wq[2][c], wq[3][c]); o.z = pkz(wq[4][c], wq[5][c]); o.w = pkz(wq[6][c], wq[7][c]);
      *(v4u*)(dstl + c * 16) = o;
    }
    __syncthreads();
    const v4u f0v = *(const v4u*)(lds + 65536 + wave * 2048 + loff), f1v = *(const v4u*)(lds + 65536 + wave * 2048 + 1024 + loff);
    char* bl = slot + B_OFF + (size_t)wave * 131072 + (b & 1) * 65536 + ((b - 64) >> 1) * 2048 + loff;
    st16_sys(bl, f0v); st16_sys(bl + 1024, f1v);
  }
  if (!local) {
    const int n = (b - 128) * 8 + wave;
    pace(t0, TCL * 100 + (wave * 128 + b - 128) * (TCW * 100 / 1024));
    st16_sys(outp + (size_t)(idx_row(G ^ 1, n >> 4) + (n & 15)) * NS + lane * 4, (v4u){0, 0, 0, 0});
    asm volatile("s_waitcnt vmcnt(0)" ::: "memory");
    if (lane == 0) st4_sys(slot + G_OFF + (40 + (n >> 5)) * 128 + (n & 31) * 4, 1u);
  }
  if (tid == 448) (void)ld_sys(local ? outp + b * 2048 : yd);
  if (!local && pr < NEP) {
    v4u A[16], B[8];
    const char* a0 = abase + (size_t)(tb0 & 63) * 65536;
    const char* a1 = abase + (size_t)((tb0 >> 8) & 63) * 65536;
    if (!wait_frags<16>(A, loff, a0, a1, ((lane & 1) ? a1 : a0) + 8188, t0)) dead = true;
#pragma unroll 1
    for (int t = 0; t < 2; ++t) {
      v4f acc[4];
#pragma unroll
      for (int c = 0; c < 4; ++c) acc[c] = (v4f){0, 0, 0, 0};
#pragma unroll
      for (int kk = 0; kk < 2; ++kk) {
        const char* bs = bsrc + (t * 2 + kk) * 65536;
        if (!wait_frags<8>(B, loff, bs, bs, bs + (lane & 3) * 2048 + 2044, t0)) dead = true;
#pragma unroll
        for (int i = 0; i < 4; ++i)
#pragma unroll
          for (int c = 0; c < 4; ++c) acc[c] = MFMA(B[i * 2 + (c & 1)], A[(c >> 1) * 8 + i * 2 + kk], acc[c]);
      }
#pragma unroll
      for (int c = 0; c < 4; ++c) put_part(part + (c >> 1) * 32768 + t * 2048 + PO(c & 1), acc[c], dead);
    }
    finish(red, yd, slot, t0, -1, yslot);
  } else {
    v4u B[32];
    if (local) {
#pragma unroll
      for (int q = 0; q < 4; q += 2) {
        const char* bs = bsrc + q * 65536;
        if (!wait_frags<16>(B + 8 * q, loff, bs, bs + 65536, bs + (lane & 3) * 2048 + (lane & 4) * 16384 + 2044, t0)) dead = true;
      }
    } else {
#pragma unroll
      for (int q = 0; q < 4; ++q) {
        const char* bs = bsrc + q * 65536;
        if (!wait_frags<8>(B + 8 * q, loff, bs, bs, bs + (lane & 3) * 2048 + 2044, t0)) dead = true;
      }
    }
#pragma unroll 1
    for (int pass = local ? 0 : 1; pass < 2; ++pass) {
      const unsigned tb = local && pass ? (a.tab[g + 8] >> (16 * half)) & 0xffffu : tb0;
      const int gate = local ? -2 : gate_read(slot, t0);
      if (pass && (local ? pr + 16 : pr) >= 27) run_inc(B, part + pass * 65536, abase, tb, lane, t0, dead);
      else run_pair(B, part + pass * 65536, abase, tb, lane, t0, dead);
      float* dst = local ? outp + (size_t)idx_row(G, (tb >> (8 * (wave >> 2))) & 63) * NS + c64 * 64 + ooff : yd;
      finish(red + pass * 65536, dst, slot, t0, gate, yslot);
    }
  }
  if (dead && lane == 0) st4_sys(me + 4096, call);
}

extern "C" void kernel_launch(int rank, int nranks, ncclComm_t comm, hipStream_t stream, void* const* d_in, const int* in_sizes, int n_in,
                              void* d_out, int out_size, const int* in_dtypes, int out_dtype, void* const* d_out_all, void* const* d_ws_all, size_t ws_bytes) {
  if (nranks != 8 || n_in < 2 || ws_bytes < DATA_OFF + 3 * SLOT) return;
  Args a;
  a.me = (char*)d_ws_all[rank];
  a.x = (const float*)d_in[0]; a.w = (const float*)d_in[1]; a.out = (float*)d_out;
  a.rank = rank;
  for (int i = 0; i < 7; ++i) a.pad[i] = 0;
  const int gi = rank & 3, own = rank & 4, oth = own ^ 4;
  for (int r = 0; r < 4; ++r)
    for (int d = 0; d < 4; ++d) {
      a.ptr[r][d] = (char*)d_ws_all[d == 3 ? rank : own | ((rank + 1 + (d + r) % 3) & 3)] + DATA_OFF + (size_t)gi * FI * 65536;
      a.ptr[4 + r][d] = (char*)d_ws_all[oth | ((d + r) & 3)] + DATA_OFF + (size_t)(3 * FI + gi * FC) * 65536;
      a.ptr[8 + r][d] = (char*)d_out_all[oth | r];
    }
  int order[64], n = 0;
  for (int rb = 0; rb < FI; ++rb) order[n++] = gi * FI + rb;
  for (int d = 1; d < 4; ++d) for (int rb = 0; rb < 4; ++rb) order[n++] = ((gi + d) & 3) * FI + rb;
  for (int j = 0; j < 4; ++j) for (int i = 0; i < 2; ++i) order[n++] = 4 * FI + j * FC + i;
  for (int d = 1; d < 4; ++d) for (int rb = 4; rb < 8; ++rb) order[n++] = ((gi + d) & 3) * FI + rb;
  for (int j = 0; j < 4; ++j) for (int i = 2; i < FC; ++i) order[n++] = 4 * FI + j * FC + i;
  for (int d = 1; d < 4; ++d) for (int rb = 8; rb < FI; ++rb) order[n++] = ((gi + d) & 3) * FI + rb;
  for (int g = 0; g < 16; ++g) a.tab[g] = order[4 * g] | (order[4 * g + 1] << 8) | (order[4 * g + 2] << 16) | (order[4 * g + 3] << 24);
  (void)hipFuncSetAttribute((const void*)agk, hipFuncAttributeMaxDynamicSharedMemorySize, LDS_BYTES);
  agk<<<256, NT, LDS_BYTES, stream>>>(a);
}
```

```cpp
#include <hip/hip_runtime.h>
#include <rccl/rccl.h>
typedef unsigned long long u64;
typedef unsigned v4u __attribute__((ext_vector_type(4)));
typedef unsigned v2u __attribute__((ext_vector_type(2)));
typedef unsigned s8u __attribute__((ext_vector_type(8)));
typedef float v4f __attribute__((ext_vector_type(4)));
typedef float v2f __attribute__((ext_vector_type(2)));
typedef _Float16 v8h __attribute__((ext_vector_type(8)));
typedef _Float16 v2h __attribute__((ext_vector_type(2)));

constexpr int PS = 60;
constexpr int PDI = 5500, PDW = 5600;
constexpr int DY = 1375;
constexpr int RPUB = 9;
constexpr int TG1 = 50, TG2 = 150, TG3 = 250, TX4 = 350, TI2 = 150, TI3 = 300;
constexpr int TCL = 100, TCW = 150, TCF = 360;
constexpr int ZDELAY = 300, ZSPLIT = 3584, YV = 50, TGA = 640;
constexpr int CATCH = 40;
constexpr int FI = 11, FC = 5, XA = 2, NEP = 10;
constexpr int KD = 2048, NS = 256, MS = 256;
constexpr size_t SLOT = 8u << 20;
constexpr size_t DATA_OFF = 1u << 20;
constexpr size_t B_OFF = 4u << 20;
constexpr size_t G_OFF = 6u << 20;
constexpr int ZERO_KB = 6144 + 9;
constexpr u64 DEADLINE = 100000000ull;
constexpr int NT = 512;
constexpr int LDS_BYTES = 131072 + 256;
struct Args {
  char* me; const float* x; const float* w; float* out;
  int rank; int pad[7];
  char* ptr[12][4];
  unsigned tab[16];
};
__device__ __forceinline__ char* gp(u64 v) { return (char*)(__attribute__((address_space(1))) char*)v; }
__device__ __forceinline__ u64 rt() { return __builtin_amdgcn_s_memrealtime(); }
__device__ __forceinline__ void st16_sys(void* p, v4u v) { asm volatile("global_store_dwordx4 %0, %1, off sc0 sc1\n\ts_nop 1" :: "v"(p), "v"(v) : "memory"); }
__device__ __forceinline__ void st16_so(u64 base, unsigned off, v4u v) { asm volatile("global_store_dwordx4 %0, %1, %2 sc0 sc1\n\ts_nop 1" :: "v"(off), "v"(v), "s"(base) : "memory"); }
__device__ __forceinline__ void st4_sys(void* p, unsigned v) { __hip_atomic_store((unsigned*)p, v, __ATOMIC_RELAXED, __HIP_MEMORY_SCOPE_SYSTEM); }
__device__ __forceinline__ unsigned ld_sys(const void* p) { return __hip_atomic_load((const unsigned*)p, __ATOMIC_RELAXED, __HIP_MEMORY_SCOPE_SYSTEM); }
__device__ __forceinline__ unsigned ld_dev(const void* p) { return __hip_atomic_load((const unsigned*)p, __ATOMIC_RELAXED, __HIP_MEMORY_SCOPE_AGENT); }
#define LDMOD " sc0 sc1"
#define L4(a, b, c, d, off, base) "global_load_dwordx4 %" #a ", %" #off ", %" #base LDMOD "\n\tglobal_load_dwordx4 %" #b ", %" #off ", %" #base " offset:1024" LDMOD \
  "\n\tglobal_load_dwordx4 %" #c ", %" #off ", %" #base " offset:2048" LDMOD "\n\tglobal_load_dwordx4 %" #d ", %" #off ", %" #base " offset:3072" LDMOD "\n\t"
__device__ __forceinline__ void ld16m(v4u* f, unsigned loff, const char* b0, const char* b1) {
  asm volatile("s_nop 4\n\t" L4(0, 1, 2, 3, 16, 17) L4(4, 5, 6, 7, 16, 18) L4(8, 9, 10, 11, 16, 19) L4(12, 13, 14, 15, 16, 20) "s_waitcnt vmcnt(0)"
               : "=&v"(f[0]), "=&v"(f[1]), "=&v"(f[2]), "=&v"(f[3]), "=&v"(f[4]), "=&v"(f[5]), "=&v"(f[6]), "=&v"(f[7]),
                 "=&v"(f[8]), "=&v"(f[9]), "=&v"(f[10]), "=&v"(f[11]), "=&v"(f[12]), "=&v"(f[13]), "=&v"(f[14]), "=&v"(f[15])
               : "v"(loff), "s"(b0), "s"(b0 + 4096), "s"(b1), "s"(b1 + 4096) : "memory");
}
__device__ __forceinline__ void ld8m(v4u* f, unsigned loff, const char* b0, const char* b1) {
  asm volatile("s_nop 4\n\t" L4(0, 1, 2, 3, 8, 9) L4(4, 5, 6, 7, 8, 10) "s_waitcnt vmcnt(0)"
               : "=&v"(f[0]), "=&v"(f[1]), "=&v"(f[2]), "=&v"(f[3]), "=&v"(f[4]), "=&v"(f[5]), "=&v"(f[6]), "=&v"(f[7])
               : "v"(loff), "s"(b0), "s"(b1) : "memory");
}
__device__ __forceinline__ unsigned min4(unsigned m, v4u f) { m = min(min(m, f.x), f.y); return min(min(m, f.z), f.w); }
template <int N> __device__ __forceinline__ bool all_ok(const v4u* f) {
  unsigned m = 0xffffffffu;
#pragma unroll
  for (int i = 0; i < N; ++i) m = min4(m, f[i]);
  return __all(m != 0);
}
__device__ __forceinline__ unsigned pkz(float a, float b) {
  v2h h; h.x = (_Float16)a; h.y = (_Float16)b; unsigned u = __builtin_bit_cast(unsigned, h); return u == 0 ? 0x8000u : u; }
template <int N> __device__ __forceinline__ bool wait_frags(v4u* f, unsigned loff, const char* b0, const char* b1, const char* hp, u64 t0) {
  for (;;) {
    int it = 0;
    while (!__all(ld_sys(hp) != 0) && ++it < 64) __builtin_amdgcn_s_sleep(1);
    if (N == 16) ld16m(f, loff, b0, b1); else ld8m(f, loff, b0, b0 + 4096);
    if (all_ok<N>(f)) return true;
    if (rt() - t0 > DEADLINE) return false;
  }
}
__device__ __forceinline__ int idx_row(int G, int idx) {
  if (idx < 4 * FI) return (G * 4 + idx / FI) * MS + (idx % FI) * 16;
  return ((G ^ 1) * 4 + (idx - 4 * FI) / FC) * MS + (FI + (idx - 4 * FI) % FC) * 16;
}
__device__ __forceinline__ int ks_of(int kp, int rb) {
  const int w6 = (kp * 43) >> 8, m = (kp - 48) >> 31;
  const int w = (w6 & m) | (kp & ~m), k = ((kp - 6 * w6) & m) | ((kp >> 3) & ~m);
  return ((w + 2 * rb) & 7) * 8 + (k & 7);
}
__device__ __forceinline__ int send_order(int u, int lane, bool intra) {
  const int f = u + 32 * lane, fl = intra ? 512 : 128;
  const int g3 = max(f - fl, 0) / 3;
  const int rb = f >= fl ? (intra ? 8 : 2) + (f - fl) - 3 * g3 : intra ? (f & 3) + ((f >> 8) << 2) : f & 1;
  const int kp = f >= fl ? g3 : intra ? f >> 2 : f >> 1;
  const int ks = intra && f >= fl ? ((kp & 7) << 3) | ((kp >> 3) & 7) : ks_of(kp & 63, rb);
  return ((intra ? rb : rb + FI) * 64 + ks) & 1023;
}
struct XL { v4f p0, p1; };
__device__ __forceinline__ XL x_load(const float* x, int e, int lane) {
  const float* src = x + (size_t)((e >> 6) * 16 + (lane >> 3)) * KD + (e & 63) * 32 + (lane & 7) * 4;
  XL r; r.p0 = *(const v4f*)src; r.p1 = *(const v4f*)(src + 8 * KD); return r;
}
__device__ __forceinline__ void x_stage(char* frag, const XL& r, int lane) {
  char* ldst = frag + ((lane >> 3) + 16 * ((lane & 7) >> 1)) * 16 + (lane & 1) * 8;
  v2u v0, v1; v0.x = pkz(r.p0.x, r.p0.y); v0.y = pkz(r.p0.z, r.p0.w); v1.x = pkz(r.p1.x, r.p1.y); v1.y = pkz(r.p1.z, r.p1.w);
  *(v2u*)ldst = v0; *(v2u*)(ldst + 128) = v1;
}
__device__ __forceinline__ void pace(u64 t0, unsigned when100) {
  const u64 until = t0 + when100 / 100;
  while (rt() < until) __builtin_amdgcn_s_sleep(1);
}
__device__ __forceinline__ int gate_all(const char* slot, u64 t0, int lane) {
  const unsigned e = ld_dev(slot + G_OFF + (lane & 31) * 128);
  if (!__all(e != 0)) return -1;
  int rel = min(max((int)(e - (unsigned)t0), 0), 100000);
#pragma unroll
  for (int m = 1; m < 32; m <<= 1) rel += __shfl_xor(rel, m);
  return __builtin_amdgcn_readfirstlane(rel) >> 5;
}
__device__ __forceinline__ int gate_read(const char* slot, u64 t0) {
  const unsigned e = __builtin_amdgcn_readfirstlane(ld_dev(slot + G_OFF + 33 * 128));
  return e ? min(max((int)(e - (unsigned)t0), 0), 100000) : -1;
}
__device__ __forceinline__ unsigned lds_ld(const unsigned* p) { return __hip_atomic_load(p, __ATOMIC_RELAXED, __HIP_MEMORY_SCOPE_WORKGROUP); }
__device__ __forceinline__ bool lds_wait(const unsigned* p, unsigned n, u64 t0) {
  for (unsigned it = 1; lds_ld(p) < n; ++it) { __builtin_amdgcn_s_sleep(1); if ((it & 255) == 0 && rt() - t0 > DEADLINE) return false; }
  asm volatile("" ::: "memory");
  return true;
}
#define MFMA(b, a, c) __builtin_amdgcn_mfma_f32_16x16x32_f16(__builtin_bit_cast(v8h, b), __builtin_bit_cast(v8h, a), c, 0, 0, 0)
#define BIDX(c, k) (((c) >> 1) * 16 + ((k) & 1) * 8 + ((k) >> 1) * 2 + ((c) & 1))
template <int K0, int K1> __device__ __forceinline__ void mma(const v4u* B, const v4u* f, v4f* acc) {
#pragma unroll
  for (int k = K0; k < K1; ++k)
#pragma unroll
    for (int c = 0; c < 4; ++c) acc[c] = MFMA(B[BIDX(c, k)], f[k], (k == 0 ? (v4f){0, 0, 0, 0} : acc[c]));
}
#define PO(c) ((c) * 1024 + (((c) & 1) ? ((threadIdx.x & 1) ? -16 : 16) : 0))
__device__ __forceinline__ void put_part(char* p, v4f v, bool dead) {
  if (dead) v = (v4f){__builtin_nanf(""), __builtin_nanf(""), __builtin_nanf(""), __builtin_nanf("")};
  *(v4f*)p = v;
}
__device__ __forceinline__ bool task_head(const v4u* B, const v4u* f, char* sl, bool dead, bool& full) {
  if (!dead && !all_ok<6>(f)) return false;
  v4f acc[4];
  mma<0, 6>(B, f, acc);
  full = dead || all_ok<2>(f + 6);
  if (full) mma<6, 8>(B, f, acc);
#pragma unroll
  for (int c = 0; c < 4; ++c) put_part(sl + PO(c), acc[c], dead);
  return true;
}
__device__ __forceinline__ bool task_tail(const v4u* B, const v4u* f, v4f* acc, char* p, bool dead) {
  if (!dead && !all_ok<2>(f + 6)) return false;
  mma<6, 8>(B, f, acc);
#pragma unroll
  for (int i = 0; i < 4; ++i) put_part(p + PO(i), acc[i], dead);
  return true;
}
__device__ __forceinline__ void run_pair(const v4u* B, char* sl, const char* abase, unsigned tb, int lane, u64 t0, bool& dead) {
  const unsigned loff = lane * 16;
  const unsigned hb = ((tb >> (8 * (lane & 1))) & 63u) * 65536u;
  const char* a0 = abase + (size_t)(tb & 63) * 65536;
  const char* a1 = abase + (size_t)((tb >> 8) & 63) * 65536;
  unsigned head = 3, tail = 0, it = 0;
  while (head | tail) {
    if ((++it & 31) == 0 && rt() - t0 > DEADLINE) dead = true;
    unsigned c = tail;
    if (head) {
      const unsigned h = ld_sys(abase + hb + ((tail >> (lane & 1)) & 1 ? 8188u : 6140u));
      c = (dead ? 3u : (unsigned)__ballot(h != 0)) & (head | tail);
      if (!c) { __builtin_amdgcn_s_sleep(1); continue; }
    }
    if ((c & tail) == 3) {
      v4u f[8], e[8]; v4f acc0[4], acc1[4];
#pragma unroll
      for (int i = 0; i < 4; ++i) { acc0[i] = *(const v4f*)(sl + PO(i)); acc1[i] = *(const v4f*)(sl + 32768 + PO(i)); }
      asm volatile("s_nop 4\n\tglobal_load_dwordx4 %0, %4, %5 offset:3072" LDMOD "\n\tglobal_load_dwordx4 %1, %4, %5 offset:2048" LDMOD
                   "\n\tglobal_load_dwordx4 %2, %4, %6 offset:3072" LDMOD "\n\tglobal_load_dwordx4 %3, %4, %6 offset:2048" LDMOD "\n\ts_waitcnt vmcnt(0)"
                   : "=&v"(f[7]), "=&v"(f[6]), "=&v"(e[7]), "=&v"(e[6]) : "v"(loff), "s"(a0 + 4096), "s"(a1 + 4096) : "memory");
      if (task_tail(B, f, acc0, sl, dead)) tail &= ~1u;
      if (task_tail(B, e, acc1, sl + 32768, dead)) tail &= ~2u;
      continue;
    }
    if (c & tail) {
      const unsigned bit = c & tail;
      char* p = sl + (bit >> 1) * 32768;
      v4u f[8]; v4f acc[4];
#pragma unroll
      for (int i = 0; i < 4; ++i) acc[i] = *(const v4f*)(p + PO(i));
      asm volatile("s_nop 4\n\tglobal_load_dwordx4 %0, %2, %3 offset:3072" LDMOD "\n\tglobal_load_dwordx4 %1, %2, %3 offset:2048" LDMOD "\n\ts_waitcnt vmcnt(0)"
                   : "=&v"(f[7]), "=&v"(f[6]) : "v"(loff), "s"((bit == 1 ? a0 : a1) + 4096) : "memory");
      if (task_tail(B, f, acc, p, dead)) tail &= ~bit;
    }
    c &= head;
    if (!c) continue;
    v4u f[16]; bool full;
    if (c == 3) {
      ld16m(f, loff, a0, a1);
      if (task_head(B, f + 8, sl + 32768, dead, full)) { head &= ~2u; if (!full) tail |= 2u; }
    } else { const char* ap = c == 1 ? a0 : a1; ld8m(f, loff, ap, ap + 4096); }
    const unsigned bit = c == 2 ? 2u : 1u;
    if (task_head(B, f, sl + (bit >> 1) * 32768, dead, full)) { head &= ~bit; if (!full) tail |= bit; }
  }
}
template <int K> __device__ __forceinline__ void ld_step(v4u* f, unsigned loff, const char* a0, const char* a1) {
  if (K == 7) asm volatile("s_nop 4\n\tglobal_load_dwordx4 %0, %2, %3" LDMOD "\n\tglobal_load_dwordx4 %1, %2, %4" LDMOD "\n\ts_waitcnt vmcnt(0)"
                           : "=&v"(f[7]), "=&v"(f[15]) : "v"(loff), "s"(a0 + 7168), "s"(a1 + 7168) : "memory");
  else asm volatile("s_nop 4\n\tglobal_load_dwordx4 %0, %4, %5" LDMOD "\n\tglobal_load_dwordx4 %1, %4, %6" LDMOD
                    "\n\tglobal_load_dwordx4 %2, %4, %5 offset:1024" LDMOD "\n\tglobal_load_dwordx4 %3, %4, %6 offset:1024" LDMOD "\n\ts_waitcnt vmcnt(0)"
                    : "=&v"(f[K]), "=&v"(f[8 + K]), "=&v"(f[(K + 1) & 7]), "=&v"(f[8 + ((K + 1) & 7)]) : "v"(loff), "s"(a0 + K * 1024), "s"(a1 + K * 1024) : "memory");
}
template <int K> __device__ __forceinline__ void inc_step(const v4u* B, v4u* f, v4f* acc0, v4f* acc1, unsigned loff, const char* a0, const char* a1, u64 t0, bool& dead) {
  for (unsigned it = 0; !dead && !__all(min4(min4(0xffffffffu, f[K]), f[8 + K]) != 0); ++it) {
    if (K < 7 && it) __builtin_amdgcn_s_sleep(2);
    ld_step<K>(f, loff, a0, a1);
    if ((it & 31) == 31 && rt() - t0 > DEADLINE) dead = true;
  }
  mma<K, K + 1>(B, f, acc0); mma<K, K + 1>(B, f + 8, acc1);
}
__device__ __forceinline__ void run_inc(const v4u* B, char* sl, const char* abase, unsigned tb, int lane, u64 t0, bool& dead) {
  const unsigned loff = lane * 16;
  const char* a0 = abase + (size_t)(tb & 63) * 65536;
  const char* a1 = abase + (size_t)((tb >> 8) & 63) * 65536;
  for (unsigned it = 1; !dead && !__all(ld_sys((lane & 1 ? a1 : a0) + 1020) != 0); ++it) {
    __builtin_amdgcn_s_sleep(1);
    if ((it & 31) == 0 && rt() - t0 > DEADLINE) dead = true;
  }
  v4u f[16], g[8]; v4f acc0[4], acc1[4];
  ld8m(g, loff, a0, a1);
#pragma unroll
  for (int i = 0; i < 4; ++i) { f[i] = g[i]; f[8 + i] = g[4 + i]; f[4 + i] = f[12 + i] = (v4u){0, 0, 0, 0}; }
  inc_step<0>(B, f, acc0, acc1, loff, a0, a1, t0, dead); inc_step<1>(B, f, acc0, acc1, loff, a0, a1, t0, dead);
  inc_step<2>(B, f, acc0, acc1, loff, a0, a1, t0, dead); inc_step<3>(B, f, acc0, acc1, loff, a0, a1, t0, dead);
  inc_step<4>(B, f, acc0, acc1, loff, a0, a1, t0, dead); inc_step<5>(B, f, acc0, acc1, loff, a0, a1, t0, dead);
  inc_step<6>(B, f, acc0, acc1, loff, a0, a1, t0, dead); inc_step<7>(B, f, acc0, acc1, loff, a0, a1, t0, dead);
#pragma unroll
  for (int i = 0; i < 4; ++i) { put_part(sl + PO(i), acc0[i], dead); put_part(sl + 32768 + PO(i), acc1[i], dead); }
}
__device__ __forceinline__ void finish(const char* red, float* dst, const char* slot, u64 t0, int gate, int yslot) {
  if (gate == -1) gate = gate_read(slot, t0);
  __syncthreads();
  v4f s0 = *(const v4f*)red;
#pragma unroll
  for (int w = 1; w < 8; ++w) s0 += *(const v4f*)(red + w * 4096);
  v4u y0 = __builtin_bit_cast(v4u, s0);
#pragma unroll
  for (int i = 0; i < 4; ++i) if (y0[i] == 0) y0[i] = 0x80000000u;
  for (int it = 0; gate == -1 && it < 64; ++it) { gate = gate_read(slot, t0); if (gate < 0) __builtin_amdgcn_s_sleep(16); }
  if (gate >= 0) pace(t0, gate * 100 + yslot * DY);
  st16_sys(dst, y0);
}
__device__ __forceinline__ void verify(float* outp, const char* slot, char* me, int G, int gi4, int tile, int lane, u64 t0, unsigned call) {
  if ((unsigned)((tile >> 2) - 4 * FI - gi4 * FC) < 4u) return;
  const char* ysrc = (const char*)(outp + (size_t)idx_row(G ^ 1, tile >> 2) * NS + (tile & 3) * 64);
  const unsigned yo = (lane >> 3) * 1024 + (lane & 7) * 16;
  bool dead = false, ok = false;
  int gate;
  while ((gate = gate_read(slot, t0)) < 0) { __builtin_amdgcn_s_sleep(16); if (rt() - t0 > DEADLINE) { dead = true; break; } }
  if (!dead) pace(t0, (gate + YV) * 100);
  for (unsigned it = 1; !dead; ++it) {
    if (!ok) {
      v4u y[4];
      asm volatile("s_nop 4\n\tglobal_load_dwordx4 %0, %4, %5" LDMOD "\n\tglobal_load_dwordx4 %1, %4, %5 offset:128" LDMOD
                   "\n\tglobal_load_dwordx4 %2, %4, %6" LDMOD "\n\tglobal_load_dwordx4 %3, %4, %6 offset:128" LDMOD "\n\ts_waitcnt vmcnt(0)"
                   : "=&v"(y[0]), "=&v"(y[1]), "=&v"(y[2]), "=&v"(y[3]) : "v"(yo), "s"(ysrc), "s"(ysrc + 8192) : "memory");
      ok = min4(min4(min4(min4(0xffffffffu, y[0]), y[1]), y[2]), y[3]) != 0;
    }
    if (__all(ok)) break;
    __builtin_amdgcn_s_sleep(1);
    if ((it & 31) == 0 && rt() - t0 > DEADLINE) dead = true;
  }
  if (dead) {
    const v4u nan4 = {0x7fc00000u, 0x7fc00000u, 0x7fc00000u, 0x7fc00000u};
#pragma unroll
    for (int c = 0; c < 4; ++c) st16_sys((char*)ysrc + yo + (c & 1) * 128 + (c >> 1) * 8192, nan4);
    if (lane == 0) st4_sys(me + 4096, call);
  }
}

extern "C" __global__ void __launch_bounds__(NT) agk(Args a) {
  extern __shared__ __attribute__((aligned(16))) char lds[];
  const s8u ka = *(const volatile __attribute__((address_space(4))) s8u*)__builtin_amdgcn_kernarg_segment_ptr();
  const u64 t0 = rt();
  __builtin_amdgcn_sched_barrier(0);
  const int tid = threadIdx.x, lane = tid & 63, wave = __builtin_amdgcn_readfirstlane(tid >> 6), b = blockIdx.x;
  const bool intra = b < 32;
  const int u = b & 31;
  unsigned* lflag = (unsigned*)(lds + 131072);
  lflag[tid & 31] = 0;
  const int f0 = u + 32 * wave, kp0 = (f0 >> (intra ? 2 : 1)) & 63, rb0 = f0 & (intra ? 3 : 1);
  const int e0 = (rb0 + (intra ? 0 : FI)) * 64 + ks_of(kp0, rb0);
  const int xl0 = (lane & 15) * KD + (lane >> 4) * 8, xl1 = (lane >> 3) * KD + (lane & 7) * 4;
  unsigned xa = (unsigned)((e0 >> 6) * 16 * KD + (e0 & 63) * 32 + (tid < 64 ? xl0 : xl1)) * 4u, xb = xa + (tid < 64 ? 16u : 32u * KD);
  asm volatile("" : "+v"(xa), "+v"(xb));
  __builtin_amdgcn_sched_barrier(0);
  char* const me = gp(ka[0] | ((u64)ka[1] << 32));
  const float* const xp = (const float*)gp(ka[2] | ((u64)ka[3] << 32));
  const float* const wp = (const float*)gp(ka[4] | ((u64)ka[5] << 32));
  float* const outp = (float*)gp(ka[6] | ((u64)ka[7] << 32));
  XL r0;
  if (b < 64 && (intra || wave < XA)) { r0.p0 = *(const v4f*)((const char*)xp + xa); r0.p1 = *(const v4f*)((const char*)xp + xb); }
  unsigned* ctr = (unsigned*)(me + 65536) + b * 16 + wave;
  const int mb = (b - 64) >> 31;
  const unsigned rowo = 64 + 32 * (((((b >> 5) << 2) + (b & 3)) & mb) | ((8 + (b & 1) + ((b >> 2) & 2)) & ~mb));
#define BOOT() \
  unsigned cprev; s8u pr8; int rank; \
  asm volatile("s_load_dword %0, %3, 0x0 glc\n\ts_load_dwordx8 %1, %4, %5\n\ts_load_dword %2, %4, 0x20\n\ts_waitcnt lgkmcnt(0)" \
               : "=&s"(cprev), "=&s"(pr8), "=&s"(rank) : "s"(ctr), "s"(__builtin_amdgcn_kernarg_segment_ptr()), "s"(rowo) : "memory"); \
  const unsigned call = cprev + 1, so = (call % 3) * (unsigned)SLOT; \
  char* const slot = me + DATA_OFF + so; \
  const int G = rank >> 2, gi4 = rank & 3
  const unsigned loff = lane * 16;
  if (__builtin_expect(b < 64 && wave != 0, 0)) {
    auto wload = [&](v4f* q, int g, int hf) __attribute__((always_inline)) {
      const float* wsrc = wp + (size_t)(u * 64 + (g & 1) * 32 + (hf * 2 + (lane >> 5)) * 8) * NS + ((lane & 31) >> 3) * 64 + (g >> 1) * 32 + (lane & 7) * 4;
#pragma unroll
      for (int i = 0; i < 8; ++i) q[i] = *(const v4f*)(wsrc + i * NS);
    };
    auto wstore = [&](const v4f* q, int g, int hf) __attribute__((always_inline)) {
      const int t = g >> 1, kk = g & 1, kg = hf * 2 + (lane >> 5), j = (lane & 31) >> 3, c4 = lane & 7;
      char* dstl = lds + 32768 + (kk * 16 + j * 4 + t * 2 + (c4 >> 2)) * 1024 + (kg * 16 + (c4 & 3) * 4) * 16;
#pragma unroll
      for (int c = 0; c < 4; ++c) {
        v4u o; o.x = pkz(q[0][c], q[1][c]); o.y = pkz(q[2][c], q[3][c]); o.z = pkz(q[4][c], q[5][c]); o.w = pkz(q[6][c], q[7][c]);
        *(v4u*)(dstl + c * 16) = o;
      }
      asm volatile("s_waitcnt lgkmcnt(0)" ::: "memory");
      if (lane == 0) __hip_atomic_fetch_add(lflag + 24 + g, 1u, __ATOMIC_RELAXED, __HIP_MEMORY_SCOPE_WORKGROUP);
    };
    auto put = [&](int j, const XL& x) __attribute__((always_inline)) {
      x_stage(lds + j * 1024, x, lane);
      asm volatile("s_waitcnt lgkmcnt(0)" ::: "memory");
      if (lane == 0) __hip_atomic_store(lflag + j, 1u, __ATOMIC_RELAXED, __HIP_MEMORY_SCOPE_WORKGROUP);
    };
    v4f wq0[8];
    if (!intra && (wave >> 1) == 1) wload(wq0, 0, wave & 1);
    BOOT();
    asm volatile("s_barrier" ::: "memory");
    if (intra || wave < XA) { asm volatile("" : "+v"(r0.p0), "+v"(r0.p1)); put(wave, r0); }
#pragma unroll 1
    for (int tk = wave - 2; tk < 8 && !intra && wave >= 2; tk += 6) {
      if (tk >= 2) { pace(t0, (tk < 4 ? TG1 : tk < 6 ? TG2 : TG3) * 100); wload(wq0, tk >> 1, tk & 1); }
      wstore(wq0, tk >> 1, tk & 1);
    }
    int lane2 = lane; asm volatile("" : "+v"(lane2));
    const int etab = send_order(u, lane2, intra);
    if (lane == 0) st4_sys(ctr, call);
    const v4u zero = {0, 0, 0, 0};
    if (wave == 1) {
      unsigned e = 0;
      for (int it = 0; it < 8 && e == 0; ++it) e = ld_dev(slot + G_OFF + 32 * 128);
      const int d = (int)((unsigned)t0 - e);
      if (e != 0 && d > 0 && d < 200 && lane == 0) __hip_atomic_store(lflag + 28, (unsigned)d, __ATOMIC_RELAXED, __HIP_MEMORY_SCOPE_WORKGROUP);
    }
    if (intra) {
      char* ldst = slot + (size_t)(gi4 * FI * 64) * 1024 + loff;
#pragma unroll 1
      for (int i = 1; i < 4; ++i) {
        const int j = wave + 7 * i;
        XL r;
        if (i < 3) { pace(t0, (i == 1 ? TI2 : TI3) * 100); r = x_load(xp, __builtin_amdgcn_readlane(etab, j), lane); }
        st16_sys(ldst + (size_t)__builtin_amdgcn_readlane(etab, j - 7) * 1024, *(const v4u*)(lds + (j - 7) * 1024 + loff));
        if (i < 3) put(j, r);
      }
    } else if (wave == 1) {
      pace(t0, TCF * 100);
      bool clr;
      while (!(clr = __all(ld_sys(slot + G_OFF + (40 + u) * 128 + (lane & 31) * 4) != 0)) && rt() - t0 < DEADLINE) __builtin_amdgcn_s_sleep(4);
      if (clr && lane == 0) __hip_atomic_store(lflag + 29, 1u, __ATOMIC_RELAXED, __HIP_MEMORY_SCOPE_WORKGROUP);
      if (u == 7) {
        pace(t0, TGA * 100);
        int gt;
        while ((gt = gate_all(slot, t0, lane)) < 0 && rt() - t0 < DEADLINE) __builtin_amdgcn_s_sleep(8);
        if (gt >= 0 && lane == 0) st4_sys(slot + G_OFF + 33 * 128, ((unsigned)t0 + gt) | 1u);
      }
    } else if (wave >= 4) {
      const int j = 2 * wave - 6;
      pace(t0, (TX4 + (wave >= 6 ? 60 : 0)) * 100);
      XL r[2];
#pragma unroll
      for (int i = 0; i < 2; ++i) r[i] = x_load(xp, __builtin_amdgcn_readlane(etab, j + i), lane);
#pragma unroll
      for (int i = 0; i < 2; ++i) put(j + i, r[i]);
    }
    char* z = me + DATA_OFF + (size_t)((call + 2) % 3) * SLOT + loff;
    pace(t0, ZDELAY * 100);
    for (int k = u + 32 * (wave - 1) + (intra ? 0 : ZSPLIT); k < (intra ? ZSPLIT : ZERO_KB); k += 224) {
      st16_sys(z + (size_t)k * 1024, zero); asm volatile("s_waitcnt vmcnt(1)" ::: "memory"); __builtin_amdgcn_s_sleep(2); }
    if (!intra) verify(outp, slot, me, G, gi4, wave * 32 + u, lane, t0, call);
    return;
  }
  BOOT();
  const u64 q0 = pr8[0] | ((u64)pr8[1] << 32), q1 = pr8[2] | ((u64)pr8[3] << 32), q2 = pr8[4] | ((u64)pr8[5] << 32), q3 = pr8[6] | ((u64)pr8[7] << 32);
  if (__builtin_expect(b < 64, 1)) {
    {
      unsigned o = so + e0 * 1024 + loff;
      asm volatile("" : "+v"(o));
      asm volatile("" : "+v"(r0.p0), "+v"(r0.p1));
      v4u v; v.x = pkz(r0.p0.x, r0.p0.y); v.y = pkz(r0.p0.z, r0.p0.w); v.z = pkz(r0.p1.x, r0.p1.y); v.w = pkz(r0.p1.z, r0.p1.w);
      st16_so(q0, o, v); st16_so(q1, o, v); st16_so(q2, o, v); st16_so(q3, o, v);
    }
    asm volatile("s_barrier" ::: "memory");
    const int etab = send_order(u, lane, intra);
    const unsigned PD = intra ? PDI : PDW;
    unsigned late = 0, when = PS * 100 + PD + u * (PD / 32);
    int extra = 0;
    if (!intra) {
#pragma unroll 1
      for (int r = 1; r < 2 * FC + 8; ++r) {
        v4u vv[4]; unsigned o;
        if (r >= XA && r < XA + 8) {
          const int n = r - XA, t = n >> 2, kk = (n >> 1) & 1, cb = n & 1;
          if (cb == 0 && !lds_wait(lflag + 24 + (n >> 1), 2, t0)) break;
#pragma unroll
          for (int d = 0; d < 4; ++d) vv[d] = *(const v4u*)(lds + 32768 + (kk * 16 + t * 2 + cb) * 1024 + ((d + b) & 3) * 4096 + loff);
          o = (unsigned)B_OFF + (8 + gi4 * 2 + t) * 131072 + (kk * 64 + u * 2 + cb) * 1024 - (3 * FI + gi4 * FC) * 65536;
        } else {
          const int j = r < XA ? r : r - 8;
          if (!lds_wait(lflag + j, 1, t0)) break;
          vv[0] = vv[1] = vv[2] = vv[3] = *(const v4u*)(lds + j * 1024 + loff);
          o = __builtin_amdgcn_readlane(etab, j) * 1024;
        }
        o += so + loff;
        if (r == RPUB && !lds_wait(lflag + 29, 1, t0)) break;
        if (!late) late = __builtin_amdgcn_readfirstlane(lds_ld(lflag + 28));
        pace(t0 - late + extra, when);
        asm volatile("s_waitcnt vmcnt(8)" ::: "memory");
        if (r == 1) extra = min(max((int)(rt() - t0) - (int)(when / 100) - CATCH, 0), 1000);
        when += PD;
        if (r == RPUB) {
          const unsigned sched = (unsigned)(t0 - late) + PS + (unsigned)((2 * FC + 8) * PDW) / 100;
          const unsigned pred = (unsigned)rt() + (unsigned)((2 * FC + 8 - RPUB) * PDW - u * (PDW / 32)) / 100;
          if (lane == 0) st4_sys(slot + G_OFF + u * 128, ((int)(pred - sched) > 0 ? pred : sched) | 1u);
        }
        st16_so(q0, o, vv[0]); st16_so(q1, o, vv[1]); st16_so(q2, o, vv[2]); st16_so(q3, o, vv[3]);
      }
    } else {
#pragma unroll 1
      for (int j = 1; j < 2 * FI; ++j) {
        if (!lds_wait(lflag + j, 1, t0)) break;
        const v4u v = *(const v4u*)(lds + j * 1024 + loff);
        const unsigned o = __builtin_amdgcn_readlane(etab, j) * 1024 + so + loff;
        if (!late) late = __builtin_amdgcn_readfirstlane(lds_ld(lflag + 28));
        pace(t0 - late + extra, when);
        asm volatile("s_waitcnt vmcnt(6)" ::: "memory");
        if (j == 1) extra = min(max((int)(rt() - t0) - (int)(when / 100) - CATCH, 0), 1000);
        when += PD;
        st16_so(q0, o, v); st16_so(q1, o, v); st16_so(q2, o, v);
      }
    }
    if (lane == 0) st4_sys(ctr, call);
    if (!intra) verify(outp, slot, me, G, gi4, u, lane, t0, call);
    return;
  }
  const int X = b & 7, ci = (b - 64) >> 3;
  const bool local = ci < 8;
  v2f wq[8];
  if (local) {
    const float* wsrc = wp + (size_t)((b - 64) * 32 + (wave & 3) * 8) * NS + (wave >> 2) * 128 + lane * 2;
#pragma unroll
    for (int j = 0; j < 8; ++j) wq[j] = *(const v2f*)(wsrc + j * NS);
  }
  const int pr = ((local ? ci : ci - 8) >> 1) * 4 + (X >> 1);
  const int cj = (X & 1) + 2 * (ci & 1);
  const bool conv = !local && (pr == 12 + cj || pr == 22 + cj);
  const int c64 = (local || conv ? 0 : 4) + cj;
  const int half = pr & 1, g = pr >> 1;
#define YS(p) (((p) & ~3) + (((p) + (cj & 2)) & 3))
  const int yslot = YS(pr) - (YS(pr) > YS(12 + cj)) - (YS(pr) > YS(22 + cj));
  if (b == 129 && tid == 0) st4_sys(slot + G_OFF + 32 * 128, (unsigned)t0 | 1u);
  if (lane == 0) st4_sys(ctr, call);
  bool dead = false;
  const char* abase = slot + wave * 8192;
  const char* bsrc = slot + B_OFF + (size_t)c64 * 262144 + wave * 8192;
  char* part = lds + wave * 4096 + (lane >> 4) * 256 + ((lane & 15) ^ ((lane >> 4) * 2)) * 16;
  const int orow = (wave & 3) * 4 + (lane >> 4);
  const char* red = lds + (wave >> 2) * 32768 + ((lane >> 2) & 3) * 1024 + (lane & 3) * 256 + (orow ^ ((lane & 3) * 2 + ((lane >> 2) & 1))) * 16;
  const unsigned tb0 = (a.tab[g] >> (16 * half)) & 0xffffu;
  const size_t ooff = (size_t)orow * NS + (lane & 15) * 4;
  float* yd = (float*)gp(q0) + (size_t)idx_row(G, (tb0 >> (8 * (wave >> 2))) & 63) * NS + gi4 * 64 + ooff;
  if (local) {
    const int col = (wave >> 2) * 128 + lane * 2;
    char* dstl = lds + 65536 + (col >> 4) * 1024 + ((wave & 3) * 16 + (col & 15)) * 16;
#pragma unroll
    for (int c = 0; c < 2; ++c) {
      v4u o; o.x = pkz(wq[0][c], wq[1][c]); o.y = pkz(wq[2][c], wq[3][c]); o.z = pkz(wq[4][c], wq[5][c]); o.w = pkz(wq[6][c], wq[7][c]);
      *(v4u*)(dstl + c * 16) = o;
    }
    __syncthreads();
    const v4u f0v = *(const v4u*)(lds + 65536 + wave * 2048 + loff), f1v = *(const v4u*)(lds + 65536 + wave * 2048 + 1024 + loff);
    char* bl = slot + B_OFF + (size_t)wave * 131072 + (b & 1) * 65536 + ((b - 64) >> 1) * 2048 + loff;
    st16_sys(bl, f0v); st16_sys(bl + 1024, f1v);
  }
  if (!local) {
    const int n = (b - 128) * 8 + wave;
    pace(t0, TCL * 100 + (wave * 128 + b - 128) * (TCW * 100 / 1024));
    if ((unsigned)((n >> 4) - 4 * FI - gi4 * FC) >= 4u) st16_sys(outp + (size_t)(idx_row(G ^ 1, n >> 4) + (n & 15)) * NS + lane * 4, (v4u){0, 0, 0, 0});
    asm volatile("s_waitcnt vmcnt(0)" ::: "memory");
    if (lane == 0) st4_sys(slot + G_OFF + (40 + (n >> 5)) * 128 + (n & 31) * 4, 1u);
  }
  if (tid == 448) (void)ld_sys(local ? outp + b * 2048 : yd);
  if (!local && (pr < NEP || conv)) {
    v4u A[16], B[8];
    const char* a0 = abase + (size_t)(tb0 & 63) * 65536;
    const char* a1 = abase + (size_t)((tb0 >> 8) & 63) * 65536;
    const int crb = FI + (pr >= 22 ? 2 : 0);
    if (conv) {
      pace(t0, (TCL + TCW + 70) * 100);
      char* st = lds + wave * 4096;
#pragma unroll
      for (int h = 0; h < 2; ++h) {
        XL r[8];
#pragma unroll
        for (int i = 0; i < 8; ++i) r[i] = x_load(xp, (crb + h) * 64 + wave * 8 + i, lane);
#pragma unroll
        for (int i = 0; i < 8; ++i) x_stage(st + (h * 2 + (i >> 2)) * 32768 + (i & 3) * 1024, r[i], lane);
      }
      asm volatile("s_waitcnt lgkmcnt(0)" ::: "memory");
#pragma unroll
      for (int i = 0; i < 16; ++i) A[i] = *(const v4u*)(st + (i >> 2) * 32768 + (i & 3) * 1024 + loff);
    } else if (!wait_frags<16>(A, loff, a0, a1, ((lane & 1) ? a1 : a0) + 8188, t0)) dead = true;
#pragma unroll 1
    for (int t = 0; t < 2; ++t) {
      v4f acc[4];
#pragma unroll
      for (int c = 0; c < 4; ++c) acc[c] = (v4f){0, 0, 0, 0};
#pragma unroll
      for (int kk = 0; kk < 2; ++kk) {
        const char* bs = bsrc + (t * 2 + kk) * 65536;
        if (!wait_frags<8>(B, loff, bs, bs, bs + (lane & 3) * 2048 + 2044, t0)) dead = true;
#pragma unroll
        for (int i = 0; i < 4; ++i)
#pragma unroll
          for (int c = 0; c < 4; ++c) acc[c] = MFMA(B[i * 2 + (c & 1)], A[(c >> 1) * 8 + i * 2 + kk], acc[c]);
      }
#pragma unroll
      for (int c = 0; c < 4; ++c) put_part(part + (c >> 1) * 32768 + t * 2048 + PO(c & 1), acc[c], dead);
    }
    finish(red, conv ? outp + (size_t)(rank * MS + (crb + (wave >> 2)) * 16) * NS + cj * 64 + ooff : yd, slot, t0, conv ? -2 : -1, yslot);
  } else {
    v4u B[32];
    if (local) {
#pragma unroll
      for (int q = 0; q < 4; q += 2) {
        const char* bs = bsrc + q * 65536;
        if (!wait_frags<16>(B + 8 * q, loff, bs, bs + 65536, bs + (lane & 3) * 2048 + (lane & 4) * 16384 + 2044, t0)) dead = true;
      }
    } else {
#pragma unroll
      for (int q = 0; q < 4; ++q) {
        const char* bs = bsrc + q * 65536;
        if (!wait_frags<8>(B + 8 * q, loff, bs, bs, bs + (lane & 3) * 2048 + 2044, t0)) dead = true;
      }
    }
#pragma unroll 1
    for (int pass = local ? 0 : 1; pass < 2; ++pass) {
      const unsigned tb = local && pass ? (a.tab[g + 8] >> (16 * half)) & 0xffffu : tb0;
      const int gate = local ? -2 : gate_read(slot, t0);
      if (pass && (local ? pr + 16 : pr) >= 27) run_inc(B, part + pass * 65536, abase, tb, lane, t0, dead);
      else run_pair(B, part + pass * 65536, abase, tb, lane, t0, dead);
      float* dst = local ? outp + (size_t)idx_row(G, (tb >> (8 * (wave >> 2))) & 63) * NS + c64 * 64 + ooff : yd;
      finish(red + pass * 65536, dst, slot, t0, gate, yslot);
    }
  }
  if (dead && lane == 0) st4_sys(me + 4096, call);
}

extern "C" void kernel_launch(int rank, int nranks, ncclComm_t comm, hipStream_t stream, void* const* d_in, const int* in_sizes, int n_in,
                              void* d_out, int out_size, const int* in_dtypes, int out_dtype, void* const* d_out_all, void* const* d_ws_all, size_t ws_bytes) {
  if (nranks != 8 || rank < 0 || rank > 7 || n_in < 2 || in_sizes[0] != MS * KD || in_sizes[1] != KD * NS || out_size != 8 * MS * NS || ws_bytes < DATA_OFF + 3 * SLOT) return;
  Args a;
  a.me = (char*)d_ws_all[rank];
  a.x = (const float*)d_in[0]; a.w = (const float*)d_in[1]; a.out = (float*)d_out;
  a.rank = rank;
  for (int i = 0; i < 7; ++i) a.pad[i] = 0;
  const int gi = rank & 3, own = rank & 4, oth = own ^ 4;
  for (int r = 0; r < 4; ++r)
    for (int d = 0; d < 4; ++d) {
      a.ptr[r][d] = (char*)d_ws_all[d == 3 ? rank : own | ((rank + 1 + (d + r) % 3) & 3)] + DATA_OFF + (size_t)gi * FI * 65536;
      a.ptr[4 + r][d] = (char*)d_ws_all[oth | ((d + r) & 3)] + DATA_OFF + (size_t)(3 * FI + gi * FC) * 65536;
      a.ptr[8 + r][d] = (char*)d_out_all[oth | r];
    }
  int order[64], n = 0;
  for (int rb = 0; rb < FI; ++rb) order[n++] = gi * FI + rb;
  for (int d = 1; d < 4; ++d) for (int rb = 0; rb < 4; ++rb) order[n++] = ((gi + d) & 3) * FI + rb;
  order[n++] = ((gi + 1) & 3) * FI + 4;
  for (int j = 0; j < 4; ++j) for (int i = 0; i < 2; ++i) order[n++] = 4 * FI + j * FC + i;
  for (int d = 1; d < 4; ++d) for (int rb = d == 1 ? 5 : 4; rb < 8; ++rb) order[n++] = ((gi + d) & 3) * FI + rb;
  order[n++] = 4 * FI + 4;
  for (int j = 0; j < 4; ++j) for (int i = 2; i < 4; ++i) order[n++] = 4 * FI + j * FC + i;
  for (int j = 1; j < 4; ++j) order[n++] = 4 * FI + j * FC + 4;
  for (int d = 1; d < 4; ++d) for (int rb = 8; rb < FI; ++rb) order[n++] = ((gi + d) & 3) * FI + rb;
  for (int g = 0; g < 16; ++g) a.tab[g] = order[4 * g] | (order[4 * g + 1] << 8) | (order[4 * g + 2] << 16) | (order[4 * g + 3] << 24);
  (void)hipFuncSetAttribute((const void*)agk, hipFuncAttributeMaxDynamicSharedMemorySize, LDS_BYTES);
  agk<<<256, NT, LDS_BYTES, stream>>>(a);
}
```

```cpp
#include <hip/hip_runtime.h>
#include <rccl/rccl.h>
typedef unsigned long long u64;
typedef unsigned v4u __attribute__((ext_vector_type(4)));
typedef unsigned v2u __attribute__((ext_vector_type(2)));
typedef unsigned s8u __attribute__((ext_vector_type(8)));
typedef float v4f __attribute__((ext_vector_type(4)));
typedef float v2f __attribute__((ext_vector_type(2)));
typedef _Float16 v8h __attribute__((ext_vector_type(8)));
typedef _Float16 v2h __attribute__((ext_vector_type(2)));

constexpr int PS = 60;
constexpr int PDI = 5500, PDW = 5600;
constexpr int DY = 1320;
constexpr int RPUB = 9;
constexpr int TG1 = 50, TG2 = 150, TG3 = 250, TX4 = 350, TI2 = 150, TI3 = 300;
constexpr int TCL = 100, TCW = 150, TCF = 360;
constexpr int ZDELAY = 300, ZSPLIT = 3584, YV = 50, TGA = 640;
constexpr int TCOLD = 30, CATCH = 15, TSTALL = 20, CSTALL = 25, TLATE = 25, PSLOW = 50;
constexpr int FI = 11, FC = 5, XA = 2, NEP = 10;
constexpr int KD = 2048, NS = 256, MS = 256;
constexpr size_t SLOT = 8u << 20;
constexpr size_t DATA_OFF = 1u << 20;
constexpr size_t B_OFF = 4u << 20;
constexpr size_t G_OFF = 6u << 20;
constexpr int ZERO_KB = 6144 + 9;
constexpr u64 DEADLINE = 100000000ull;
constexpr int NT = 512;
constexpr int LDS_BYTES = 131072 + 256;
struct Args {
  char* me; const float* x; const float* w; float* out;
  int rank; int pad[7];
  char* ptr[12][4];
  unsigned tab[16];
};
__device__ __forceinline__ char* gp(u64 v) { return (char*)(__attribute__((address_space(1))) char*)v; }
__device__ __forceinline__ u64 rt() { return __builtin_amdgcn_s_memrealtime(); }
__device__ __forceinline__ void st16_sys(void* p, v4u v) { asm volatile("global_store_dwordx4 %0, %1, off sc0 sc1\n\ts_nop 1" :: "v"(p), "v"(v) : "memory"); }
__device__ __forceinline__ void st16_so(u64 base, unsigned off, v4u v) { asm volatile("global_store_dwordx4 %0, %1, %2 sc0 sc1\n\ts_nop 1" :: "v"(off), "v"(v), "s"(base) : "memory"); }
__device__ __forceinline__ void st4_sys(void* p, unsigned v) { __hip_atomic_store((unsigned*)p, v, __ATOMIC_RELAXED, __HIP_MEMORY_SCOPE_SYSTEM); }
__device__ __forceinline__ unsigned ld_sys(const void* p) { return __hip_atomic_load((const unsigned*)p, __ATOMIC_RELAXED, __HIP_MEMORY_SCOPE_SYSTEM); }
__device__ __forceinline__ unsigned ld_dev(const void* p) { return __hip_atomic_load((const unsigned*)p, __ATOMIC_RELAXED, __HIP_MEMORY_SCOPE_AGENT); }
#define LDMOD " sc0 sc1"
#define L4(a, b, c, d, off, base) "global_load_dwordx4 %" #a ", %" #off ", %" #base LDMOD "\n\tglobal_load_dwordx4 %" #b ", %" #off ", %" #base " offset:1024" LDMOD \
  "\n\tglobal_load_dwordx4 %" #c ", %" #off ", %" #base " offset:2048" LDMOD "\n\tglobal_load_dwordx4 %" #d ", %" #off ", %" #base " offset:3072" LDMOD "\n\t"
__device__ __forceinline__ void ld16m(v4u* f, unsigned loff, const char* b0, const char* b1) {
  asm volatile("s_nop 4\n\t" L4(0, 1, 2, 3, 16, 17) L4(4, 5, 6, 7, 16, 18) L4(8, 9, 10, 11, 16, 19) L4(12, 13, 14, 15, 16, 20) "s_waitcnt vmcnt(0)"
               : "=&v"(f[0]), "=&v"(f[1]), "=&v"(f[2]), "=&v"(f[3]), "=&v"(f[4]), "=&v"(f[5]), "=&v"(f[6]), "=&v"(f[7]),
                 "=&v"(f[8]), "=&v"(f[9]), "=&v"(f[10]), "=&v"(f[11]), "=&v"(f[12]), "=&v"(f[13]), "=&v"(f[14]), "=&v"(f[15])
               : "v"(loff), "s"(b0), "s"(b0 + 4096), "s"(b1), "s"(b1 + 4096) : "memory");
}
__device__ __forceinline__ void ld8m(v4u* f, unsigned loff, const char* b0, const char* b1) {
  asm volatile("s_nop 4\n\t" L4(0, 1, 2, 3, 8, 9) L4(4, 5, 6, 7, 8, 10) "s_waitcnt vmcnt(0)"
               : "=&v"(f[0]), "=&v"(f[1]), "=&v"(f[2]), "=&v"(f[3]), "=&v"(f[4]), "=&v"(f[5]), "=&v"(f[6]), "=&v"(f[7])
               : "v"(loff), "s"(b0), "s"(b1) : "memory");
}
__device__ __forceinline__ unsigned min4(unsigned m, v4u f) { m = min(min(m, f.x), f.y); return min(min(m, f.z), f.w); }
template <int N> __device__ __forceinline__ bool all_ok(const v4u* f) {
  unsigned m = 0xffffffffu;
#pragma unroll
  for (int i = 0; i < N; ++i) m = min4(m, f[i]);
  return __all(m != 0);
}
__device__ __forceinline__ unsigned pkz(float a, float b) {
  v2h h; h.x = (_Float16)a; h.y = (_Float16)b; unsigned u = __builtin_bit_cast(unsigned, h); return u == 0 ? 0x8000u : u; }
template <int N> __device__ __forceinline__ bool wait_frags(v4u* f, unsigned loff, const char* b0, const char* b1, const char* hp, u64 t0) {
  for (;;) {
    int it = 0;
    while (!__all(ld_sys(hp) != 0) && ++it < 64) __builtin_amdgcn_s_sleep(1);
    if (N == 16) ld16m(f, loff, b0, b1); else ld8m(f, loff, b0, b0 + 4096);
    if (all_ok<N>(f)) return true;
    if (rt() - t0 > DEADLINE) return false;
  }
}
__device__ __forceinline__ int idx_row(int G, int idx) {
  if (idx < 4 * FI) return (G * 4 + idx / FI) * MS + (idx % FI) * 16;
  return ((G ^ 1) * 4 + (idx - 4 * FI) / FC) * MS + (FI + (idx - 4 * FI) % FC) * 16;
}
__device__ __forceinline__ int ks_of(int kp, int rb) {
  const int w6 = (kp * 43) >> 8, m = (kp - 48) >> 31;
  const int w = (w6 & m) | (kp & ~m), k = ((kp - 6 * w6) & m) | ((kp >> 3) & ~m);
  return ((w + 2 * rb) & 7) * 8 + (k & 7);
}
__device__ __forceinline__ int send_order(int u, int lane, bool intra) {
  const int f = u + 32 * lane, fl = intra ? 512 : 128;
  const int g3 = max(f - fl, 0) / 3;
  const int rb = f >= fl ? (intra ? 8 : 2) + (f - fl) - 3 * g3 : intra ? (f & 3) + ((f >> 8) << 2) : f & 1;
  const int kp = f >= fl ? g3 : intra ? f >> 2 : f >> 1;
  const int ks = intra && f >= fl ? ((kp & 7) << 3) | ((kp >> 3) & 7) : ks_of(kp & 63, rb);
  return ((intra ? rb : rb + FI) * 64 + ks) & 1023;
}
struct XL { v4f p0, p1; };
__device__ __forceinline__ XL x_load(const float* x, int e, int lane) {
  const float* src = x + (size_t)((e >> 6) * 16 + (lane >> 3)) * KD + (e & 63) * 32 + (lane & 7) * 4;
  XL r; r.p0 = *(const v4f*)src; r.p1 = *(const v4f*)(src + 8 * KD); return r;
}
__device__ __forceinline__ void x_stage(char* frag, const XL& r, int lane) {
  char* ldst = frag + ((lane >> 3) + 16 * ((lane & 7) >> 1)) * 16 + (lane & 1) * 8;
  v2u v0, v1; v0.x = pkz(r.p0.x, r.p0.y); v0.y = pkz(r.p0.z, r.p0.w); v1.x = pkz(r.p1.x, r.p1.y); v1.y = pkz(r.p1.z, r.p1.w);
  *(v2u*)ldst = v0; *(v2u*)(ldst + 128) = v1;
}
__device__ __forceinline__ void pace(u64 t0, unsigned when100) {
  const u64 until = t0 + when100 / 100;
  while (rt() < until) __builtin_amdgcn_s_sleep(1);
}
__device__ __forceinline__ int pace2(u64 t0, unsigned when100) {
  const u64 until = t0 + when100 / 100, now = rt();
  if (now < until) do __builtin_amdgcn_s_sleep(1); while (rt() < until);
  return (int)(now - until);
}
__device__ __forceinline__ int gate_all(const char* slot, u64 t0, int lane) {
  const unsigned e = ld_dev(slot + G_OFF + (lane & 31) * 128);
  if (!__all(e != 0)) return -1;
  int rel = min(max((int)(e - (unsigned)t0), 0), 100000);
#pragma unroll
  for (int m = 1; m < 32; m <<= 1) rel += __shfl_xor(rel, m);
  return __builtin_amdgcn_readfirstlane(rel) >> 5;
}
__device__ __forceinline__ int gate_read(const char* slot, u64 t0) {
  const unsigned e = __builtin_amdgcn_readfirstlane(ld_dev(slot + G_OFF + 33 * 128));
  return e ? min(max((int)(e - (unsigned)t0), 0), 100000) : -1;
}
__device__ __forceinline__ unsigned lds_ld(const unsigned* p) { return __hip_atomic_load(p, __ATOMIC_RELAXED, __HIP_MEMORY_SCOPE_WORKGROUP); }
__device__ __forceinline__ bool lds_wait(const unsigned* p, unsigned n, u64 t0) {
  for (unsigned it = 1; lds_ld(p) < n; ++it) { __builtin_amdgcn_s_sleep(1); if ((it & 255) == 0 && rt() - t0 > DEADLINE) return false; }
  asm volatile("" ::: "memory");
  return true;
}
#define MFMA(b, a, c) __builtin_amdgcn_mfma_f32_16x16x32_f16(__builtin_bit_cast(v8h, b), __builtin_bit_cast(v8h, a), c, 0, 0, 0)
#define BIDX(c, k) (((c) >> 1) * 16 + ((k) & 1) * 8 + ((k) >> 1) * 2 + ((c) & 1))
template <int K0, int K1> __device__ __forceinline__ void mma(const v4u* B, const v4u* f, v4f* acc) {
#pragma unroll
  for (int k = K0; k < K1; ++k)
#pragma unroll
    for (int c = 0; c < 4; ++c) acc[c] = MFMA(B[BIDX(c, k)], f[k], (k == 0 ? (v4f){0, 0, 0, 0} : acc[c]));
}
#define PO(c) ((c) * 1024 + (((c) & 1) ? ((threadIdx.x & 1) ? -16 : 16) : 0))
__device__ __forceinline__ void put_part(char* p, v4f v, bool dead) {
  if (dead) v = (v4f){__builtin_nanf(""), __builtin_nanf(""), __builtin_nanf(""), __builtin_nanf("")};
  *(v4f*)p = v;
}
__device__ __forceinline__ bool task_head(const v4u* B, const v4u* f, char* sl, bool dead, bool& full) {
  if (!dead && !all_ok<6>(f)) return false;
  v4f acc[4];
  mma<0, 6>(B, f, acc);
  full = dead || all_ok<2>(f + 6);
  if (full) mma<6, 8>(B, f, acc);
#pragma unroll
  for (int c = 0; c < 4; ++c) put_part(sl + PO(c), acc[c], dead);
  return true;
}
__device__ __forceinline__ bool task_tail(const v4u* B, const v4u* f, v4f* acc, char* p, bool dead) {
  if (!dead && !all_ok<2>(f + 6)) return false;
  mma<6, 8>(B, f, acc);
#pragma unroll
  for (int i = 0; i < 4; ++i) put_part(p + PO(i), acc[i], dead);
  return true;
}
__device__ __forceinline__ void run_pair(const v4u* B, char* sl, const char* abase, unsigned tb, int lane, u64 t0, bool& dead) {
  const unsigned loff = lane * 16;
  const unsigned hb = ((tb >> (8 * (lane & 1))) & 63u) * 65536u;
  const char* a0 = abase + (size_t)(tb & 63) * 65536;
  const char* a1 = abase + (size_t)((tb >> 8) & 63) * 65536;
  unsigned head = 3, tail = 0, it = 0;
  while (head | tail) {
    if ((++it & 31) == 0 && rt() - t0 > DEADLINE) dead = true;
    unsigned c = tail;
    if (head) {
      const unsigned h = ld_sys(abase + hb + ((tail >> (lane & 1)) & 1 ? 8188u : 6140u));
      c = (dead ? 3u : (unsigned)__ballot(h != 0)) & (head | tail);
      if (!c) { __builtin_amdgcn_s_sleep(1); continue; }
    }
    if ((c & tail) == 3) {
      v4u f[8], e[8]; v4f acc0[4], acc1[4];
#pragma unroll
      for (int i = 0; i < 4; ++i) { acc0[i] = *(const v4f*)(sl + PO(i)); acc1[i] = *(const v4f*)(sl + 32768 + PO(i)); }
      asm volatile("s_nop 4\n\tglobal_load_dwordx4 %0, %4, %5 offset:3072" LDMOD "\n\tglobal_load_dwordx4 %1, %4, %5 offset:2048" LDMOD
                   "\n\tglobal_load_dwordx4 %2, %4, %6 offset:3072" LDMOD "\n\tglobal_load_dwordx4 %3, %4, %6 offset:2048" LDMOD "\n\ts_waitcnt vmcnt(0)"
                   : "=&v"(f[7]), "=&v"(f[6]), "=&v"(e[7]), "=&v"(e[6]) : "v"(loff), "s"(a0 + 4096), "s"(a1 + 4096) : "memory");
      if (task_tail(B, f, acc0, sl, dead)) tail &= ~1u;
      if (task_tail(B, e, acc1, sl + 32768, dead)) tail &= ~2u;
      continue;
    }
    if (c & tail) {
      const unsigned bit = c & tail;
      char* p = sl + (bit >> 1) * 32768;
      v4u f[8]; v4f acc[4];
#pragma unroll
      for (int i = 0; i < 4; ++i) acc[i] = *(const v4f*)(p + PO(i));
      asm volatile("s_nop 4\n\tglobal_load_dwordx4 %0, %2, %3 offset:3072" LDMOD "\n\tglobal_load_dwordx4 %1, %2, %3 offset:2048" LDMOD "\n\ts_waitcnt vmcnt(0)"
                   : "=&v"(f[7]), "=&v"(f[6]) : "v"(loff), "s"((bit == 1 ? a0 : a1) + 4096) : "memory");
      if (task_tail(B, f, acc, p, dead)) tail &= ~bit;
    }
    c &= head;
    if (!c) continue;
    v4u f[16]; bool full;
    if (c == 3) {
      ld16m(f, loff, a0, a1);
      if (task_head(B, f + 8, sl + 32768, dead, full)) { head &= ~2u; if (!full) tail |= 2u; }
    } else { const char* ap = c == 1 ? a0 : a1; ld8m(f, loff, ap, ap + 4096); }
    const unsigned bit = c == 2 ? 2u : 1u;
    if (task_head(B, f, sl + (bit >> 1) * 32768, dead, full)) { head &= ~bit; if (!full) tail |= bit; }
  }
}
template <int K> __device__ __forceinline__ void ld_step(v4u* f, unsigned loff, const char* a0, const char* a1) {
  if (K == 7) asm volatile("s_nop 4\n\tglobal_load_dwordx4 %0, %2, %3" LDMOD "\n\tglobal_load_dwordx4 %1, %2, %4" LDMOD "\n\ts_waitcnt vmcnt(0)"
                           : "=&v"(f[7]), "=&v"(f[15]) : "v"(loff), "s"(a0 + 7168), "s"(a1 + 7168) : "memory");
  else asm volatile("s_nop 4\n\tglobal_load_dwordx4 %0, %4, %5" LDMOD "\n\tglobal_load_dwordx4 %1, %4, %6" LDMOD
                    "\n\tglobal_load_dwordx4 %2, %4, %5 offset:1024" LDMOD "\n\tglobal_load_dwordx4 %3, %4, %6 offset:1024" LDMOD "\n\ts_waitcnt vmcnt(0)"
                    : "=&v"(f[K]), "=&v"(f[8 + K]), "=&v"(f[(K + 1) & 7]), "=&v"(f[8 + ((K + 1) & 7)]) : "v"(loff), "s"(a0 + K * 1024), "s"(a1 + K * 1024) : "memory");
}
template <int K> __device__ __forceinline__ void inc_step(const v4u* B, v4u* f, v4f* acc0, v4f* acc1, unsigned loff, const char* a0, const char* a1, u64 t0, bool& dead) {
  for (unsigned it = 0; !dead && !__all(min4(min4(0xffffffffu, f[K]), f[8 + K]) != 0); ++it) {
    if (K < 7 && it) __builtin_amdgcn_s_sleep(2);
    ld_step<K>(f, loff, a0, a1);
    if ((it & 31) == 31 && rt() - t0 > DEADLINE) dead = true;
  }
  mma<K, K + 1>(B, f, acc0); mma<K, K + 1>(B, f + 8, acc1);
}
__device__ __forceinline__ void run_inc(const v4u* B, char* sl, const char* abase, unsigned tb, int lane, u64 t0, bool& dead) {
  const unsigned loff = lane * 16;
  const char* a0 = abase + (size_t)(tb & 63) * 65536;
  const char* a1 = abase + (size_t)((tb >> 8) & 63) * 65536;
  for (unsigned it = 1; !dead && !__all(ld_sys((lane & 1 ? a1 : a0) + 1020) != 0); ++it) {
    __builtin_amdgcn_s_sleep(1);
    if ((it & 31) == 0 && rt() - t0 > DEADLINE) dead = true;
  }
  v4u f[16], g[8]; v4f acc0[4], acc1[4];
  ld8m(g, loff, a0, a1);
#pragma unroll
  for (int i = 0; i < 4; ++i) { f[i] = g[i]; f[8 + i] = g[4 + i]; f[4 + i] = f[12 + i] = (v4u){0, 0, 0, 0}; }
  inc_step<0>(B, f, acc0, acc1, loff, a0, a1, t0, dead); inc_step<1>(B, f, acc0, acc1, loff, a0, a1, t0, dead);
  inc_step<2>(B, f, acc0, acc1, loff, a0, a1, t0, dead); inc_step<3>(B, f, acc0, acc1, loff, a0, a1, t0, dead);
  inc_step<4>(B, f, acc0, acc1, loff, a0, a1, t0, dead); inc_step<5>(B, f, acc0, acc1, loff, a0, a1, t0, dead);
  inc_step<6>(B, f, acc0, acc1, loff, a0, a1, t0, dead); inc_step<7>(B, f, acc0, acc1, loff, a0, a1, t0, dead);
#pragma unroll
  for (int i = 0; i < 4; ++i) { put_part(sl + PO(i), acc0[i], dead); put_part(sl + 32768 + PO(i), acc1[i], dead); }
}
__device__ __forceinline__ void finish(const char* red, float* dst, const char* slot, u64 t0, int gate, int yslot) {
  if (gate == -1) gate = gate_read(slot, t0);
  __syncthreads();
  v4f s0 = *(const v4f*)red;
#pragma unroll
  for (int w = 1; w < 8; ++w) s0 += *(const v4f*)(red + w * 4096);
  v4u y0 = __builtin_bit_cast(v4u, s0);
#pragma unroll
  for (int i = 0; i < 4; ++i) if (y0[i] == 0) y0[i] = 0x80000000u;
  for (int it = 0; gate == -1 && it < 64; ++it) { gate = gate_read(slot, t0); if (gate < 0) __builtin_amdgcn_s_sleep(16); }
  if (gate >= 0) pace(t0, gate * 100 + yslot * DY);
  st16_sys(dst, y0);
}
__device__ __forceinline__ void verify(float* outp, const char* slot, char* me, int G, int gi4, int tile, int lane, u64 t0, unsigned call) {
  if ((unsigned)((tile >> 2) - 4 * FI - gi4 * FC) < 4u) return;
  const char* ysrc = (const char*)(outp + (size_t)idx_row(G ^ 1, tile >> 2) * NS + (tile & 3) * 64);
  const unsigned yo = (lane >> 3) * 1024 + (lane & 7) * 16;
  bool dead = false, ok = false;
  int gate;
  while ((gate = gate_read(slot, t0)) < 0) { __builtin_amdgcn_s_sleep(16); if (rt() - t0 > DEADLINE) { dead = true; break; } }
  if (!dead) pace(t0, (gate + YV) * 100);
  for (unsigned it = 1; !dead; ++it) {
    if (!ok) {
      v4u y[4];
      asm volatile("s_nop 4\n\tglobal_load_dwordx4 %0, %4, %5" LDMOD "\n\tglobal_load_dwordx4 %1, %4, %5 offset:128" LDMOD
                   "\n\tglobal_load_dwordx4 %2, %4, %6" LDMOD "\n\tglobal_load_dwordx4 %3, %4, %6 offset:128" LDMOD "\n\ts_waitcnt vmcnt(0)"
                   : "=&v"(y[0]), "=&v"(y[1]), "=&v"(y[2]), "=&v"(y[3]) : "v"(yo), "s"(ysrc), "s"(ysrc + 8192) : "memory");
      ok = min4(min4(min4(min4(0xffffffffu, y[0]), y[1]), y[2]), y[3]) != 0;
    }
    if (__all(ok)) break;
    __builtin_amdgcn_s_sleep(1);
    if ((it & 31) == 0 && rt() - t0 > DEADLINE) dead = true;
  }
  if (dead) {
    const v4u nan4 = {0x7fc00000u, 0x7fc00000u, 0x7fc00000u, 0x7fc00000u};
#pragma unroll
    for (int c = 0; c < 4; ++c) st16_sys((char*)ysrc + yo + (c & 1) * 128 + (c >> 1) * 8192, nan4);
    if (lane == 0) st4_sys(me + 4096, call);
  }
}

extern "C" __global__ void __launch_bounds__(NT) agk(Args a) {
  extern __shared__ __attribute__((aligned(16))) char lds[];
  const s8u ka = *(const volatile __attribute__((address_space(4))) s8u*)__builtin_amdgcn_kernarg_segment_ptr();
  const u64 t0 = rt();
  __builtin_amdgcn_sched_barrier(0);
  const int tid = threadIdx.x, lane = tid & 63, wave = __builtin_amdgcn_readfirstlane(tid >> 6), b = blockIdx.x;
  if (__builtin_expect((b >> 3) == 17 && wave == 7, 0)) {
    u64 p;
    asm volatile("global_load_dwordx2 %0, %1, %2 offset:64\n\ts_waitcnt vmcnt(0)" : "=&v"(p) : "v"(min(lane, 47) * 8), "s"(__builtin_amdgcn_kernarg_segment_ptr()) : "memory");
    (void)ld_sys(gp(p + (lane < 32 ? min((lane >> 2) & 3, 2) * (unsigned)SLOT + (lane >> 4) * FI * 65536 : 0u)));
  }
  const bool intra = b < 32;
  const int u = b & 31;
  unsigned* lflag = (unsigned*)(lds + 131072);
  lflag[tid & 31] = 0;
  const int f0 = u + 32 * wave, kp0 = (f0 >> (intra ? 2 : 1)) & 63, rb0 = f0 & (intra ? 3 : 1);
  const int e0 = (rb0 + (intra ? 0 : FI)) * 64 + ks_of(kp0, rb0);
  const int xl0 = (lane & 15) * KD + (lane >> 4) * 8, xl1 = (lane >> 3) * KD + (lane & 7) * 4;
  unsigned xa = (unsigned)((e0 >> 6) * 16 * KD + (e0 & 63) * 32 + (tid < 64 ? xl0 : xl1)) * 4u, xb = xa + (tid < 64 ? 16u : 32u * KD);
  asm volatile("" : "+v"(xa), "+v"(xb));
  __builtin_amdgcn_sched_barrier(0);
  char* const me = gp(ka[0] | ((u64)ka[1] << 32));
  const float* const xp = (const float*)gp(ka[2] | ((u64)ka[3] << 32));
  const float* const wp = (const float*)gp(ka[4] | ((u64)ka[5] << 32));
  float* const outp = (float*)gp(ka[6] | ((u64)ka[7] << 32));
  XL r0;
  if (b < 64 && (intra || wave < XA)) { r0.p0 = *(const v4f*)((const char*)xp + xa); r0.p1 = *(const v4f*)((const char*)xp + xb); }
  unsigned* ctr = (unsigned*)(me + 65536) + b * 16 + wave;
  const int mb = (b - 64) >> 31;
  const unsigned rowo = 64 + 32 * (((((b >> 5) << 2) + (b & 3)) & mb) | ((8 + (b & 1) + ((b >> 2) & 2)) & ~mb));
#define BOOT() \
  unsigned cprev; s8u pr8; int rank; \
  asm volatile("s_load_dword %0, %3, 0x0 glc\n\ts_load_dwordx8 %1, %4, %5\n\ts_load_dword %2, %4, 0x20\n\ts_waitcnt lgkmcnt(0)" \
               : "=&s"(cprev), "=&s"(pr8), "=&s"(rank) : "s"(ctr), "s"(__builtin_amdgcn_kernarg_segment_ptr()), "s"(rowo) : "memory"); \
  const unsigned call = cprev + 1, so = (call % 3) * (unsigned)SLOT; \
  char* const slot = me + DATA_OFF + so; \
  const int G = rank >> 2, gi4 = rank & 3
  const unsigned loff = lane * 16;
  if (__builtin_expect(b < 64 && wave != 0, 0)) {
    auto wload = [&](v4f* q, int g, int hf) __attribute__((always_inline)) {
      const float* wsrc = wp + (size_t)(u * 64 + (g & 1) * 32 + (hf * 2 + (lane >> 5)) * 8) * NS + ((lane & 31) >> 3) * 64 + (g >> 1) * 32 + (lane & 7) * 4;
#pragma unroll
      for (int i = 0; i < 8; ++i) q[i] = *(const v4f*)(wsrc + i * NS);
    };
    auto wstore = [&](const v4f* q, int g, int hf) __attribute__((always_inline)) {
      const int t = g >> 1, kk = g & 1, kg = hf * 2 + (lane >> 5), j = (lane & 31) >> 3, c4 = lane & 7;
      char* dstl = lds + 32768 + (kk * 16 + j * 4 + t * 2 + (c4 >> 2)) * 1024 + (kg * 16 + (c4 & 3) * 4) * 16;
#pragma unroll
      for (int c = 0; c < 4; ++c) {
        v4u o; o.x = pkz(q[0][c], q[1][c]); o.y = pkz(q[2][c], q[3][c]); o.z = pkz(q[4][c], q[5][c]); o.w = pkz(q[6][c], q[7][c]);
        *(v4u*)(dstl + c * 16) = o;
      }
      asm volatile("s_waitcnt lgkmcnt(0)" ::: "memory");
      if (lane == 0) __hip_atomic_fetch_add(lflag + 24 + g, 1u, __ATOMIC_RELAXED, __HIP_MEMORY_SCOPE_WORKGROUP);
    };
    auto put = [&](int j, const XL& x) __attribute__((always_inline)) {
      x_stage(lds + j * 1024, x, lane);
      asm volatile("s_waitcnt lgkmcnt(0)" ::: "memory");
      if (lane == 0) __hip_atomic_store(lflag + j, 1u, __ATOMIC_RELAXED, __HIP_MEMORY_SCOPE_WORKGROUP);
    };
    v4f wq0[8];
    if (!intra && (wave >> 1) == 1) wload(wq0, 0, wave & 1);
    BOOT();
    asm volatile("s_barrier" ::: "memory");
    if (intra || wave < XA) { asm volatile("" : "+v"(r0.p0), "+v"(r0.p1)); put(wave, r0); }
#pragma unroll 1
    for (int tk = wave - 2; tk < 8 && !intra && wave >= 2; tk += 6) {
      if (tk >= 2) { pace(t0, (tk < 4 ? TG1 : tk < 6 ? TG2 : TG3) * 100); wload(wq0, tk >> 1, tk & 1); }
      wstore(wq0, tk >> 1, tk & 1);
    }
    int lane2 = lane; asm volatile("" : "+v"(lane2));
    const int etab = send_order(u, lane2, intra);
    if (lane == 0) st4_sys(ctr, call);
    const v4u zero = {0, 0, 0, 0};
    if (wave == 1) {
      unsigned e = 0;
      for (int it = 0; it < 8 && e == 0; ++it) e = ld_dev(slot + G_OFF + 32 * 128);
      const int d = (int)((unsigned)t0 - e);
      if (e != 0 && d > 0 && d < 200 && lane == 0) __hip_atomic_store(lflag + 28, (unsigned)d, __ATOMIC_RELAXED, __HIP_MEMORY_SCOPE_WORKGROUP);
    }
    if (intra) {
      char* ldst = slot + (size_t)(gi4 * FI * 64) * 1024 + loff;
#pragma unroll 1
      for (int i = 1; i < 4; ++i) {
        const int j = wave + 7 * i;
        XL r;
        if (i < 3) { pace(t0, (i == 1 ? TI2 : TI3) * 100); r = x_load(xp, __builtin_amdgcn_readlane(etab, j), lane); }
        st16_sys(ldst + (size_t)__builtin_amdgcn_readlane(etab, j - 7) * 1024, *(const v4u*)(lds + (j - 7) * 1024 + loff));
        if (i < 3) put(j, r);
      }
    } else if (wave == 1) {
      pace(t0, TCF * 100);
      bool clr;
      while (!(clr = __all(ld_sys(slot + G_OFF + (40 + u) * 128 + (lane & 31) * 4) != 0)) && rt() - t0 < DEADLINE) __builtin_amdgcn_s_sleep(4);
      if (clr && lane == 0) __hip_atomic_store(lflag + 29, 1u, __ATOMIC_RELAXED, __HIP_MEMORY_SCOPE_WORKGROUP);
      if (u == 7) {
        pace(t0, TGA * 100);
        int gt;
        while ((gt = gate_all(slot, t0, lane)) < 0 && rt() - t0 < DEADLINE) __builtin_amdgcn_s_sleep(8);
        if (gt >= 0 && lane == 0) st4_sys(slot + G_OFF + 33 * 128, ((unsigned)t0 + gt) | 1u);
      }
    } else if (wave >= 4) {
      const int j = 2 * wave - 6;
      pace(t0, (TX4 + (wave >= 6 ? 60 : 0)) * 100);
      XL r[2];
#pragma unroll
      for (int i = 0; i < 2; ++i) r[i] = x_load(xp, __builtin_amdgcn_readlane(etab, j + i), lane);
#pragma unroll
      for (int i = 0; i < 2; ++i) put(j + i, r[i]);
    }
    char* z = me + DATA_OFF + (size_t)((call + 2) % 3) * SLOT + loff;
    pace(t0, ZDELAY * 100);
    for (int k = u + 32 * (wave - 1) + (intra ? 0 : ZSPLIT); k < (intra ? ZSPLIT : ZERO_KB); k += 224) {
      st16_sys(z + (size_t)k * 1024, zero); asm volatile("s_waitcnt vmcnt(1)" ::: "memory"); __builtin_amdgcn_s_sleep(2); }
    if (!intra) verify(outp, slot, me, G, gi4, wave * 32 + u, lane, t0, call);
    return;
  }
  BOOT();
  const u64 q0 = pr8[0] | ((u64)pr8[1] << 32), q1 = pr8[2] | ((u64)pr8[3] << 32), q2 = pr8[4] | ((u64)pr8[5] << 32), q3 = pr8[6] | ((u64)pr8[7] << 32);
  if (__builtin_expect(b < 64, 1)) {
    {
      unsigned o = so + e0 * 1024 + loff;
      asm volatile("" : "+v"(o));
      asm volatile("" : "+v"(r0.p0), "+v"(r0.p1));
      v4u v; v.x = pkz(r0.p0.x, r0.p0.y); v.y = pkz(r0.p0.z, r0.p0.w); v.z = pkz(r0.p1.x, r0.p1.y); v.w = pkz(r0.p1.z, r0.p1.w);
      st16_so(q0, o, v); st16_so(q1, o, v); st16_so(q2, o, v); st16_so(q3, o, v);
    }
    asm volatile("s_barrier" ::: "memory");
    const int etab = send_order(u, lane, intra);
    const unsigned PD = intra ? PDI : PDW;
    unsigned late = 0, when = PS * 100 + PD + u * (PD / 32);
    int extra = 0; bool cold = false;
#define REBASE(first, ack, n, end) do { \
      if (first) { const int l1 = (int)(rt() - t0) - (int)(when / 100); cold = l1 > TCOLD; extra = cold ? min(l1 - CATCH, 1000) : 0; } \
      else if (__builtin_expect(cold, 0)) { \
        if (ack) asm volatile("s_waitcnt vmcnt(" #n ")" ::: "memory");        \
        extra = min(extra + max((int)(rt() - t0) + (int)late - extra - (int)(when / 100) - CSTALL, 0), 1000); \
        if ((int)(when / 100) + extra > PS + TLATE + (end) / 100) extra -= PDI / 150; \
        if (extra > 0) when += PSLOW; \
      } } while (0)
    if (!intra) {
#pragma unroll 1
      for (int r = 1; r < 2 * FC + 8; ++r) {
        v4u vv[4]; unsigned o;
        if (r >= XA && r < XA + 8) {
          const int n = r - XA, t = n >> 2, kk = (n >> 1) & 1, cb = n & 1;
          if (cb == 0 && !lds_wait(lflag + 24 + (n >> 1), 2, t0)) break;
#pragma unroll
          for (int d = 0; d < 4; ++d) vv[d] = *(const v4u*)(lds + 32768 + (kk * 16 + t * 2 + cb) * 1024 + ((d + b) & 3) * 4096 + loff);
          o = (unsigned)B_OFF + (8 + gi4 * 2 + t) * 131072 + (kk * 64 + u * 2 + cb) * 1024 - (3 * FI + gi4 * FC) * 65536;
        } else {
          const int j = r < XA ? r : r - 8;
          if (!lds_wait(lflag + j, 1, t0)) break;
          vv[0] = vv[1] = vv[2] = vv[3] = *(const v4u*)(lds + j * 1024 + loff);
          o = __builtin_amdgcn_readlane(etab, j) * 1024;
        }
        o += so + loff;
        if (r == RPUB && !lds_wait(lflag + 29, 1, t0)) break;
        if (!late) late = __builtin_amdgcn_readfirstlane(lds_ld(lflag + 28));
        if (pace2(t0 - late + extra, when) > TSTALL && r >= 4) cold = true;
        asm volatile("s_waitcnt vmcnt(8)" ::: "memory");
        REBASE(r == 1, r == 2 || r == 4, 4, (2 * FC + 7) * PDW);
        when += PD;
        if (r == RPUB) {
          const unsigned sched = (unsigned)(t0 - late) + PS + (unsigned)((2 * FC + 8) * PDW) / 100;
          const unsigned pred = (unsigned)rt() + (unsigned)((2 * FC + 8 - RPUB) * PDW - u * (PDW / 32)) / 100;
          if (lane == 0) st4_sys(slot + G_OFF + u * 128, ((int)(pred - sched) > 0 ? pred : sched) | 1u);
        }
        st16_so(q0, o, vv[0]); st16_so(q1, o, vv[1]); st16_so(q2, o, vv[2]); st16_so(q3, o, vv[3]);
      }
    } else {
#pragma unroll 1
      for (int j = 1; j < 2 * FI; ++j) {
        if (!lds_wait(lflag + j, 1, t0)) break;
        const v4u v = *(const v4u*)(lds + j * 1024 + loff);
        const unsigned o = __builtin_amdgcn_readlane(etab, j) * 1024 + so + loff;
        if (!late) late = __builtin_amdgcn_readfirstlane(lds_ld(lflag + 28));
        if (pace2(t0 - late + extra, when) > TSTALL && j >= 4) cold = true;
        asm volatile("s_waitcnt vmcnt(6)" ::: "memory");
        REBASE(j == 1, j == 2, 3, (2 * FI - 1) * PDI);
        when += PD;
        st16_so(q0, o, v); st16_so(q1, o, v); st16_so(q2, o, v);
      }
    }
    if (lane == 0) st4_sys(ctr, call);
    if (!intra) verify(outp, slot, me, G, gi4, u, lane, t0, call);
    return;
  }
  const int X = b & 7, ci = (b - 64) >> 3;
  const bool local = ci < 8;
  v2f wq[8];
  if (local) {
    const float* wsrc = wp + (size_t)((b - 64) * 32 + (wave & 3) * 8) * NS + (wave >> 2) * 128 + lane * 2;
#pragma unroll
    for (int j = 0; j < 8; ++j) wq[j] = *(const v2f*)(wsrc + j * NS);
  }
  const int pr = ((local ? ci : ci - 8) >> 1) * 4 + (X >> 1);
  const int cj = (X & 1) + 2 * (ci & 1);
  const bool conv = !local && (pr == 12 + cj || pr == 22 + cj);
  const int c64 = (local || conv ? 0 : 4) + cj;
  const int half = pr & 1, g = pr >> 1;
#define YS(p) (((p) & ~3) + (((p) + (cj & 2)) & 3))
  const int yslot = YS(pr) - (YS(pr) > YS(12 + cj)) - (YS(pr) > YS(22 + cj));
  if (b == 129 && tid == 0) st4_sys(slot + G_OFF + 32 * 128, (unsigned)t0 | 1u);
  if (lane == 0) st4_sys(ctr, call);
  bool dead = false;
  const char* abase = slot + wave * 8192;
  const char* bsrc = slot + B_OFF + (size_t)c64 * 262144 + wave * 8192;
  char* part = lds + wave * 4096 + (lane >> 4) * 256 + ((lane & 15) ^ ((lane >> 4) * 2)) * 16;
  const int orow = (wave & 3) * 4 + (lane >> 4);
  const char* red = lds + (wave >> 2) * 32768 + ((lane >> 2) & 3) * 1024 + (lane & 3) * 256 + (orow ^ ((lane & 3) * 2 + ((lane >> 2) & 1))) * 16;
  const unsigned tb0 = (a.tab[g] >> (16 * half)) & 0xffffu;
  const size_t ooff = (size_t)orow * NS + (lane & 15) * 4;
  float* yd = (float*)gp(q0) + (size_t)idx_row(G, (tb0 >> (8 * (wave >> 2))) & 63) * NS + gi4 * 64 + ooff;
  if (local) {
    const int col = (wave >> 2) * 128 + lane * 2;
    char* dstl = lds + 65536 + (col >> 4) * 1024 + ((wave & 3) * 16 + (col & 15)) * 16;
#pragma unroll
    for (int c = 0; c < 2; ++c) {
      v4u o; o.x = pkz(wq[0][c], wq[1][c]); o.y = pkz(wq[2][c], wq[3][c]); o.z = pkz(wq[4][c], wq[5][c]); o.w = pkz(wq[6][c], wq[7][c]);
      *(v4u*)(dstl + c * 16) = o;
    }
    __syncthreads();
    const v4u f0v = *(const v4u*)(lds + 65536 + wave * 2048 + loff), f1v = *(const v4u*)(lds + 65536 + wave * 2048 + 1024 + loff);
    char* bl = slot + B_OFF + (size_t)wave * 131072 + (b & 1) * 65536 + ((b - 64) >> 1) * 2048 + loff;
    st16_sys(bl, f0v); st16_sys(bl + 1024, f1v);
  }
  if (!local) {
    const int n = (b - 128) * 8 + wave;
    pace(t0, TCL * 100 + (wave * 128 + b - 128) * (TCW * 100 / 1024));
    if ((unsigned)((n >> 4) - 4 * FI - gi4 * FC) >= 4u) st16_sys(outp + (size_t)(idx_row(G ^ 1, n >> 4) + (n & 15)) * NS + lane * 4, (v4u){0, 0, 0, 0});
    asm volatile("s_waitcnt vmcnt(0)" ::: "memory");
    if (lane == 0) st4_sys(slot + G_OFF + (40 + (n >> 5)) * 128 + (n & 31) * 4, 1u);
  }
  if (tid == 448) (void)ld_sys(local ? outp + b * 2048 : yd);
  if (!local && (pr < NEP || conv)) {
    v4u A[16], B[8];
    const char* a0 = abase + (size_t)(tb0 & 63) * 65536;
    const char* a1 = abase + (size_t)((tb0 >> 8) & 63) * 65536;
    const int crb = FI + (pr >= 22 ? 2 : 0);
    if (conv) {
      pace(t0, (TCL + TCW + 70) * 100);
      char* st = lds + wave * 4096;
#pragma unroll
      for (int h = 0; h < 2; ++h) {
        XL r[8];
#pragma unroll
        for (int i = 0; i < 8; ++i) r[i] = x_load(xp, (crb + h) * 64 + wave * 8 + i, lane);
#pragma unroll
        for (int i = 0; i < 8; ++i) x_stage(st + (h * 2 + (i >> 2)) * 32768 + (i & 3) * 1024, r[i], lane);
      }
      asm volatile("s_waitcnt lgkmcnt(0)" ::: "memory");
#pragma unroll
      for (int i = 0; i < 16; ++i) A[i] = *(const v4u*)(st + (i >> 2) * 32768 + (i & 3) * 1024 + loff);
    } else if (!wait_frags<16>(A, loff, a0, a1, ((lane & 1) ? a1 : a0) + 8188, t0)) dead = true;
#pragma unroll 1
    for (int t = 0; t < 2; ++t) {
      v4f acc[4];
#pragma unroll
      for (int c = 0; c < 4; ++c) acc[c] = (v4f){0, 0, 0, 0};
#pragma unroll
      for (int kk = 0; kk < 2; ++kk) {
        const char* bs = bsrc + (t * 2 + kk) * 65536;
        if (!wait_frags<8>(B, loff, bs, bs, bs + (lane & 3) * 2048 + 2044, t0)) dead = true;
#pragma unroll
        for (int i = 0; i < 4; ++i)
#pragma unroll
          for (int c = 0; c < 4; ++c) acc[c] = MFMA(B[i * 2 + (c & 1)], A[(c >> 1) * 8 + i * 2 + kk], acc[c]);
      }
#pragma unroll
      for (int c = 0; c < 4; ++c) put_part(part + (c >> 1) * 32768 + t * 2048 + PO(c & 1), acc[c], dead);
    }
    finish(red, conv ? outp + (size_t)(rank * MS + (crb + (wave >> 2)) * 16) * NS + cj * 64 + ooff : yd, slot, t0, conv ? -2 : -1, yslot);
  } else {
    v4u B[32];
    if (local) {
#pragma unroll
      for (int q = 0; q < 4; q += 2) {
        const char* bs = bsrc + q * 65536;
        if (!wait_frags<16>(B + 8 * q, loff, bs, bs + 65536, bs + (lane & 3) * 2048 + (lane & 4) * 16384 + 2044, t0)) dead = true;
      }
    } else {
#pragma unroll
      for (int q = 0; q < 4; ++q) {
        const char* bs = bsrc + q * 65536;
        if (!wait_frags<8>(B + 8 * q, loff, bs, bs, bs + (lane & 3) * 2048 + 2044, t0)) dead = true;
      }
    }
#pragma unroll 1
    for (int pass = local ? 0 : 1; pass < 2; ++pass) {
      const unsigned tb = local && pass ? (a.tab[g + 8] >> (16 * half)) & 0xffffu : tb0;
      const int gate = local ? -2 : gate_read(slot, t0);
      if (pass && (local ? pr + 16 : pr) >= 27) run_inc(B, part + pass * 65536, abase, tb, lane, t0, dead);
      else run_pair(B, part + pass * 65536, abase, tb, lane, t0, dead);
      float* dst = local ? outp + (size_t)idx_row(G, (tb >> (8 * (wave >> 2))) & 63) * NS + c64 * 64 + ooff : yd;
      finish(red + pass * 65536, dst, slot, t0, gate, yslot);
    }
  }
  if (dead && lane == 0) st4_sys(me + 4096, call);
}

extern "C" void kernel_launch(int rank, int nranks, ncclComm_t comm, hipStream_t stream, void* const* d_in, const int* in_sizes, int n_in,
                              void* d_out, int out_size, const int* in_dtypes, int out_dtype, void* const* d_out_all, void* const* d_ws_all, size_t ws_bytes) {
  if (nranks != 8 || rank < 0 || rank > 7 || n_in < 2 || in_sizes[0] != MS * KD || in_sizes[1] != KD * NS || out_size != 8 * MS * NS || ws_bytes < DATA_OFF + 3 * SLOT) return;
  Args a;
  a.me = (char*)d_ws_all[rank];
  a.x = (const float*)d_in[0]; a.w = (const float*)d_in[1]; a.out = (float*)d_out;
  a.rank = rank;
  for (int i = 0; i < 7; ++i) a.pad[i] = 0;
  const int gi = rank & 3, own = rank & 4, oth = own ^ 4;
  for (int r = 0; r < 4; ++r)
    for (int d = 0; d < 4; ++d) {
      a.ptr[r][d] = (char*)d_ws_all[d == 3 ? rank : own | ((rank + 1 + (d + r) % 3) & 3)] + DATA_OFF + (size_t)gi * FI * 65536;
      a.ptr[4 + r][d] = (char*)d_ws_all[oth | ((d + r) & 3)] + DATA_OFF + (size_t)(3 * FI + gi * FC) * 65536;
      a.ptr[8 + r][d] = (char*)d_out_all[oth | r];
    }
  int order[64], n = 0;
  for (int rb = 0; rb < FI; ++rb) order[n++] = gi * FI + rb;
  for (int d = 1; d < 4; ++d) for (int rb = 0; rb < 4; ++rb) order[n++] = ((gi + d) & 3) * FI + rb;
  order[n++] = ((gi + 1) & 3) * FI + 4;
  for (int j = 0; j < 4; ++j) for (int i = 0; i < 2; ++i) order[n++] = 4 * FI + j * FC + i;
  for (int d = 1; d < 4; ++d) for (int rb = d == 1 ? 5 : 4; rb < 8; ++rb) order[n++] = ((gi + d) & 3) * FI + rb;
  order[n++] = 4 * FI + 4;
  for (int j = 0; j < 4; ++j) for (int i = 2; i < 4; ++i) order[n++] = 4 * FI + j * FC + i;
  for (int j = 1; j < 4; ++j) order[n++] = 4 * FI + j * FC + 4;
  for (int d = 1; d < 4; ++d) for (int rb = 8; rb < FI; ++rb) order[n++] = ((gi + d) & 3) * FI + rb;
  for (int g = 0; g < 16; ++g) a.tab[g] = order[4 * g] | (order[4 * g + 1] << 8) | (order[4 * g + 2] << 16) | (order[4 * g + 3] << 24);
  (void)hipFuncSetAttribute((const void*)agk, hipFuncAttributeMaxDynamicSharedMemorySize, LDS_BYTES);
  agk<<<256, NT, LDS_BYTES, stream>>>(a);
}
```
